```python
import numpy as np

kernel_name = "gpu_gemm_rs_m4096_k4096_n4096_e4m3e4m3_dqsilu"


ABSMAX_RTOL = 0.03

COMPUTE_DTYPE = "float32"
EPILOGUE = "dequant_silu"
IN_QUANT = None
OUT_QUANT = None
IN_FP8 = "e4m3e4m3"
IN_DTYPES = {"x": "e4m3", "w_mat": "e4m3",
             "scale_x": "float32", "scale_w": "float32"}
WORLD_SIZE = 8

def _snap_fp8(x, mant_bits, max_abs, min_normal):
    a = np.abs(x.astype(np.float64))
    _m, _e = np.frexp(a)
    q = np.where(a >= min_normal, np.ldexp(1.0, _e - 1 - mant_bits),
                 min_normal * 2.0 ** -mant_bits)
    snapped = np.minimum(np.round(a / q) * q, max_abs)
    return (np.sign(x) * snapped).astype(np.float32)

def setup_inputs():

    x = _snap_fp8(np.random.randn(4096, 4096).astype(np.float32), 3, 448.0, 2.0 ** -6)
    w_mat = _snap_fp8(np.random.randn(4096, 4096).astype(np.float32), 3, 448.0, 2.0 ** -6)
    scale_x = (0.02 * (1.0 + np.random.rand(1))).astype(np.float32)
    scale_w = (0.02 * (1.0 + np.random.rand(1))).astype(np.float32)
    return {"x": x, "w_mat": w_mat, "scale_x": scale_x, "scale_w": scale_w}

def reference(x, w_mat, scale_x, scale_w):
    acc = (x @ w_mat).astype(np.float32)
    y = (acc * (scale_x * scale_w)).astype(np.float32)
    return (y / (1.0 + np.exp(-np.clip(y, -60.0, 60.0)))).astype(np.float32)
```

```cpp
#include <hip/hip_runtime.h>
#include <rccl/rccl.h>
#include <cstdio>
#include <cstdint>

namespace {

constexpr int NR = 8;
constexpr int M_ = 4096, K_ = 4096, N_ = 4096;
constexpr int MS = M_ / NR;
constexpr int KS = K_ / NR;
constexpr int NKB = KS / 128;
constexpr int NJ = N_ / 64;
constexpr int NI = MS / 128;
constexpr int NPUSH = 16;
constexpr int NGEMM = NI * NJ;
constexpr int GRID = NPUSH + NGEMM;
constexpr int PPW = NKB * NJ / NPUSH;
constexpr int NB = 4;

constexpr size_t WPIECE = 8192;
constexpr size_t WSTAGE = (size_t)NR * NKB * NJ * WPIECE;
constexpr size_t XPIECE = 16384;
constexpr size_t XSTAGE = (size_t)NR * NKB * NI * XPIECE;
constexpr size_t OFF_W = 0;
constexpr size_t OFF_X = 2 * WSTAGE;
constexpr size_t OFF_WFLAG = OFF_X + 2 * XSTAGE;
constexpr size_t OFF_XFLAG = OFF_WFLAG + NJ * 128;
constexpr size_t OFF_DONE = OFF_XFLAG + NI * 128;
constexpr size_t OFF_TMO = OFF_DONE + 128;
constexpr size_t WS_NEED = OFF_TMO + 128;

constexpr unsigned long long DEADLINE_TICKS = 50000000ull;

typedef unsigned u32x4 __attribute__((ext_vector_type(4)));
typedef int i32x2 __attribute__((ext_vector_type(2)));
typedef int i32x8 __attribute__((ext_vector_type(8)));
typedef float f32x4 __attribute__((ext_vector_type(4)));
#define LDS_AS __attribute__((address_space(3)))
#define GLB_AS __attribute__((address_space(1)))
constexpr int AUX_SYS = 17;

struct Params {
  const uint8_t* x;
  const uint8_t* w;
  const float* sx;
  const float* sw;
  float* out;
  char* ws[NR];
  int rank;
};

__device__ __forceinline__ __amdgpu_buffer_rsrc_t mk_rsrc(const void* p, unsigned bytes) {
  return __builtin_amdgcn_make_buffer_rsrc(const_cast<void*>(p), 0, bytes, 0x00020000);
}

__global__ void __launch_bounds__(512, 4) gemm_rs_kernel(Params p) {
  __shared__ __attribute__((aligned(16))) unsigned char lds[NB * WPIECE + 256];
  const int tid = threadIdx.x;
  const int lane = tid & 63;
  const int wave = __builtin_amdgcn_readfirstlane(tid >> 6);
  const int rank = p.rank;
  char* const myws = p.ws[rank];
  unsigned* const done_ctr = (unsigned*)(myws + OFF_DONE);
  unsigned* const tmo = (unsigned*)(myws + OFF_TMO);
  volatile LDS_AS unsigned* ctl = (volatile LDS_AS unsigned*)(lds + NB * WPIECE);

  if (blockIdx.x < NPUSH) {
    const int pb = blockIdx.x;
    if (tid < PPW) ctl[tid] = 0;
    const int r = (lane & 15) ^ ((wave & 1) << 3);
    const unsigned src_lane = (unsigned)(wave * 16 + r) * N_ + (lane >> 4) * 16;
    u32x4 v[PPW];
#pragma unroll
    for (int n = 0; n < PPW; ++n) {
      const int piece = n * NPUSH + pb, kb = piece / NJ, j = piece % NJ;
      v[n] = *(const u32x4*)(p.w + (size_t)kb * 128 * N_ + j * 64 + src_lane);
    }
    const unsigned call = __builtin_amdgcn_readfirstlane(
        __hip_atomic_load(done_ctr, __ATOMIC_RELAXED, __HIP_MEMORY_SCOPE_AGENT)) / GRID + 1;
    const size_t stage = OFF_W + (call & 1) * WSTAGE + (size_t)rank * NKB * NJ * WPIECE;
    __amdgpu_buffer_rsrc_t dst[NR - 1];
#pragma unroll
    for (int k = 0; k < NR - 1; ++k) {
      const int d = (rank + 1 + (k + pb) % (NR - 1)) & (NR - 1);
      dst[k] = mk_rsrc(p.ws[d] + stage, NKB * NJ * WPIECE);
    }
    unsigned* fl = (unsigned*)(p.ws[(rank + 1 + (lane % (NR - 1))) & (NR - 1)] + OFF_WFLAG) + rank;
    asm volatile("s_waitcnt vmcnt(0)" ::: "memory");
    __syncthreads();
#pragma unroll
    for (int n = 0; n <= PPW; ++n) {
      if (n < PPW) {
        const int piece = n * NPUSH + pb;
#pragma unroll
        for (int k = 0; k < NR - 1; ++k)
          __builtin_amdgcn_raw_buffer_store_b128(v[n], dst[k], tid * 16, piece * (int)WPIECE, AUX_SYS);
      }
      if (n > 0) {
        if (n < PPW) asm volatile("s_waitcnt vmcnt(7)" ::: "memory");
        else asm volatile("s_waitcnt vmcnt(0)" ::: "memory");
        unsigned t = 0;
        if (lane == 0)
          t = __hip_atomic_fetch_add((LDS_AS unsigned*)&ctl[n - 1], 1u, __ATOMIC_RELAXED, __HIP_MEMORY_SCOPE_WORKGROUP);
        t = __builtin_amdgcn_readfirstlane(t);
        if (t == 7) {
          const int piece = (n - 1) * NPUSH + pb, kb = piece / NJ, j = piece % NJ;
          if (lane < NR - 1)
            __hip_atomic_store(fl + j * 32 + kb * 8, call, __ATOMIC_RELAXED, __HIP_MEMORY_SCOPE_SYSTEM);
        }
      }
    }
    asm volatile("s_waitcnt vmcnt(0)" ::: "memory");
    __syncthreads();
    if (tid == 0) __hip_atomic_fetch_add(done_ctr, 1u, __ATOMIC_RELAXED, __HIP_MEMORY_SCOPE_AGENT);
    return;
  }

  const int t = blockIdx.x - NPUSH;
  const int fr = lane & 15, fg = lane >> 4;
  unsigned call;
  {
    const int dd = (t + (t >> 3)) & 7, xi = (t >> 3) & 3, xkb = (t >> 5) & 3;
    const bool xpush = (t < 128) && (dd != 0);
    const int d = (rank + dd) & (NR - 1);
    u32x4 a0, a1;
    if (xpush) {
      const uint8_t* s = p.x + (size_t)(d * MS + xi * 128 + wave * 16 + fr) * KS + xkb * 128 + fg * 16;
      a0 = *(const u32x4*)s;
      a1 = *(const u32x4*)(s + 64);
    }
    call = __builtin_amdgcn_readfirstlane(
        __hip_atomic_load(done_ctr, __ATOMIC_RELAXED, __HIP_MEMORY_SCOPE_AGENT)) / GRID + 1;
    if (xpush) {
      char* dbase = p.ws[d] + OFF_X + (call & 1) * XSTAGE + (size_t)((rank * NKB + xkb) * NI + xi) * XPIECE;
      auto rs = mk_rsrc(dbase, XPIECE);
      __builtin_amdgcn_raw_buffer_store_b128(a0, rs, wave * 2048 + lane * 16, 0, AUX_SYS);
      __builtin_amdgcn_raw_buffer_store_b128(a1, rs, wave * 2048 + 1024 + lane * 16, 0, AUX_SYS);
      asm volatile("s_waitcnt vmcnt(0)" ::: "memory");
      __syncthreads();
      if (tid == 0)
        __hip_atomic_store((unsigned*)(p.ws[d] + OFF_XFLAG) + xi * 32 + xkb * 8 + rank, call,
                           __ATOMIC_RELAXED, __HIP_MEMORY_SCOPE_SYSTEM);
    }
  }

  const int xcd = t & 7, idx = t >> 3;
  const int ti = idx & 3, tj = (idx >> 2) * 8 + xcd;
  const char* wstage = myws + OFF_W + (call & 1) * WSTAGE;
  const char* xstage = myws + OFF_X + (call & 1) * XSTAGE;
  const unsigned* pollp = lane < 32 ? (const unsigned*)(myws + OFF_WFLAG) + tj * 32 + lane
                                    : (const unsigned*)(myws + OFF_XFLAG) + ti * 32 + (lane - 32);
  const unsigned own = 0x01010101u << rank;
  const unsigned w_own_lane = (unsigned)(wave * 16 + (fr ^ ((wave & 1) << 3))) * N_ + fg * 16 + tj * 64;
  const unsigned x_own_lane = (unsigned)(rank * MS + ti * 128 + wave * 16 + fr) * KS + fg * 16;
  const auto x_own_rs = mk_rsrc(p.x, M_ * KS);
  const auto x_st_rs = mk_rsrc(xstage, XSTAGE);
  const unsigned tr_lane = fg * 1024 + ((fg & 1) << 7) + (lane & 15) * 8;

  f32x4 acc[4] = {};
  unsigned done = 0;
  bool failed = false;
  while (done != 0xffffffffu) {
    if (wave == 0) {
      unsigned avail;
      const unsigned long long t0 = __builtin_amdgcn_s_memrealtime();
      for (;;) {
        const unsigned f = __hip_atomic_load(pollp, __ATOMIC_RELAXED, __HIP_MEMORY_SCOPE_SYSTEM);
        const unsigned long long rdy = __ballot((int)(f - call) >= 0);
        avail = (((unsigned)rdy & (unsigned)(rdy >> 32)) | own) & ~done;
        if (avail) break;
        if (__hip_atomic_load(tmo, __ATOMIC_RELAXED, __HIP_MEMORY_SCOPE_AGENT) == call ||
            __builtin_amdgcn_s_memrealtime() - t0 > DEADLINE_TICKS) {
          __hip_atomic_store(tmo, call, __ATOMIC_RELAXED, __HIP_MEMORY_SCOPE_AGENT);
          break;
        }
        __builtin_amdgcn_s_sleep(4);
      }
      unsigned batch = 0;
#pragma unroll
      for (int b = 0; b < NB; ++b) { const unsigned low = avail & (0u - avail); batch |= low; avail ^= low; }
      if (lane == 0) ctl[0] = batch;
    }
    __syncthreads();
    const unsigned batch = __builtin_amdgcn_readfirstlane(ctl[0]);
    if (batch == 0) { failed = true; break; }
    done |= batch;
    u32x4 xf[NB][2];
    unsigned rem = batch;
#pragma unroll
    for (int b = 0; b < NB; ++b) {
      if (rem) {
        const int bit = __builtin_ctz(rem);
        rem &= rem - 1;
        const int kb = bit >> 3, s = bit & 7;
        LDS_AS void* ldst = (LDS_AS void*)(lds + b * WPIECE + wave * 1024);
        if (s == rank) {
          __builtin_amdgcn_global_load_lds((const GLB_AS void*)(p.w + (size_t)kb * 128 * N_ + w_own_lane), ldst, 16, 0, 0);
          xf[b][0] = __builtin_amdgcn_raw_buffer_load_b128(x_own_rs, x_own_lane, kb * 128, 0);
          xf[b][1] = __builtin_amdgcn_raw_buffer_load_b128(x_own_rs, x_own_lane + 64, kb * 128, 0);
        } else {
          __builtin_amdgcn_global_load_lds(
              (const GLB_AS void*)(wstage + (size_t)((s * NKB + kb) * NJ + tj) * WPIECE + tid * 16), ldst, 16, 0, AUX_SYS);
          const int so = ((s * NKB + kb) * NI + ti) * (int)XPIECE;
          xf[b][0] = __builtin_amdgcn_raw_buffer_load_b128(x_st_rs, wave * 2048 + lane * 16, so, AUX_SYS);
          xf[b][1] = __builtin_amdgcn_raw_buffer_load_b128(x_st_rs, wave * 2048 + 1024 + lane * 16, so, AUX_SYS);
        }
      }
    }
    asm volatile("s_waitcnt vmcnt(0)" ::: "memory");
    __syncthreads();
    const int nb = __builtin_popcount(batch);
#pragma unroll
    for (int b = 0; b < NB; ++b) {
      if (b < nb) {
        i32x8 bx;
        bx[0] = xf[b][0][0]; bx[1] = xf[b][0][1]; bx[2] = xf[b][0][2]; bx[3] = xf[b][0][3];
        bx[4] = xf[b][1][0]; bx[5] = xf[b][1][1]; bx[6] = xf[b][1][2]; bx[7] = xf[b][1][3];
#pragma unroll
        for (int q = 0; q < 4; ++q) {
          i32x8 aw;
#pragma unroll
          for (int c = 0; c < 4; ++c) {
            const unsigned off = b * WPIECE + (c >> 1) * 4096 + q * 256 + (tr_lane ^ ((c & 1) << 7));
            const i32x2 r = __builtin_amdgcn_ds_read_tr8_b64_v2i32((LDS_AS i32x2*)(lds + off));
            aw[2 * c] = r[0]; aw[2 * c + 1] = r[1];
          }
          acc[q] = __builtin_amdgcn_mfma_scale_f32_16x16x128_f8f6f4(aw, bx, acc[q], 0, 0, 0, 0x7f7f7f7f, 0, 0x7f7f7f7f);
        }
      }
    }
  }

  const float sc = p.sx[0] * p.sw[0];
  if (__hip_atomic_load(tmo, __ATOMIC_RELAXED, __HIP_MEMORY_SCOPE_AGENT) == call) failed = true;
  float* orow = p.out + (size_t)(ti * 128 + wave * 16 + fr) * N_ + tj * 64 + fg * 4;
#pragma unroll
  for (int q = 0; q < 4; ++q) {
    f32x4 o;
#pragma unroll
    for (int e = 0; e < 4; ++e) {
      const float y = acc[q][e] * sc;
      const float z = __builtin_amdgcn_fmed3f(y, -60.0f, 60.0f);
      o[e] = failed ? __builtin_nanf("") : y * __builtin_amdgcn_rcpf(1.0f + __expf(-z));
    }
    *(f32x4*)(orow + q * 16) = o;
  }
  if (tid == 0) __hip_atomic_fetch_add(done_ctr, 1u, __ATOMIC_RELAXED, __HIP_MEMORY_SCOPE_AGENT);
}

}

extern "C" void kernel_launch(int rank, int nranks, ncclComm_t comm,
                              hipStream_t stream,
                              void* const* d_in, const int* in_sizes, int n_in,
                              void* d_out, int out_size,
                              const int* in_dtypes, int out_dtype,
                              void* const* d_out_all,
                              void* const* d_ws_all, size_t ws_bytes) {
  if (nranks != NR || (unsigned)rank >= (unsigned)NR || ws_bytes < WS_NEED || n_in < 4) {
    fprintf(stderr, "kernel_launch: unexpected configuration\n");
    return;
  }
  Params p;
  p.x = (const uint8_t*)d_in[0];
  p.w = (const uint8_t*)d_in[1];
  p.sx = (const float*)d_in[2];
  p.sw = (const float*)d_in[3];
  p.out = (float*)d_out;
  for (int i = 0; i < NR; ++i) p.ws[i] = (char*)d_ws_all[i];
  p.rank = rank;
  hipLaunchKernelGGL(gemm_rs_kernel, dim3(GRID), dim3(512), 0, stream, p);
}
```

```cpp
#include <hip/hip_runtime.h>
#include <rccl/rccl.h>
#include <cstdio>
#include <cstdint>

namespace {

constexpr int NR = 8;
constexpr int M_ = 4096, K_ = 4096, N_ = 4096;
constexpr int MS = M_ / NR;
constexpr int KS = K_ / NR;
constexpr int NKB = KS / 128;
constexpr int NJ = N_ / 64;
#define WIN_ 1
#define OUT_AUX 17
#define ZERO_AUX 17
constexpr int NPUSH = 32;
constexpr int NGEMM = 224;
constexpr int GRID = NPUSH + NGEMM;
constexpr int PPW = NKB * NJ / NPUSH;

constexpr unsigned WPIECE = 8192;
constexpr unsigned WSRC = NKB * NJ * WPIECE;
constexpr size_t WSTAGE = (size_t)NR * WSRC;
constexpr unsigned XFRAG = 2048;
constexpr unsigned XSRC = NKB * 32 * XFRAG;
constexpr size_t XSTAGE = (size_t)NR * XSRC;
constexpr size_t OFF_W = 0;
constexpr size_t OFF_X = 3 * WSTAGE;
constexpr size_t OFF_DONE = OFF_X + 3 * XSTAGE;
constexpr size_t OFF_TMO = OFF_DONE + 128;
constexpr size_t WS_NEED = OFF_TMO + 128;

constexpr unsigned SLOT = WPIECE + 2048;
constexpr unsigned LDS_CTL = NR * SLOT;

#define ST(i) do {} while (0)
#define STW0(i) do {} while (0)
constexpr unsigned long long DEADLINE_TICKS = 50000000ull;

typedef unsigned u32x4 __attribute__((ext_vector_type(4)));
typedef unsigned u32x8 __attribute__((ext_vector_type(8)));
typedef int i32x2 __attribute__((ext_vector_type(2)));
typedef int i32x8 __attribute__((ext_vector_type(8)));
typedef float f32x4 __attribute__((ext_vector_type(4)));
#define LDS_AS __attribute__((address_space(3)))
constexpr int AUX_SYS = 17;

struct Params {
  const uint8_t* x;
  const uint8_t* w;
  const float* sx;
  const float* sw;
  float* out;
  char* ws[NR];
  int rank;
};

__device__ __forceinline__ __amdgpu_buffer_rsrc_t mk_rsrc(const void* p, unsigned bytes) {
  return __builtin_amdgcn_make_buffer_rsrc(const_cast<void*>(p), 0, bytes, 0x00020000);
}
template <int N> __device__ __forceinline__ void wait_vm() {
  asm volatile("s_waitcnt vmcnt(%0)" :: "n"(N) : "memory");
}
__device__ __forceinline__ unsigned rfl(unsigned v) { return __builtin_amdgcn_readfirstlane(v); }
__device__ __forceinline__ u32x4 nonzero(u32x4 v) {
#pragma unroll
  for (int e = 0; e < 4; ++e) v[e] = v[e] ? v[e] : 0x80u;
  return v;
}
__device__ __forceinline__ unsigned min4(u32x4 v) {
  return min(min(v[0], v[1]), min(v[2], v[3]));
}
__device__ __forceinline__ unsigned min8(u32x8 v) {
  return min(min(min(v[0], v[1]), min(v[2], v[3])), min(min(v[4], v[5]), min(v[6], v[7])));
}
__device__ __forceinline__ bool expired(unsigned* tmo, unsigned call, unsigned long long t0) {
  if (__builtin_amdgcn_readfirstlane(__hip_atomic_load(tmo, __ATOMIC_RELAXED, __HIP_MEMORY_SCOPE_AGENT)) == call ||
      __builtin_amdgcn_s_memrealtime() - t0 > DEADLINE_TICKS) {
    __hip_atomic_store(tmo, call, __ATOMIC_RELAXED, __HIP_MEMORY_SCOPE_AGENT);
    return true;
  }
  return false;
}

__global__ void __launch_bounds__(1024, 4) gemm_rs_kernel(Params p) {
  __shared__ __attribute__((aligned(16))) unsigned char lds[LDS_CTL + 320];
  const int tid = threadIdx.x;
  const int lane = tid & 63;
  const int wave = __builtin_amdgcn_readfirstlane(tid >> 6);
  const int rank = p.rank;
  char* const myws = p.ws[0];
  unsigned* const done_ctr = (unsigned*)(myws + OFF_DONE);
  unsigned* const tmo = (unsigned*)(myws + OFF_TMO);
  const int fr = lane & 15, fg = lane >> 4;
  const unsigned done_raw = __hip_atomic_load(done_ctr, __ATOMIC_RELAXED, __HIP_MEMORY_SCOPE_AGENT);

  STW0(0);
  if (blockIdx.x < NPUSH) {
    const int pb = blockIdx.x;
    if (wave >= 8) return;
    const unsigned src_lane = (unsigned)(wave * 16 + (fr ^ ((wave & 1) << 3))) * N_ + fg * 16;
    u32x4 v[PPW];
#pragma unroll
    for (int n = 0; n < PPW; ++n) {
      const int piece = n * NPUSH + pb, kb = piece / NJ, j = piece % NJ;
      v[n] = *(const u32x4*)(p.w + (size_t)kb * 128 * N_ + j * 64 + src_lane);
    }
    const unsigned call = __builtin_amdgcn_readfirstlane(done_raw) / GRID + 1;
    const size_t stage = OFF_W + (call % 3) * WSTAGE + (size_t)rank * WSRC;
    __amdgpu_buffer_rsrc_t dst[NR - 1];
#pragma unroll
    for (int k = 0; k < NR - 1; ++k) dst[k] = mk_rsrc(p.ws[1 + (k + pb) % (NR - 1)] + stage, WSRC);
    STW0(1);
#pragma unroll
    for (int n = 0; n < PPW; ++n) {
      const int piece = n * NPUSH + pb;
      const u32x4 d = nonzero(v[n]);
#pragma unroll
      for (int k = 0; k < NR - 1; ++k)
        __builtin_amdgcn_raw_buffer_store_b128(d, dst[k], tid * 16, piece * (int)WPIECE, AUX_SYS);
      if (n + 1 < PPW) {
        wait_vm<7 * (WIN_ - 1)>();
        __builtin_amdgcn_s_barrier();
        STW0(2 + n);
      }
    }
    wait_vm<0>();
    __builtin_amdgcn_s_barrier();
    STW0(2 + PPW - 1);
    if (tid == 0) __hip_atomic_fetch_add(done_ctr, 1u, __ATOMIC_RELAXED, __HIP_MEMORY_SCOPE_AGENT);
    return;
  }

  const int t = blockIdx.x - NPUSH;
  const int xcd = t & 7, idx = t >> 3;
  const int ti = idx & 3, tu = idx >> 2;
  const int tj = 8 + tu * 8 + xcd;
  const bool extra = tu < 4;
  volatile LDS_AS unsigned* ready = (volatile LDS_AS unsigned*)(lds + LDS_CTL);
  volatile LDS_AS unsigned* consumed = (volatile LDS_AS unsigned*)(lds + LDS_CTL + 64);
  volatile LDS_AS unsigned* abortf = (volatile LDS_AS unsigned*)(lds + LDS_CTL + 32);

  if (wave >= 8) {
    const int sv = wave - 8;
    const int xi = t * 8 + sv;
    const int xdd = 1 + xi % 7, xu = xi / 7;
    const int xkb = xu >> 6, xmb = (xu >> 1) & 31, xh = xu & 1;
    const int xd = (rank + xdd) & (NR - 1);
    const u32x4 xa = *(const u32x4*)(p.x + (size_t)(xd * MS + xmb * 16 + fr) * KS + xkb * 128 + xh * 64 + fg * 16);
    if (tid < 512 + 80) ((volatile LDS_AS unsigned*)(lds + LDS_CTL))[tid - 512] = 0;
    const unsigned call = __builtin_amdgcn_readfirstlane(done_raw) / GRID + 1;
    const unsigned buf = call % 3, zbuf = (call + 2) % 3;
    {
      const auto rs = mk_rsrc(p.ws[xdd] + OFF_X + buf * XSTAGE + (size_t)rank * XSRC, XSRC);
      __builtin_amdgcn_raw_buffer_store_b128(nonzero(xa), rs, lane * 16, (xkb * 32 + xmb) * XFRAG + xh * 1024, AUX_SYS);
    }
    asm volatile("s_waitcnt lgkmcnt(0)" ::: "memory");
    __builtin_amdgcn_s_barrier();
    const int s = (rank + sv) & (NR - 1);
    LDS_AS unsigned char* slot = (LDS_AS unsigned char*)lds + sv * SLOT + lane * 16;
    const unsigned ex_lane = fg * 1024 + tu * 256 + fr * 16;
    if (sv == 0) {
      const auto rs = mk_rsrc(p.w, KS * N_);
      const unsigned v_even = (unsigned)fr * N_ + fg * 16 + tj * 64, v_odd = (unsigned)(fr ^ 8) * N_ + fg * 16 + tj * 64;
      const unsigned v_ex = (unsigned)(fg * 16 + (fr ^ ((fg & 1) << 3))) * N_ + xcd * 64 + tu * 16;
      for (int kb = 0; kb < NKB; ++kb) {
        u32x4 d[10];
#pragma unroll
        for (int v = 0; v < 8; ++v)
          d[v] = __builtin_amdgcn_raw_buffer_load_b128(rs, (v & 1) ? v_odd : v_even, (kb * 128 + v * 16) * N_, 0);
        if (extra) {
          d[8] = __builtin_amdgcn_raw_buffer_load_b128(rs, v_ex, (kb * 128) * N_, 0);
          d[9] = __builtin_amdgcn_raw_buffer_load_b128(rs, v_ex, (kb * 128 + 64) * N_, 0);
        }
        while (__builtin_amdgcn_ballot_w64(consumed[lane & 7] < (unsigned)kb)) __builtin_amdgcn_s_sleep(2);
#pragma unroll
        for (int v = 0; v < 8; ++v) *(LDS_AS u32x4*)(slot + v * 1024) = d[v];
        if (extra) { *(LDS_AS u32x4*)(slot + 8192) = d[8]; *(LDS_AS u32x4*)(slot + 9216) = d[9]; }
        asm volatile("s_waitcnt lgkmcnt(0)" ::: "memory");
        if (lane == 0) ready[0] = kb + 1;
      }
      return;
    }
    wait_vm<0>();
    {
      const auto zs = mk_rsrc(myws + OFF_W + zbuf * WSTAGE + (size_t)s * WSRC, WSRC);
      const u32x4 z = {0u, 0u, 0u, 0u};
#pragma unroll
      for (int v = 0; v < 8; ++v)
        __builtin_amdgcn_raw_buffer_store_b128(z, zs, lane * 16 + v * 1024, (ti * NJ + tj) * WPIECE, ZERO_AUX);
      if (extra) {
        __builtin_amdgcn_raw_buffer_store_b128(z, zs, ex_lane, (ti * NJ + xcd) * WPIECE, ZERO_AUX);
        __builtin_amdgcn_raw_buffer_store_b128(z, zs, ex_lane + 4096, (ti * NJ + xcd) * WPIECE, ZERO_AUX);
      }
    }
    const auto rs = mk_rsrc(myws + OFF_W + buf * WSTAGE + (size_t)s * WSRC, WSRC);
    const unsigned long long t0 = __builtin_amdgcn_s_memrealtime();
    for (int kb = 0; kb < NKB; ++kb) {
      const unsigned so = (kb * NJ + tj) * WPIECE, so_ex = (kb * NJ + xcd) * WPIECE;
      bool dead = false;
      for (unsigned it = 0;; ++it) {
        const unsigned f = __builtin_amdgcn_raw_buffer_load_b32(rs, 0, so, AUX_SYS);
        if (__builtin_amdgcn_readfirstlane(f) != 0) break;
        if ((it & 63) == 63 && expired(tmo, call, t0)) { dead = true; break; }
        __builtin_amdgcn_s_sleep(2);
      }
      u32x4 d[10];
      for (unsigned it = 0; !dead; ++it) {
        unsigned m = 0xffffffffu;
#pragma unroll
        for (int v = 0; v < 8; ++v) d[v] = __builtin_amdgcn_raw_buffer_load_b128(rs, lane * 16 + v * 1024, so, AUX_SYS);
        if (extra) {
          d[8] = __builtin_amdgcn_raw_buffer_load_b128(rs, ex_lane, so_ex, AUX_SYS);
          d[9] = __builtin_amdgcn_raw_buffer_load_b128(rs, ex_lane + 4096, so_ex, AUX_SYS);
          m = min(min4(d[8]), min4(d[9]));
        }
#pragma unroll
        for (int v = 0; v < 8; ++v) m = min(m, min4(d[v]));
        if (__builtin_amdgcn_ballot_w64(m == 0) == 0) break;
        if ((it & 63) == 63 && expired(tmo, call, t0)) dead = true;
      }
      if (dead) { if (lane == 0) *abortf = 1; return; }
      while (__builtin_amdgcn_ballot_w64(consumed[sv * 8 + (lane & 7)] < (unsigned)kb) && !rfl(*abortf)) __builtin_amdgcn_s_sleep(2);
#pragma unroll
      for (int v = 0; v < 8; ++v) *(LDS_AS u32x4*)(slot + v * 1024) = d[v];
      if (extra) { *(LDS_AS u32x4*)(slot + 8192) = d[8]; *(LDS_AS u32x4*)(slot + 9216) = d[9]; }
      asm volatile("s_waitcnt lgkmcnt(0)" ::: "memory");
      if (lane == 0) ready[sv] = kb + 1;
      ST(4 + kb * 8 + sv);
    }
    return;
  }

  const unsigned call = __builtin_amdgcn_readfirstlane(done_raw) / GRID + 1;
  const unsigned buf = call % 3, zbuf = (call + 2) % 3;
  const int mb = ti * 8 + wave;
  const auto x_own_rs = mk_rsrc(p.x, M_ * KS);
  const auto x_st_rs = mk_rsrc(myws + OFF_X + buf * XSTAGE, XSTAGE);
  const unsigned x_own_lane = (unsigned)(rank * MS + mb * 16 + fr) * KS + fg * 16;
  const unsigned x_st_lane = mb * XFRAG + lane * 16;
  const float sc = p.sx[0] * p.sw[0];
  auto ldx_own = [&](unsigned kb) {
    const u32x4 lo = __builtin_amdgcn_raw_buffer_load_b128(x_own_rs, x_own_lane, kb * 128, 0);
    const u32x4 hi = __builtin_amdgcn_raw_buffer_load_b128(x_own_rs, x_own_lane + 64, kb * 128, 0);
    return __builtin_shufflevector(lo, hi, 0, 1, 2, 3, 4, 5, 6, 7);
  };
  auto ldx_st = [&](int sv, unsigned kb) {
    const unsigned so = ((rank + sv) & (NR - 1)) * XSRC + kb * 32 * XFRAG;
    const u32x4 lo = __builtin_amdgcn_raw_buffer_load_b128(x_st_rs, x_st_lane, so, AUX_SYS);
    const u32x4 hi = __builtin_amdgcn_raw_buffer_load_b128(x_st_rs, x_st_lane + 1024, so, AUX_SYS);
    return __builtin_shufflevector(lo, hi, 0, 1, 2, 3, 4, 5, 6, 7);
  };
  __builtin_amdgcn_s_barrier();
  {
    const int n = tu * 8 + xcd;
    if (n < 28) {
      const int zs_s = (rank + 1 + n % 7) & (NR - 1), zs_kb = n / 7;
      const auto zs = mk_rsrc(myws + OFF_X + zbuf * XSTAGE, XSTAGE);
      const u32x4 z = {0u, 0u, 0u, 0u};
      __builtin_amdgcn_raw_buffer_store_b128(z, zs, x_st_lane, zs_s * XSRC + zs_kb * 32 * XFRAG, ZERO_AUX);
      __builtin_amdgcn_raw_buffer_store_b128(z, zs, x_st_lane + 1024, zs_s * XSRC + zs_kb * 32 * XFRAG, ZERO_AUX);
    }
  }
  STW0(1);
  const unsigned tr_lane = fg * 1024 + ((fg & 1) << 7) + fr * 8;

  f32x4 acc[5] = {};
  bool failed = false;
  auto step = [&](int sv, u32x8 xv, unsigned kb) {
    const i32x8 bx = __builtin_bit_cast(i32x8, xv);
    LDS_AS unsigned char* sl = (LDS_AS unsigned char*)lds + sv * SLOT;
    unsigned tl = tr_lane;
    asm volatile("" : "+v"(tl));
    if (extra) {
      i32x8 aw;
      const unsigned te = tl - fg * 768 + 8192;
#pragma unroll
      for (int c = 0; c < 4; ++c) {
        const i32x2 r = __builtin_amdgcn_ds_read_tr8_b64_v2i32((LDS_AS i32x2*)(sl + (c >> 1) * 1024 + (te ^ ((c & 1) << 7))));
        aw[2 * c] = r[0]; aw[2 * c + 1] = r[1];
      }
      acc[4] = __builtin_amdgcn_mfma_scale_f32_16x16x128_f8f6f4(aw, bx, acc[4], 0, 0, 0, 0x7f7f7f7f, 0, 0x7f7f7f7f);
    }
#pragma unroll
    for (int q = 0; q < 4; ++q) {
      i32x8 aw;
#pragma unroll
      for (int c = 0; c < 4; ++c) {
        const i32x2 r = __builtin_amdgcn_ds_read_tr8_b64_v2i32((LDS_AS i32x2*)(sl + (c >> 1) * 4096 + q * 256 + (tl ^ ((c & 1) << 7))));
        aw[2 * c] = r[0]; aw[2 * c + 1] = r[1];
      }
      acc[q] = __builtin_amdgcn_mfma_scale_f32_16x16x128_f8f6f4(aw, bx, acc[q], 0, 0, 0, 0x7f7f7f7f, 0, 0x7f7f7f7f);
    }
    asm volatile("s_waitcnt lgkmcnt(0)" ::: "memory");
    consumed[sv * 8 + wave] = kb + 1;
  };
  for (unsigned kb = 0; kb < NKB; ++kb) {
    const u32x8 xv = ldx_own(kb);
    while (rfl(ready[0]) <= kb) __builtin_amdgcn_s_sleep(1);
    step(0, xv, kb);
  }
  const unsigned long long t0 = __builtin_amdgcn_s_memrealtime();
  for (unsigned kb = 0; kb < NKB && !failed; ++kb) {
    u32x8 xf[NR];
    for (unsigned it = 1;; ++it) {
      unsigned m = 0xffffffffu;
#pragma unroll
      for (int sv = 1; sv < NR; ++sv) xf[sv] = ldx_st(sv, kb);
#pragma unroll
      for (int sv = 1; sv < NR; ++sv) m = min(m, min8(xf[sv]));
      if (__builtin_amdgcn_ballot_w64(m == 0) == 0) break;
      if ((it & 15) == 0 && expired(tmo, call, t0)) { failed = true; break; }
      __builtin_amdgcn_s_sleep(32);
    }
    unsigned pend = failed ? 0u : 0xfeu;
    while (pend) {
      unsigned rdy;
      for (;;) {
        const unsigned rv = ((volatile LDS_AS unsigned*)(lds + LDS_CTL))[lane & 15];
        rdy = (unsigned)__builtin_amdgcn_ballot_w64(rv > ((lane & 15) < 8 ? kb : 0u)) & (pend | 0x100u);
        if (rdy) break;
        __builtin_amdgcn_s_sleep(1);
      }
      pend &= ~rdy;
      if (rdy & 0x100u) { failed = true; pend = 0; rdy = 0; }
#pragma unroll
      for (int sv = 1; sv < NR; ++sv)
        if (rdy & (1u << sv)) step(sv, xf[sv], kb);
    }
  }
  if (failed && lane == 0) *abortf = 1;

  STW0(2);
  const auto o_rs = mk_rsrc(p.out, MS * N_ * 4);
  const unsigned o_lane = ((unsigned)(mb * 16 + fr) * N_ + fg * 4) * 4;
#pragma unroll
  for (int q = 0; q < 5; ++q) {
    if (q < 4 || extra) {
      u32x4 o;
#pragma unroll
      for (int e = 0; e < 4; ++e) {
        const float y = acc[q][e] * sc;
        const float z = __builtin_amdgcn_fmed3f(y, -60.0f, 60.0f);
        o[e] = __builtin_bit_cast(unsigned, failed ? __builtin_nanf("") : y * __builtin_amdgcn_rcpf(1.0f + __expf(-z)));
      }
      const unsigned col = q < 4 ? tj * 64 + q * 16 : xcd * 64 + tu * 16;
      __builtin_amdgcn_raw_buffer_store_b128(o, o_rs, o_lane, col * 4, OUT_AUX);
    }
  }
  if (tid == 0) __hip_atomic_fetch_add(done_ctr, 1u, __ATOMIC_RELAXED, __HIP_MEMORY_SCOPE_AGENT);
}

}

extern "C" void kernel_launch(int rank, int nranks, ncclComm_t comm,
                              hipStream_t stream,
                              void* const* d_in, const int* in_sizes, int n_in,
                              void* d_out, int out_size,
                              const int* in_dtypes, int out_dtype,
                              void* const* d_out_all,
                              void* const* d_ws_all, size_t ws_bytes) {
  if (nranks != NR || (unsigned)rank >= (unsigned)NR || ws_bytes < WS_NEED || n_in < 4) {
    fprintf(stderr, "kernel_launch: unexpected configuration\n");
    return;
  }
  Params p;
  p.x = (const uint8_t*)d_in[0];
  p.w = (const uint8_t*)d_in[1];
  p.sx = (const float*)d_in[2];
  p.sw = (const float*)d_in[3];
  p.out = (float*)d_out;
  for (int i = 0; i < NR; ++i) p.ws[i] = (char*)d_ws_all[(rank + i) & (NR - 1)];
  p.rank = rank;
  hipLaunchKernelGGL(gemm_rs_kernel, dim3(GRID), dim3(1024), 0, stream, p);
}
```

```cpp
#include <hip/hip_runtime.h>
#include <rccl/rccl.h>
#include <cstdio>
#include <cstdint>

namespace {

constexpr int NR = 8;
constexpr int M_ = 4096, K_ = 4096, N_ = 4096;
constexpr int MS = M_ / NR;
constexpr int KS = K_ / NR;
constexpr int NKB = KS / 128;
constexpr int NJ = N_ / 64;
#define OUT_AUX 17
#define XPUSH 72
#define PULL0 150
#define OWN0 300
#define ROTW 16
#define EARLYW 0
#define NPULL_ 36
#define WSER 1
#define ZMODE 1
#define OWNSP 0
#define PMAP 0
constexpr int NCOMM = 32;
constexpr int GRID = 256;
constexpr int NPULL = NPULL_;
constexpr int XU = 256 - XPUSH;
constexpr int NUNIT = XU + NKB * NJ * 8;
static_assert(XU % 2 == 0, "units are claimed in pairs");

constexpr unsigned WPIECE = 8192;
constexpr unsigned WSRC = NKB * NJ * WPIECE;
constexpr unsigned WSTAGE = NR * WSRC;
constexpr unsigned XFRAG = 2048;
constexpr unsigned XSRC = NKB * 32 * XFRAG;
constexpr unsigned XSTAGE = NR * XSRC;
constexpr unsigned OFF_W = 0;
constexpr unsigned OFF_X = OFF_W + 2 * WSTAGE;
constexpr unsigned OFF_WE = OFF_X + 3 * XSTAGE;
constexpr unsigned OFF_XE = OFF_WE + 4 * WSRC;
constexpr unsigned OFF_DONE = OFF_XE + 4 * XSTAGE;
constexpr unsigned OFF_TMO = OFF_DONE + 128;
constexpr unsigned OFF_TKT = OFF_TMO + 128;
constexpr unsigned WS_NEED = OFF_TKT + 32 * 128;

constexpr unsigned SLOT = WPIECE + 2048;
constexpr unsigned LDS_OWN = 7 * SLOT;
constexpr unsigned LDS_CTL = LDS_OWN + NKB * SLOT;

#define ST(i) do {} while (0)
#define STW0(i) do {} while (0)
constexpr unsigned long long DEADLINE_TICKS = 50000000ull;

typedef unsigned u32x4 __attribute__((ext_vector_type(4)));
typedef unsigned u32x8 __attribute__((ext_vector_type(8)));
typedef int i32x2 __attribute__((ext_vector_type(2)));
typedef int i32x8 __attribute__((ext_vector_type(8)));
typedef float f32x4 __attribute__((ext_vector_type(4)));
#define LDS_AS __attribute__((address_space(3)))
#define GLB_AS __attribute__((address_space(1)))
typedef unsigned long long u64;
typedef GLB_AS char gchar;
typedef GLB_AS unsigned gu32;
constexpr int AUX_SYS = 17;

struct Params {
  const uint8_t* x;
  const uint8_t* w;
  const float* sx;
  const float* sw;
  float* out;
  char* ws[NR];
  int rank;
};

__device__ __forceinline__ __amdgpu_buffer_rsrc_t mk_rsrc(const gchar* p, unsigned bytes) {
  return __builtin_amdgcn_make_buffer_rsrc((void*)(u64)p, 0, bytes, 0x00020000);
}
template <int AUX> __device__ __forceinline__ void st16(u32x4 v, __amdgpu_buffer_rsrc_t rs, unsigned off) {
  __builtin_amdgcn_raw_buffer_store_b128(v, rs, off, 0, AUX);
}
template <int AUX> __device__ __forceinline__ u32x4 ld16(__amdgpu_buffer_rsrc_t rs, unsigned voff, unsigned soff) {
  return __builtin_amdgcn_raw_buffer_load_b128(rs, voff, soff, AUX);
}
template <int N> __device__ __forceinline__ void wait_vm() {
  asm volatile("s_waitcnt vmcnt(%0)" :: "n"(N) : "memory");
}
__device__ __forceinline__ gchar* sel8(int k, u64 a0, u64 a1, u64 a2, u64 a3, u64 a4, u64 a5, u64 a6, u64 a7) {
  const u64 b0 = (k & 1) ? a1 : a0, b1 = (k & 1) ? a3 : a2, b2 = (k & 1) ? a5 : a4, b3 = (k & 1) ? a7 : a6;
  const u64 c0 = (k & 2) ? b1 : b0, c1 = (k & 2) ? b3 : b2;
  return (gchar*)((k & 4) ? c1 : c0);
}
__device__ __forceinline__ unsigned rfl(unsigned v) { return __builtin_amdgcn_readfirstlane(v); }
__device__ __forceinline__ u32x4 nonzero(u32x4 v) {
#pragma unroll
  for (int e = 0; e < 4; ++e) v[e] = v[e] ? v[e] : 0x80u;
  return v;
}
__device__ __forceinline__ unsigned min4(u32x4 v) { return min(min(v[0], v[1]), min(v[2], v[3])); }
__device__ __forceinline__ unsigned min8(u32x8 v) {
  return min(min(min(v[0], v[1]), min(v[2], v[3])), min(min(v[4], v[5]), min(v[6], v[7])));
}
__device__ __forceinline__ int piece_at(int pos, int k) {
  constexpr int B = NKB * NJ - ROTW;
  return ROTW && pos >= B ? B + ((pos - B + k * (ROTW / 8)) & (ROTW - 1)) : pos;
}
__device__ __forceinline__ int pos_of(int piece, int k) {
  constexpr int B = NKB * NJ - ROTW;
  return ROTW && piece >= B ? B + ((piece - B - k * (ROTW / 8)) & (ROTW - 1)) : piece;
}
__device__ __forceinline__ void sleep_until(unsigned long long t0, unsigned ticks) {
  while (__builtin_amdgcn_s_memrealtime() - t0 < ticks) __builtin_amdgcn_s_sleep(2);
}
__device__ __forceinline__ bool expired(gu32* tmo, unsigned call, unsigned long long t0) {
  if (rfl(__hip_atomic_load(tmo, __ATOMIC_RELAXED, __HIP_MEMORY_SCOPE_AGENT)) == call ||
      __builtin_amdgcn_s_memrealtime() - t0 > DEADLINE_TICKS) {
    __hip_atomic_store(tmo, call, __ATOMIC_RELAXED, __HIP_MEMORY_SCOPE_AGENT);
    return true;
  }
  return false;
}

__global__ void __launch_bounds__(1024, 4) gemm_rs_kernel(Params pin) {
  u64 ax = (u64)pin.x, aw_ = (u64)pin.w, asx = (u64)pin.sx, asw = (u64)pin.sw, aout = (u64)pin.out;
  u64 w0 = (u64)pin.ws[0], w1 = (u64)pin.ws[1], w2 = (u64)pin.ws[2], w3 = (u64)pin.ws[3];
  u64 w4 = (u64)pin.ws[4], w5 = (u64)pin.ws[5], w6 = (u64)pin.ws[6], w7 = (u64)pin.ws[7];
  int rank = pin.rank;
  asm volatile("" : "+s"(ax), "+s"(aw_), "+s"(asx), "+s"(asw), "+s"(aout), "+s"(rank),
                    "+s"(w0), "+s"(w1), "+s"(w2), "+s"(w3), "+s"(w4), "+s"(w5), "+s"(w6), "+s"(w7));
#define ws_of(k) sel8((k), w0, w1, w2, w3, w4, w5, w6, w7)
  const gchar* const px = (const gchar*)ax;
  const gchar* const pw = (const gchar*)aw_;
  __shared__ __attribute__((aligned(16))) unsigned char lds[LDS_CTL + 384];
  const int tid = threadIdx.x;
  const int lane = tid & 63;
  const int wave = __builtin_amdgcn_readfirstlane(tid >> 6);
  gchar* const myws = (gchar*)w0;
  gu32* const done_ctr = (gu32*)(myws + OFF_DONE);
  gu32* const tmo = (gu32*)(myws + OFF_TMO);
  const int fr = lane & 15, fg = lane >> 4;
  const unsigned done_raw = __hip_atomic_load(done_ctr, __ATOMIC_RELAXED, __HIP_MEMORY_SCOPE_AGENT);
  const unsigned long long t0 = __builtin_amdgcn_s_memrealtime();
  const auto my_rs = mk_rsrc(myws, OFF_DONE);

  STW0(0);
  const int t = blockIdx.x;
  const bool comm = t < NCOMM;
  const int xcd = t & 7, idx = (t - NCOMM) >> 3;
  const int ti = idx & 3, tu = idx >> 2;
  const int tj = 8 + tu * 8 + xcd;
  const bool extra = tu < 4;
  volatile LDS_AS unsigned* ctl = (volatile LDS_AS unsigned*)(lds + LDS_CTL);
  volatile LDS_AS unsigned* ready = ctl;
  volatile LDS_AS unsigned* abortf = ctl + 8;
  volatile LDS_AS unsigned* ownrdy = ctl + 12;
  volatile LDS_AS unsigned* consumed = ctl + 16;

  if (comm && wave >= 8) {
    const unsigned call = rfl(done_raw) / GRID + 1;
    const int g = t * 8 + (wave - 8);
    const int k = 1 + g / NPULL, q = g % NPULL;
    const bool idle = g >= NPULL * (NR - 1);
    if (idle) {
      if (wave == 15 && lane < NR)
        __hip_atomic_store((gu32*)(myws + OFF_TKT + (((call + 2) & 3) * 8 + lane) * 128), 0u, __ATOMIC_RELAXED, __HIP_MEMORY_SCOPE_AGENT);
      return;
    }
    const int s = (rank + k) & (NR - 1);
    const auto src = mk_rsrc(ws_of(k), OFF_DONE);
    const unsigned sx0 = OFF_XE + (call & 3) * XSTAGE + rank * XSRC, sw0 = OFF_WE + (call & 3) * WSRC;
    const unsigned dx0 = OFF_X + (call % 3) * XSTAGE + s * XSRC, dw0 = OFF_W + (call & 1) * WSTAGE + s * WSRC;
    const unsigned lo = lane * 16;
    auto uoff = [&](int u) -> unsigned {
      return u < XU ? (u < NCOMM ? u : u + XPUSH) * 1024 : piece_at((u - XU) >> 3, k) * WPIECE + ((u - XU) & 7) * 1024;
    };
    auto soff = [&](int u) { return (u < XU ? sx0 : sw0) + uoff(u); };
    auto doff = [&](int u) { return (u < XU ? dx0 : dw0) + uoff(u); };
    sleep_until(t0, PULL0);
    for (unsigned it = 1;; ++it) {
      const unsigned f = __builtin_amdgcn_raw_buffer_load_b32(src, 0, soff(2 * q), AUX_SYS);
      if (rfl(f) != 0) break;
      if ((it & 63) == 0 && expired(tmo, call, t0)) return;
      __builtin_amdgcn_s_sleep(4);
    }
    ST(40 + wave - 8);
    const u64 sp = (u64)ws_of(k);
    const u32x4 srd_s = {rfl((unsigned)sp), rfl((unsigned)(sp >> 32) & 0xffffu), OFF_DONE, 0x00020000u};
    const u32x4 srd_d = {rfl((unsigned)w0), rfl((unsigned)(w0 >> 32) & 0xffffu), OFF_DONE, 0x00020000u};
    constexpr unsigned OOB = 0x80000000u;
#define PLD(v, u) asm volatile("buffer_load_dwordx4 %0, %1, %2, 0 offen sc0 sc1" \
        : "=v"(v) : "v"(lo + ((u) < NUNIT ? soff(u) : OOB)), "s"(srd_s) : "memory")
#define PST(v, o) asm volatile("buffer_store_dwordx4 %0, %1, %2, 0 offen sc0 sc1\n\ts_nop 1" \
        :: "v"(v), "v"(lo + (o)), "s"(srd_d) : "memory")
    auto settle = [&](u32x4& v, int u) {
      for (unsigned it = 1; __builtin_expect(__builtin_amdgcn_ballot_w64(min4(v) == 0) != 0, 0); ++it) {
        if ((it & 63) == 0 && expired(tmo, call, t0)) return false;
        __builtin_amdgcn_s_sleep(8);
        asm volatile("buffer_load_dwordx4 %0, %1, %2, 0 offen sc0 sc1\n\ts_waitcnt vmcnt(0)"
                     : "=&v"(v) : "v"(lo + soff(u)), "s"(srd_s) : "memory");
      }
      return true;
    };
    const unsigned tkt = OFF_TKT + ((call & 3) * 8 + k) * 128;
    const unsigned one = lane == 0 ? 1u : 0u;
#define TKT(r) asm volatile("s_mov_b64 exec, 1\n\tglobal_atomic_add %0, %1, %2, %3 sc0\n\ts_mov_b64 exec, -1" \
        : "=&v"(r) : "v"(tkt), "v"(one), "s"(w0) : "memory")
    u32x4 v[2][2];
    unsigned tk[2];
    asm volatile("s_nop 4" :: "s"(srd_s), "s"(srd_d));
    int cur = 2 * q, nxt = 2 * (q + NPULL);
    v[1][0] = v[1][1] = u32x4{0u, 0u, 0u, 0u};
    PLD(v[0][0], cur);
    PST(v[1][0], OOB);
    PLD(v[0][1], cur + 1);
    PST(v[1][1], OOB);
#define HALF(h, first) \
      asm volatile("s_waitcnt vmcnt(3)" : "+v"(v[h][0]), "+v"(tk[h ^ 1]) :: "memory"); \
      if (!(first)) nxt = 2 * (int)(rfl(tk[h ^ 1]) + 2 * NPULL); \
      if (!settle(v[h][0], cur)) return; \
      TKT(tk[h]); \
      PLD(v[h ^ 1][0], nxt); \
      PST(v[h][0], doff(cur)); \
      asm volatile("s_waitcnt vmcnt(4)" : "+v"(v[h][1]) :: "memory"); \
      if (!settle(v[h][1], cur + 1)) return; \
      PLD(v[h ^ 1][1], nxt + 1); \
      PST(v[h][1], doff(cur + 1)); \
      PSTAMP \
      cur = nxt;
#define PSTAMP
    tk[1] = 0;
    HALF(0, true)
    while (cur < NUNIT) {
      HALF(1, false)
      if (cur >= NUNIT) break;
      HALF(0, false)
    }
    wait_vm<0>();
    ST(48 + wave - 8);
    return;
  }

  if (wave == 8) {
    if (lane < 16) ctl[lane] = 0;
    consumed[lane] = 0;
    asm volatile("s_waitcnt lgkmcnt(0)" ::: "memory");
    __builtin_amdgcn_s_barrier();
    sleep_until(t0, OWN0 + tu * OWNSP);
    const gchar* w_even = pw + (size_t)fr * N_ + fg * 16 + tj * 64;
    const gchar* w_odd = pw + (size_t)(fr ^ 8) * N_ + fg * 16 + tj * 64;
    const gchar* w_ex = pw + (size_t)(fg * 16 + (fr ^ ((fg & 1) << 3))) * N_ + xcd * 64 + tu * 16;
    for (int kb = 0; kb < NKB; ++kb) {
#pragma unroll
      for (int v = 0; v < 8; ++v)
        __builtin_amdgcn_global_load_lds((const GLB_AS void*)(((v & 1) ? w_odd : w_even) + (size_t)(kb * 128 + v * 16) * N_),
                                         (LDS_AS void*)(lds + LDS_OWN + kb * SLOT + v * 1024), 16, 0, 0);
      if (extra) {
        __builtin_amdgcn_global_load_lds((const GLB_AS void*)(w_ex + (size_t)(kb * 128) * N_),
                                         (LDS_AS void*)(lds + LDS_OWN + kb * SLOT + 8192), 16, 0, 0);
        __builtin_amdgcn_global_load_lds((const GLB_AS void*)(w_ex + (size_t)(kb * 128 + 64) * N_),
                                         (LDS_AS void*)(lds + LDS_OWN + kb * SLOT + 9216), 16, 0, 0);
      }
      wait_vm<0>();
      if (lane == 0) ownrdy[kb] = 1;
    }
    return;
  }

  if (wave > 8) {
    const int sv = wave - 8;
    const int s = (rank + sv) & (NR - 1);
    const unsigned call = rfl(done_raw) / GRID + 1;
    const unsigned ex_lane = fg * 1024 + tu * 256 + fr * 16;
    __builtin_amdgcn_s_barrier();
    sleep_until(t0, OWN0);
    const unsigned zo = OFF_W + ((call + 1) & 1) * WSTAGE + s * WSRC + ti * NJ * WPIECE;
    auto rezero = [&](int part) {
      const u32x4 z = {0u, 0u, 0u, 0u};
      st16<AUX_SYS>(z, my_rs, zo + tj * WPIECE + lane * 16 + part * 3072);
      st16<AUX_SYS>(z, my_rs, zo + tj * WPIECE + lane * 16 + part * 3072 + 1024);
      if (part < 2) {
        st16<AUX_SYS>(z, my_rs, zo + tj * WPIECE + lane * 16 + part * 3072 + 2048);
        if (extra) st16<AUX_SYS>(z, my_rs, zo + xcd * WPIECE + ex_lane + part * 4096);
      }
    };
    if (!ZMODE) { rezero(0); rezero(1); rezero(2); }
    LDS_AS unsigned char* slot = (LDS_AS unsigned char*)lds + (sv - 1) * SLOT + lane * 16;
    for (int kb = 0; kb < NKB; ++kb) {
      const unsigned sb = OFF_W + (call & 1) * WSTAGE + s * WSRC + kb * NJ * WPIECE;
      const unsigned so = sb + tj * WPIECE, so_ex = sb + xcd * WPIECE;
      const unsigned so_ind = kb == NKB - 1 ? sb - kb * NJ * WPIECE + piece_at(pos_of(kb * NJ + tj, sv) - EARLYW, sv) * WPIECE : so;
      bool dead = false;
      for (unsigned it = 1;; ++it) {
        const unsigned f = __builtin_amdgcn_raw_buffer_load_b32(my_rs, 0, so_ind, AUX_SYS);
        if (rfl(f) != 0) break;
        if ((it & 63) == 0 && expired(tmo, call, t0)) { dead = true; break; }
        __builtin_amdgcn_s_sleep(2);
      }
      u32x4 d[10];
      for (unsigned it = 1; !dead; ++it) {
        unsigned m = 0xffffffffu;
#pragma unroll
        for (int v = 0; v < 8; ++v) d[v] = ld16<AUX_SYS>(my_rs, lane * 16 + v * 1024, so);
        if (extra) {
          d[8] = ld16<AUX_SYS>(my_rs, ex_lane, so_ex);
          d[9] = ld16<AUX_SYS>(my_rs, ex_lane + 4096, so_ex);
          m = min(min4(d[8]), min4(d[9]));
        }
#pragma unroll
        for (int v = 0; v < 8; ++v) m = min(m, min4(d[v]));
        if (__builtin_amdgcn_ballot_w64(m == 0) == 0) break;
        if ((it & 63) == 0 && expired(tmo, call, t0)) dead = true;
      }
      if (dead) { if (lane == 0) *abortf = 1; return; }
      while (__builtin_amdgcn_ballot_w64(consumed[sv * 8 + (lane & 7)] < (unsigned)kb) && !rfl(*abortf))
        __builtin_amdgcn_s_sleep(2);
#pragma unroll
      for (int v = 0; v < 8; ++v) *(LDS_AS u32x4*)(slot + v * 1024) = d[v];
      if (extra) { *(LDS_AS u32x4*)(slot + 8192) = d[8]; *(LDS_AS u32x4*)(slot + 9216) = d[9]; }
      asm volatile("s_waitcnt lgkmcnt(0)" ::: "memory");
      if (lane == 0) ready[sv] = kb + 1;
      ST(4 + kb * 8 + sv);
      if (ZMODE && kb < NKB - 1) rezero(kb);
    }
    return;
  }

  const int xs = (rank + wave) & (NR - 1);
  const int er = lane >> 3, ec = lane & 7;
  const u32x4 xa = *(const GLB_AS u32x4*)(px + (size_t)(xs * MS + (t & 63) * 8 + er) * KS + (t >> 6) * 128 + ec * 16);
  const unsigned x_img = (t >> 1) * XFRAG + (ec >> 2) * 1024 + ((ec & 3) * 16 + (t & 1) * 8 + er) * 16;
  const int ek16 = (t & 1) * 4 + (wave >> 1);
  const gchar* wsrc = pw + (size_t)((t >> 6) * 128 + ek16 * 16 + (wave & 1) * 8 + er) * N_ + (t & 62) * 64 + ec * 16;
  const unsigned w_img = ((t & ~1) + (ec >> 2)) * WPIECE + ek16 * 1024 + (ec & 3) * 256 + ((((wave ^ ek16) & 1) * 8 + er) * 16);
  const unsigned call = rfl(done_raw) / GRID + 1;
  if (wave > 0) {
    if (t >= NCOMM && t < NCOMM + XPUSH) st16<AUX_SYS>(nonzero(xa), mk_rsrc(ws_of(wave), OFF_DONE), OFF_X + (call % 3) * XSTAGE + rank * XSRC + x_img);
    else st16<AUX_SYS>(nonzero(xa), my_rs, OFF_XE + (call & 3) * XSTAGE + xs * XSRC + x_img);
  }
  asm volatile("" ::: "memory");
  const u32x4 wa = *(const GLB_AS u32x4*)wsrc;
  st16<AUX_SYS>(nonzero(wa), my_rs, OFF_WE + (call & 3) * WSRC + w_img);
  if (!comm) __builtin_amdgcn_s_barrier();
  STW0(1);
  auto rezero = [&]() {
    const u32x4 z = {0u, 0u, 0u, 0u};
    st16<AUX_SYS>(z, my_rs, OFF_X + ((call + 2) % 3) * XSTAGE + xs * XSRC + t * 1024 + lane * 16);
    st16<AUX_SYS>(z, my_rs, OFF_XE + ((call + 2) & 3) * XSTAGE + xs * XSRC + t * 1024 + lane * 16);
    st16<AUX_SYS>(z, my_rs, OFF_WE + ((call + 2) & 3) * WSRC + t * WPIECE + tid * 16);
  };
  if (comm) rezero();
  else {
    sleep_until(t0, OWN0 + tu * OWNSP);
    if (!ZMODE) rezero();
  }
  if (comm) {
    if (tid == 0) __hip_atomic_fetch_add(done_ctr, 1u, __ATOMIC_RELAXED, __HIP_MEMORY_SCOPE_AGENT);
    return;
  }
  const int mb = ti * 8 + wave;
  const auto x_own_rs = mk_rsrc(px, M_ * KS);
  const unsigned x_own_lane = (unsigned)(rank * MS + mb * 16 + fr) * KS + fg * 16;
  const unsigned x_st_lane = OFF_X + (call % 3) * XSTAGE + mb * XFRAG + lane * 16;
  const float sc = *(const GLB_AS float*)asx * *(const GLB_AS float*)asw;
  auto ldx_own = [&](unsigned kb) {
    const u32x4 lo = ld16<0>(x_own_rs, x_own_lane, kb * 128);
    const u32x4 hi = ld16<0>(x_own_rs, x_own_lane + 64, kb * 128);
    return __builtin_shufflevector(lo, hi, 0, 1, 2, 3, 4, 5, 6, 7);
  };
  auto ldx_st = [&](int sv, unsigned kb) {
    const unsigned so = ((rank + sv) & (NR - 1)) * XSRC + kb * 32 * XFRAG;
    const u32x4 lo = ld16<AUX_SYS>(my_rs, x_st_lane, so);
    const u32x4 hi = ld16<AUX_SYS>(my_rs, x_st_lane + 1024, so);
    return __builtin_shufflevector(lo, hi, 0, 1, 2, 3, 4, 5, 6, 7);
  };
  const unsigned tr_lane = fg * 1024 + ((fg & 1) << 7) + fr * 8;

  f32x4 acc[5] = {};
  bool failed = false;
  auto step = [&](unsigned base, u32x8 xv) {
    const i32x8 bx = __builtin_bit_cast(i32x8, xv);
    unsigned tl = tr_lane;
    asm volatile("" : "+v"(tl));
    LDS_AS unsigned char* sl = (LDS_AS unsigned char*)lds + base;
    if (extra) {
      i32x8 aw;
      const unsigned te = tl - fg * 768 + 8192;
#pragma unroll
      for (int c = 0; c < 4; ++c) {
        const i32x2 r = __builtin_amdgcn_ds_read_tr8_b64_v2i32((LDS_AS i32x2*)(sl + (c >> 1) * 1024 + (te ^ ((c & 1) << 7))));
        aw[2 * c] = r[0]; aw[2 * c + 1] = r[1];
      }
      acc[4] = __builtin_amdgcn_mfma_scale_f32_16x16x128_f8f6f4(aw, bx, acc[4], 0, 0, 0, 0x7f7f7f7f, 0, 0x7f7f7f7f);
    }
#pragma unroll
    for (int q = 0; q < 4; ++q) {
      i32x8 aw;
#pragma unroll
      for (int c = 0; c < 4; ++c) {
        const i32x2 r = __builtin_amdgcn_ds_read_tr8_b64_v2i32((LDS_AS i32x2*)(sl + (c >> 1) * 4096 + q * 256 + (tl ^ ((c & 1) << 7))));
        aw[2 * c] = r[0]; aw[2 * c + 1] = r[1];
      }
      acc[q] = __builtin_amdgcn_mfma_scale_f32_16x16x128_f8f6f4(aw, bx, acc[q], 0, 0, 0, 0x7f7f7f7f, 0, 0x7f7f7f7f);
    }
  };
#pragma unroll
  for (unsigned kb = 0; kb < NKB; ++kb) {
    const u32x8 xv = ldx_own(kb);
    while (!rfl(ownrdy[kb])) __builtin_amdgcn_s_sleep(1);
    step(LDS_OWN + kb * SLOT, xv);
  }
  for (unsigned kb = 0; kb < NKB && !failed; ++kb) {
    u32x8 xf[NR];
    if (kb == 0) {
      while (!((unsigned)__builtin_amdgcn_ballot_w64(ctl[lane & 15] != 0u) & 0x1feu)) __builtin_amdgcn_s_sleep(4);
      if (ZMODE) rezero();
    }
    for (unsigned it = 1;; ++it) {
      unsigned m = 0xffffffffu;
#pragma unroll
      for (int sv = 1; sv < NR; ++sv) xf[sv] = ldx_st(sv, kb);
#pragma unroll
      for (int sv = 1; sv < NR; ++sv) m = min(m, min8(xf[sv]));
      if (__builtin_amdgcn_ballot_w64(m == 0) == 0) break;
      if ((it & 15) == 0 && expired(tmo, call, t0)) { failed = true; break; }
      __builtin_amdgcn_s_sleep(32);
    }
    unsigned pend = failed ? 0u : 0xfeu;
    while (pend) {
      unsigned rdy;
      for (;;) {
        const unsigned rv = ctl[lane & 15];
        rdy = (unsigned)__builtin_amdgcn_ballot_w64(rv > ((lane & 15) < 8 ? kb : 0u)) & (pend | 0x100u);
        if (rdy) break;
        __builtin_amdgcn_s_sleep(1);
      }
      pend &= ~rdy;
      if (rdy & 0x100u) { failed = true; pend = 0; rdy = 0; }
#pragma unroll
      for (int sv = 1; sv < NR; ++sv)
        if (rdy & (1u << sv)) {
          step((sv - 1) * SLOT, xf[sv]);
          asm volatile("s_waitcnt lgkmcnt(0)" ::: "memory");
          consumed[sv * 8 + wave] = kb + 1;
        }
    }
  }
  if (failed && lane == 0) *abortf = 1;

  STW0(2);
  const float sce = failed ? __builtin_nanf("") : sc;
  const auto o_rs = mk_rsrc((gchar*)aout, MS * N_ * 4);
  const unsigned o_lane = ((unsigned)(mb * 16 + fr) * N_ + fg * 4) * 4;
#pragma unroll
  for (int q = 0; q < 5; ++q) {
    if (q < 4 || extra) {
      u32x4 o;
#pragma unroll
      for (int e = 0; e < 4; ++e) {
        const float y = acc[q][e] * sce;
        const float z = __builtin_amdgcn_fmed3f(y, -60.0f, 60.0f);
        o[e] = __builtin_bit_cast(unsigned, y * __builtin_amdgcn_rcpf(1.0f + __expf(-z)));
      }
      st16<OUT_AUX>(o, o_rs, o_lane + (q < 4 ? tj * 64 + q * 16 : xcd * 64 + tu * 16) * 4);
    }
  }
  if (tid == 0) __hip_atomic_fetch_add(done_ctr, 1u, __ATOMIC_RELAXED, __HIP_MEMORY_SCOPE_AGENT);
}

}

extern "C" void kernel_launch(int rank, int nranks, ncclComm_t comm,
                              hipStream_t stream,
                              void* const* d_in, const int* in_sizes, int n_in,
                              void* d_out, int out_size,
                              const int* in_dtypes, int out_dtype,
                              void* const* d_out_all,
                              void* const* d_ws_all, size_t ws_bytes) {
  if (nranks != NR || (unsigned)rank >= (unsigned)NR || ws_bytes < WS_NEED || n_in < 4) {
    fprintf(stderr, "kernel_launch: unexpected configuration\n");
    return;
  }
  Params p;
  p.x = (const uint8_t*)d_in[0];
  p.w = (const uint8_t*)d_in[1];
  p.sx = (const float*)d_in[2];
  p.sw = (const float*)d_in[3];
  p.out = (float*)d_out;
  for (int i = 0; i < NR; ++i) p.ws[i] = (char*)d_ws_all[(rank + i) & (NR - 1)];
  p.rank = rank;
  hipLaunchKernelGGL(gemm_rs_kernel, dim3(GRID), dim3(1024), 0, stream, p);
}
```

```cpp
#include <hip/hip_runtime.h>
#include <rccl/rccl.h>
#include <cstdio>
#include <cstdint>

namespace {

constexpr int NR = 8;
constexpr int M_ = 4096, K_ = 4096, N_ = 4096;
constexpr int MS = M_ / NR;
constexpr int KS = K_ / NR;
constexpr int NKB = KS / 128;
constexpr int NJ = N_ / 64;
#define XPUSH 108
#define PULL0 100
#define OWN0 300
#define NPULL_ 36
#define DIRECT 1
#define LEAD 14
#define REFP 32
#define SLOT_T 134
#define NOPROBE 1
#define ARRS 3
#define ARRN ARRS
#define WSPLIT 1
#define WDELAY 0
#define ROTALL 1
#define TLBW 1
#define WARM 1
#define DRY 1
constexpr int ROTW = 16;
constexpr int GRID = 256;
constexpr int NCOMM = 32;
constexpr int NPULL = NPULL_;
constexpr int PWAVES = (NPULL * (NR - 1) + NCOMM - 1) / NCOMM;
constexpr int XU = 256 - XPUSH;
constexpr int NUNIT = XU + (NKB * NJ - (DIRECT ? ROTW : 0)) * 8;
static_assert(DIRECT && XU % 2 == 0 && (NOPROBE || XU >= 2 * NPULL) && XPUSH <= GRID - NCOMM && PWAVES <= 8 && LEAD <= REFP, "");

constexpr unsigned WPIECE = 8192;
constexpr unsigned WSRC = NKB * NJ * WPIECE;
constexpr unsigned WSTAGE = NR * WSRC;
constexpr unsigned XFRAG = 2048;
constexpr unsigned XSRC = NKB * 32 * XFRAG;
constexpr unsigned XSTAGE = NR * XSRC;
constexpr unsigned OFF_W = 0;
constexpr unsigned OFF_X = OFF_W + 2 * WSTAGE;
constexpr unsigned OFF_WE = OFF_X + 3 * XSTAGE;
constexpr unsigned OFF_XE = OFF_WE + 4 * WSRC;
constexpr unsigned OFF_CNT = OFF_XE + 4 * XSTAGE;
constexpr unsigned OFF_TMO = OFF_CNT + GRID * 128;
constexpr unsigned OFF_TKT = OFF_TMO + 128;
constexpr unsigned WS_NEED = OFF_TKT + 32 * 128;

constexpr unsigned SLOT = WPIECE + 2048;
constexpr unsigned LDS_OWN = 7 * SLOT;
constexpr unsigned LDS_CTL = LDS_OWN + NKB * SLOT;

#define ST(i) do {} while (0)
#define STW0(i) do {} while (0)
constexpr unsigned long long DEADLINE_TICKS = 50000000ull;

typedef unsigned u32x4 __attribute__((ext_vector_type(4)));
typedef unsigned u32x8 __attribute__((ext_vector_type(8)));
typedef int i32x2 __attribute__((ext_vector_type(2)));
typedef int i32x8 __attribute__((ext_vector_type(8)));
typedef float f32x4 __attribute__((ext_vector_type(4)));
#define LDS_AS __attribute__((address_space(3)))
#define GLB_AS __attribute__((address_space(1)))
typedef unsigned long long u64;
typedef GLB_AS char gchar;
typedef GLB_AS unsigned gu32;
constexpr int AUX_SYS = 17;

struct Params {
  const uint8_t* x;
  const uint8_t* w;
  const float* sx;
  const float* sw;
  float* out;
  char* ws[NR];
  int rank;
};

__device__ __forceinline__ __amdgpu_buffer_rsrc_t mk_rsrc(const gchar* p, unsigned bytes) {
  return __builtin_amdgcn_make_buffer_rsrc((void*)(u64)p, 0, bytes, 0x00020000);
}
__device__ __forceinline__ void st16(u32x4 v, __amdgpu_buffer_rsrc_t rs, unsigned off) {
  __builtin_amdgcn_raw_buffer_store_b128(v, rs, off, 0, AUX_SYS);
}
template <int AUX> __device__ __forceinline__ u32x4 ld16(__amdgpu_buffer_rsrc_t rs, unsigned voff, unsigned soff) {
  return __builtin_amdgcn_raw_buffer_load_b128(rs, voff, soff, AUX);
}
__device__ __forceinline__ unsigned ld4(__amdgpu_buffer_rsrc_t rs, unsigned voff, unsigned soff) {
  return __builtin_amdgcn_raw_buffer_load_b32(rs, voff, soff, AUX_SYS);
}
template <int N> __device__ __forceinline__ void wait_vm() {
  asm volatile("s_waitcnt vmcnt(%0)" :: "n"(N) : "memory");
}
__device__ __forceinline__ gchar* sel8(int k, u64 a0, u64 a1, u64 a2, u64 a3, u64 a4, u64 a5, u64 a6, u64 a7) {
  const u64 b0 = (k & 1) ? a1 : a0, b1 = (k & 1) ? a3 : a2, b2 = (k & 1) ? a5 : a4, b3 = (k & 1) ? a7 : a6;
  const u64 c0 = (k & 2) ? b1 : b0, c1 = (k & 2) ? b3 : b2;
  return (gchar*)((k & 4) ? c1 : c0);
}
__device__ __forceinline__ unsigned rfl(unsigned v) { return __builtin_amdgcn_readfirstlane(v); }
__device__ __forceinline__ u32x4 nonzero(u32x4 v) {
#pragma unroll
  for (int e = 0; e < 4; ++e) v[e] = v[e] ? v[e] : 0x80u;
  return v;
}
__device__ __forceinline__ unsigned min4(u32x4 v) { return min(min(v[0], v[1]), min(v[2], v[3])); }
__device__ __forceinline__ unsigned min8(u32x8 v) {
  return min(min(min(v[0], v[1]), min(v[2], v[3])), min(min(v[4], v[5]), min(v[6], v[7])));
}
__device__ __forceinline__ int piece_at(int pos, int k) {
  constexpr int B = ROTALL ? 0 : NKB * NJ - ROTW;
  return pos >= B ? (pos & ~(ROTW - 1)) | ((pos + k * (ROTW / 8)) & (ROTW - 1)) : pos;
}
__device__ __forceinline__ int pos_of(int piece, int k) {
  constexpr int B = ROTALL ? 0 : NKB * NJ - ROTW;
  return piece >= B ? (piece & ~(ROTW - 1)) | ((piece - k * (ROTW / 8)) & (ROTW - 1)) : piece;
}
__device__ __forceinline__ void sleep_until(unsigned long long t0, unsigned ticks) {
  while (__builtin_amdgcn_s_memrealtime() - t0 < ticks) __builtin_amdgcn_s_sleep(2);
}
__device__ __forceinline__ bool expired(gu32* tmo, unsigned call, unsigned long long t0) {
  if (rfl(__hip_atomic_load(tmo, __ATOMIC_RELAXED, __HIP_MEMORY_SCOPE_AGENT)) == call ||
      __builtin_amdgcn_s_memrealtime() - t0 > DEADLINE_TICKS) {
    __hip_atomic_store(tmo, call, __ATOMIC_RELAXED, __HIP_MEMORY_SCOPE_AGENT);
    return true;
  }
  return false;
}

__global__ void __launch_bounds__(1024, 4) gemm_rs_kernel(Params pin) {
  u64 ax = (u64)pin.x, aw_ = (u64)pin.w, asx = (u64)pin.sx, asw = (u64)pin.sw, aout = (u64)pin.out;
  u64 w0 = (u64)pin.ws[0], w1 = (u64)pin.ws[1], w2 = (u64)pin.ws[2], w3 = (u64)pin.ws[3];
  u64 w4 = (u64)pin.ws[4], w5 = (u64)pin.ws[5], w6 = (u64)pin.ws[6], w7 = (u64)pin.ws[7];
  int rank = pin.rank;
  asm volatile("" : "+s"(ax), "+s"(aw_), "+s"(asx), "+s"(asw), "+s"(aout), "+s"(rank),
                    "+s"(w0), "+s"(w1), "+s"(w2), "+s"(w3), "+s"(w4), "+s"(w5), "+s"(w6), "+s"(w7));
#define ws_of(k) sel8((k), w0, w1, w2, w3, w4, w5, w6, w7)
  const gchar* const px = (const gchar*)ax;
  const gchar* const pw = (const gchar*)aw_;
  __shared__ __attribute__((aligned(16))) unsigned char lds[LDS_CTL + 384];
  const int tid = threadIdx.x;
  const int lane = tid & 63;
  const int wave = __builtin_amdgcn_readfirstlane(tid >> 6);
  const int t = blockIdx.x;
  gchar* const myws = (gchar*)w0;
  gu32* const cnt = (gu32*)(myws + OFF_CNT + t * 128);
  gu32* const tmo = (gu32*)(myws + OFF_TMO);
  const int fr = lane & 15, fg = lane >> 4;
  const unsigned cnt_raw = __hip_atomic_load(cnt, __ATOMIC_RELAXED, __HIP_MEMORY_SCOPE_AGENT);
  const unsigned long long t0 = __builtin_amdgcn_s_memrealtime();
  const auto my_rs = mk_rsrc(myws, OFF_CNT);

  STW0(0);
  const bool comm = t < NCOMM;
  const int xcd = t & 7, idx = (t - NCOMM) >> 3;
  const int ti = idx & 3, tu = idx >> 2;
  const int tj = 8 + tu * 8 + xcd;
  const bool extra = tu < 4;
  volatile LDS_AS unsigned* ctl = (volatile LDS_AS unsigned*)(lds + LDS_CTL);
  volatile LDS_AS unsigned* ready = ctl;
  volatile LDS_AS unsigned* abortf = ctl + 8;
  volatile LDS_AS unsigned* invdone = ctl + 9;
  volatile LDS_AS unsigned* ownrdy = ctl + 12;
  volatile LDS_AS unsigned* consumed = ctl + 16;
  const int xs = (rank + wave) & (NR - 1);
  unsigned call = 0;

  const int er = lane >> 3, ec = lane & 7;
  auto export_w = [&](bool bar) {
    const int v = wave & 7, ek16 = (t & 1) * 4 + (v >> 1);
    const u32x4 wa = *(const GLB_AS u32x4*)(pw + (size_t)((t >> 6) * 128 + ek16 * 16 + (v & 1) * 8 + er) * N_ + (t & 62) * 64 + ec * 16);
    const unsigned w_img = ((t & ~1) + (ec >> 2)) * WPIECE + ek16 * 1024 + (ec & 3) * 256 + ((((v ^ ek16) & 1) * 8 + er) * 16);
    call = rfl(cnt_raw) + 1;
    if (bar) { asm volatile("s_waitcnt lgkmcnt(0)" :: "s"(call) : "memory"); __builtin_amdgcn_s_barrier(); }
    st16(nonzero(wa), my_rs, OFF_WE + (call & 3) * WSRC + w_img);
  };
  if (wave < 8) {
    const int e = (t - NCOMM) & (GRID - 1);
    const u32x4 xa = *(const GLB_AS u32x4*)(px + (size_t)(xs * MS + (e & 63) * 8 + er) * KS + (e >> 6) * 128 + ec * 16);
    const unsigned x_img = (e >> 1) * XFRAG + (ec >> 2) * 1024 + ((ec & 3) * 16 + (e & 1) * 8 + er) * 16;
    call = rfl(cnt_raw) + 1;
    if (wave > 0) {
      if (e < XPUSH) st16(nonzero(xa), mk_rsrc(ws_of(wave), OFF_CNT), OFF_X + (call % 3) * XSTAGE + rank * XSRC + x_img);
      else st16(nonzero(xa), my_rs, OFF_XE + (call & 3) * XSTAGE + xs * XSRC + x_img);
    }
    asm volatile("" ::: "memory");
    if (comm || !WSPLIT) export_w(false);
    __builtin_amdgcn_s_barrier();
    if (tid == 0) __hip_atomic_store(cnt, call, __ATOMIC_RELAXED, __HIP_MEMORY_SCOPE_AGENT);
  } else if (!comm) {
    if (wave == 8) {
      if (lane < 16) ctl[lane] = 0;
      consumed[lane] = 0;
    }
    if (WSPLIT) {
      sleep_until(t0, WDELAY);
      export_w(true);
    } else {
      call = rfl(cnt_raw) + 1;
      asm volatile("s_waitcnt lgkmcnt(0)" :: "s"(call) : "memory");
      __builtin_amdgcn_s_barrier();
    }
  }
  auto rezero3 = [&]() {
    const u32x4 z = {0u, 0u, 0u, 0u};
    st16(z, my_rs, OFF_X + ((call + 2) % 3) * XSTAGE + xs * XSRC + t * 1024 + lane * 16);
    st16(z, my_rs, OFF_XE + ((call + 2) & 3) * XSTAGE + xs * XSRC + t * 1024 + lane * 16);
    st16(z, my_rs, OFF_WE + ((call + 2) & 3) * WSRC + t * WPIECE + tid * 16);
  };

  if (comm) {
    if (wave < 8) {
      rezero3();
      if (t == 0 && wave == 0 && lane < NR)
        __hip_atomic_store((gu32*)(myws + OFF_TKT + (((call + 2) & 3) * 8 + lane) * 128), 0u, __ATOMIC_RELAXED, __HIP_MEMORY_SCOPE_AGENT);
      return;
    }
    const int pwv = wave - 8;
    const int g = t * PWAVES + pwv;
    if (pwv >= PWAVES || g >= NPULL * (NR - 1)) return;
    const int k = 1 + g / NPULL, q = g % NPULL;
    const int s = (rank + k) & (NR - 1);
    const u64 sp = (u64)ws_of(k);
    const u32x4 srd_s = {rfl((unsigned)sp), rfl((unsigned)(sp >> 32) & 0xffffu), OFF_CNT, 0x00020000u};
    const u32x4 srd_d = {rfl((unsigned)w0), rfl((unsigned)(w0 >> 32) & 0xffffu), OFF_CNT, 0x00020000u};
    constexpr unsigned NPAIR = NUNIT / 2, INV = 1u << 20;
    unsigned status = 0, sx0 = 0, sw0 = 0, dx0 = 0, dw0 = 0, tkt = 0;
    const unsigned warm = ld4(mk_rsrc(ws_of(k), OFF_CNT), OFF_XE + s * XSRC + (rank * 64 + q) * 128, 0);
#pragma nounroll
    for (unsigned pass = WARM ? 0 : 1; pass < 2; ++pass) {
      asm volatile("" : "+s"(pass));
      if (pass) {
        call = rfl(cnt_raw) + 1;
        asm volatile("" :: "v"(warm));
        sx0 = OFF_XE + (call & 3) * XSTAGE + rank * XSRC + XPUSH * 1024;
        sw0 = OFF_WE + (call & 3) * WSRC;
        dx0 = OFF_X + (call % 3) * XSTAGE + s * XSRC + XPUSH * 1024;
        dw0 = OFF_W + (call & 1) * WSTAGE + s * WSRC;
        tkt = OFF_TKT + ((call & 3) * 8 + k) * 128;
        sleep_until(t0, PULL0);
        {
          const unsigned e = (q + (lane % ARRS) * NPULL) % XPUSH;
          for (unsigned n = 1;; ++n) {
            const unsigned f = ld4(my_rs, dx0 - XPUSH * 1024 + (e >> 1) * XFRAG + (e & 1) * 128, 0);
            if (__builtin_popcountll(__builtin_amdgcn_ballot_w64(f != 0) & ((1ull << ARRS) - 1)) >= ARRN) break;
            if ((n & 63) == 0 && expired(tmo, call, t0)) return;
            __builtin_amdgcn_s_sleep(1);
          }
        }
        const auto src = mk_rsrc(ws_of(k), OFF_CNT);
        for (unsigned n = 1; !NOPROBE; ++n) {
          if (rfl(ld4(src, 0, sx0 + q * 2048)) != 0) break;
          if ((n & 63) == 0 && expired(tmo, call, t0)) return;
          __builtin_amdgcn_s_sleep(2);
        }
      }
      unsigned cur, nxt, s1, s2;
      u64 tnow;
#define VERIFY(N, X0, X1, X2, X3, L) \
      "s_cmp_lt_u32 %[cur], %[npair]\n\t" \
      "s_cbranch_scc0 .Lok" L "_%=\n\t" \
      "s_waitcnt vmcnt(" N ")\n\t" \
      ".Lchk" L "_%=:\n\t" \
      "v_min_u32 v121, " X2 ", " X3 "\n\t" \
      "v_min3_u32 v121, " X0 ", " X1 ", v121\n\t" \
      "v_cmp_eq_u32 vcc, 0, v121\n\t" \
      "s_nop 0\n\t" \
      "s_cbranch_vccnz .Lretry" L "_%=\n\t" \
      ".Lok" L "_%=:\n\t"
#define RETRY(X, OFS, L) \
      ".Lretry" L "_%=:\n\t" \
      "s_sleep 8\n\t" \
      "s_memrealtime %[tnow]\n\t" \
      "s_waitcnt lgkmcnt(0)\n\t" \
      "v_mov_b64 v[122:123], %[tnow]\n\t" \
      "v_cmp_lt_u64 vcc, %[tend], v[122:123]\n\t" \
      "s_nop 0\n\t" \
      "s_cbranch_vccnz .Lfail_%=\n\t" \
      "buffer_load_dwordx4 " X ", v119, %[srs], 0 offen " OFS " sc0 sc1\n\t" \
      "s_waitcnt vmcnt(0)\n\t" \
      "s_branch .Lchk" L "_%=\n\t"
#define HALF(XA, XA0, XA1, XA2, XA3, XB, XB0, XB1, XB2, XB3, YA, YB, TIN, TOUT, LA, LB) \
      VERIFY("3", XA0, XA1, XA2, XA3, LA) \
      "v_readfirstlane_b32 %[nxt], " TIN "\n\t" \
      "s_mov_b64 exec, %[em]\n\t" \
      "global_atomic_add " TOUT ", %[tkt], %[inc], %[ws] sc0\n\t" \
      "s_mov_b64 exec, -1\n\t" \
      "s_add_u32 %[nxt], %[nxt], %[base]\n\t" \
      "s_sub_u32 %[s1], %[nxt], %[xpair]\n\t" \
      "s_add_u32 %[s2], %[s1], %[rot]\n\t" \
      "s_and_b32 %[s2], %[s2], %[rmask]\n\t" \
      "s_andn2_b32 %[s1], %[s1], %[rmask]\n\t" \
      "s_or_b32 %[s2], %[s2], %[s1]\n\t" \
      "s_cmp_lt_u32 %[nxt], %[xpair]\n\t" \
      "s_cselect_b32 %[s2], %[nxt], %[s2]\n\t" \
      "s_cselect_b32 %[s1], %[sx], %[sw]\n\t" \
      "s_lshl_b32 %[s2], %[s2], 11\n\t" \
      "s_add_u32 %[s1], %[s1], %[s2]\n\t" \
      "s_cmp_lt_u32 %[nxt], %[npair]\n\t" \
      "s_cselect_b32 %[s1], %[s1], 0x80000000\n\t" \
      "v_add_u32 v118, %[s1], %[lo]\n\t" \
      "buffer_load_dwordx4 " YA ", v118, %[srs], 0 offen sc0 sc1\n\t" \
      "buffer_store_dwordx4 " XA ", v120, %[srd], 0 offen sc0 sc1\n\t" \
      VERIFY("4", XB0, XB1, XB2, XB3, LB) \
      "buffer_load_dwordx4 " YB ", v118, %[srs], 0 offen offset:1024 sc0 sc1\n\t" \
      "buffer_store_dwordx4 " XB ", v120, %[srd], 0 offen offset:1024 sc0 sc1\n\t" \
      "s_cmp_lt_u32 %[nxt], %[xpair]\n\t" \
      "s_cselect_b32 %[s1], %[dx], %[dw]\n\t" \
      "s_add_u32 %[s1], %[s1], %[s2]\n\t" \
      "s_cmp_lt_u32 %[nxt], %[npair]\n\t" \
      "s_cselect_b32 %[s1], %[s1], 0x80000000\n\t" \
      "v_mov_b32 v119, v118\n\t" \
      "v_add_u32 v120, %[s1], %[lo]\n\t" \
      PSTAMP \
      "s_mov_b32 %[cur], %[nxt]\n\t"
#define PSTAMP
#define PSTAMP_OPS
      asm volatile(
          "s_mov_b32 %[cur], %[inv]\n\t"
          "v_mov_b32 v117, %[first]\n\t"
          "v_mov_b32 v116, 0\n\t"
          "v_mov_b32 v120, 0x80000000\n\t"
          "s_nop 4\n\t"
          ".Lloop_%=:\n\t"
          HALF("v[100:103]", "v100", "v101", "v102", "v103", "v[104:107]", "v104", "v105", "v106", "v107",
               "v[108:111]", "v[112:115]", "v117", "v116", "a", "b")
          "s_cmp_ge_u32 %[cur], %[lim1]\n\t"
          "s_cbranch_scc1 .Ldone_%=\n\t"
          HALF("v[108:111]", "v108", "v109", "v110", "v111", "v[112:115]", "v112", "v113", "v114", "v115",
               "v[100:103]", "v[104:107]", "v116", "v117", "c", "d")
          "s_cmp_lt_u32 %[cur], %[npair]\n\t"
          "s_cbranch_scc1 .Lloop_%=\n\t"
          "s_branch .Ldone_%=\n\t"
          RETRY("v[100:103]", "", "a")
          RETRY("v[104:107]", "offset:1024", "b")
          RETRY("v[108:111]", "", "c")
          RETRY("v[112:115]", "offset:1024", "d")
          ".Lfail_%=:\n\t"
          "s_mov_b32 %[status], 1\n\t"
          ".Ldone_%=:\n\t"
          "s_waitcnt vmcnt(0)"
          : [cur] "=&s"(cur), [nxt] "=&s"(nxt), [s1] "=&s"(s1), [s2] "=&s"(s2), [tnow] "=&s"(tnow), [status] "+s"(status) PSTAMP_OPS
          : [srs] "s"(srd_s), [srd] "s"(srd_d), [ws] "s"(w0), [tend] "s"(t0 + DEADLINE_TICKS),
            [sx] "s"(rfl(sx0)), [sw] "s"(rfl(sw0)), [dx] "s"(rfl(dx0)), [dw] "s"(rfl(dw0)),
            [base] "s"(pass ? (unsigned)NPULL : INV), [lim1] "s"(pass ? NPAIR : 0xffffffffu),
            [first] "s"(rfl(pass ? (unsigned)(q - NPULL) : 0u)),
            [rot] "s"(rfl(ROTALL ? k * (ROTW / 8) * 4 : 0)), [rmask] "n"(ROTW * 4 - 1), [npair] "n"(NPAIR), [xpair] "n"(XU / 2), [inv] "n"(INV),
            [em] "s"((u64)pass), [tkt] "v"(tkt), [inc] "v"(1u), [lo] "v"(lane * 16)
          : "memory", "vcc", "scc", "v100", "v101", "v102", "v103", "v104", "v105", "v106", "v107", "v108", "v109",
            "v110", "v111", "v112", "v113", "v114", "v115", "v116", "v117", "v118", "v119", "v120", "v121", "v122", "v123");
    }
    if (status) __hip_atomic_store(tmo, call, __ATOMIC_RELAXED, __HIP_MEMORY_SCOPE_AGENT);
    if (pwv < 8) ST(48 + pwv);
    return;
  }

  if (wave == 8) {
    sleep_until(t0, OWN0);
    const gchar* w_even = pw + (size_t)fr * N_ + fg * 16 + tj * 64;
    const gchar* w_odd = pw + (size_t)(fr ^ 8) * N_ + fg * 16 + tj * 64;
    const gchar* w_ex = pw + (size_t)(fg * 16 + (fr ^ ((fg & 1) << 3))) * N_ + xcd * 64 + tu * 16;
    for (int kb = 0; kb < NKB; ++kb) {
#pragma unroll
      for (int v = 0; v < 8; ++v)
        __builtin_amdgcn_global_load_lds((const GLB_AS void*)(((v & 1) ? w_odd : w_even) + (size_t)(kb * 128 + v * 16) * N_),
                                         (LDS_AS void*)(lds + LDS_OWN + kb * SLOT + v * 1024), 16, 0, 0);
      if (extra) {
        __builtin_amdgcn_global_load_lds((const GLB_AS void*)(w_ex + (size_t)(kb * 128) * N_),
                                         (LDS_AS void*)(lds + LDS_OWN + kb * SLOT + 8192), 16, 0, 0);
        __builtin_amdgcn_global_load_lds((const GLB_AS void*)(w_ex + (size_t)(kb * 128 + 64) * N_),
                                         (LDS_AS void*)(lds + LDS_OWN + kb * SLOT + 9216), 16, 0, 0);
      }
      wait_vm<0>();
      if (lane == 0) ownrdy[kb] = 1;
    }
    if (DIRECT && tj >= NJ - ROTW) {
      sleep_until(t0, 1000 + (t & 63) * 20);
      __builtin_amdgcn_fence(__ATOMIC_ACQUIRE, "");
      wait_vm<0>();
      if (lane == 0) *invdone = 1;
    }
    return;
  }

  if (wave > 8) {
    const int sv = wave - 8;
    const int s = (rank + sv) & (NR - 1);
    const unsigned ex_lane = fg * 1024 + tu * 256 + fr * 16;
    sleep_until(t0, OWN0);
    const unsigned zo = OFF_W + ((call + 1) & 1) * WSTAGE + s * WSRC + ti * NJ * WPIECE;
    auto rezero = [&](int part) {
      const u32x4 z = {0u, 0u, 0u, 0u};
      st16(z, my_rs, zo + tj * WPIECE + lane * 16 + part * 3072);
      st16(z, my_rs, zo + tj * WPIECE + lane * 16 + part * 3072 + 1024);
      if (part < 2) {
        st16(z, my_rs, zo + tj * WPIECE + lane * 16 + part * 3072 + 2048);
        if (extra) st16(z, my_rs, zo + xcd * WPIECE + ex_lane + part * 4096);
      }
    };
    LDS_AS unsigned char* slot = (LDS_AS unsigned char*)lds + (sv - 1) * SLOT + lane * 16;
    const unsigned sb = OFF_W + (call & 1) * WSTAGE + s * WSRC;
    for (int kb = 0; kb < NKB; ++kb) {
      const unsigned so = sb + (kb * NJ + tj) * WPIECE, so_ex = sb + (kb * NJ + xcd) * WPIECE;
      const bool direct = DIRECT && kb == NKB - 1 && tj >= NJ - ROTW;
      const unsigned so_ind = direct ? sb + piece_at(NKB * NJ - ROTW - REFP, sv) * WPIECE : so;
      bool dead = false;
      for (unsigned it = 1;; ++it) {
        if (rfl(ld4(my_rs, 0, so_ind)) != 0) break;
        if ((it & 63) == 0 && expired(tmo, call, t0)) { dead = true; break; }
        __builtin_amdgcn_s_sleep(2);
      }
      u32x4 d[10];
      bool have = false;
      if (direct && !dead) {
        const unsigned long long tr = __builtin_amdgcn_s_memrealtime();
        while (!rfl(*invdone)) __builtin_amdgcn_s_sleep(2);
        sleep_until(tr, (pos_of(kb * NJ + tj, sv) - (NKB * NJ - ROTW) + REFP - LEAD) * SLOT_T / 10);
        const auto src = mk_rsrc(ws_of(sv), OFF_CNT);
        const unsigned eo = OFF_WE + (call & 3) * WSRC + (kb * NJ + tj) * WPIECE;
        unsigned m = 0xffffffffu;
#pragma unroll
        for (int v = 0; v < 8; ++v) d[v] = ld16<0>(src, lane * 16 + v * 1024, eo);
#pragma unroll
        for (int v = 0; v < 8; ++v) m = min(m, min4(d[v]));
        for (unsigned it = 1; __builtin_amdgcn_ballot_w64(m == 0) != 0 && !dead; ++it) {
          m = 0xffffffffu;
#pragma unroll
          for (int v = 0; v < 8; ++v) d[v] = ld16<AUX_SYS>(src, lane * 16 + v * 1024, eo);
#pragma unroll
          for (int v = 0; v < 8; ++v) m = min(m, min4(d[v]));
          if ((it & 63) == 0 && expired(tmo, call, t0)) dead = true;
        }
        have = true;
      }
      for (unsigned it = 1; !dead && !have; ++it) {
        unsigned m = 0xffffffffu;
#pragma unroll
        for (int v = 0; v < 8; ++v) d[v] = ld16<AUX_SYS>(my_rs, lane * 16 + v * 1024, so);
        if (extra) {
          d[8] = ld16<AUX_SYS>(my_rs, ex_lane, so_ex);
          d[9] = ld16<AUX_SYS>(my_rs, ex_lane + 4096, so_ex);
          m = min(min4(d[8]), min4(d[9]));
        }
#pragma unroll
        for (int v = 0; v < 8; ++v) m = min(m, min4(d[v]));
        if (__builtin_amdgcn_ballot_w64(m == 0) == 0) break;
        if ((it & 63) == 0 && expired(tmo, call, t0)) dead = true;
      }
      if (dead) { if (lane == 0) *abortf = 1; return; }
      while (__builtin_amdgcn_ballot_w64(consumed[sv * 8 + (lane & 7)] < (unsigned)kb) && !rfl(*abortf))
        __builtin_amdgcn_s_sleep(2);
#pragma unroll
      for (int v = 0; v < 8; ++v) *(LDS_AS u32x4*)(slot + v * 1024) = d[v];
      if (extra) { *(LDS_AS u32x4*)(slot + 8192) = d[8]; *(LDS_AS u32x4*)(slot + 9216) = d[9]; }
      asm volatile("s_waitcnt lgkmcnt(0)" ::: "memory");
      if (lane == 0) ready[sv] = kb + 1;
      ST(4 + kb * 8 + sv);
      if (kb < NKB - 1) rezero(kb);
      if (kb == 1 && ti == 0) {
        const unsigned f = ld4(my_rs, lane * 128, so + 2 * NJ * WPIECE);
        asm volatile("" :: "v"(f));
      }
    }
    return;
  }

  sleep_until(t0, OWN0);
  const int mb = ti * 8 + wave;
  const auto x_own_rs = mk_rsrc(px, M_ * KS);
  const unsigned x_own_lane = (unsigned)(rank * MS + mb * 16 + fr) * KS + fg * 16;
  const unsigned x_st_lane = OFF_X + (call % 3) * XSTAGE + mb * XFRAG + lane * 16;
  const float sc = *(const GLB_AS float*)asx * *(const GLB_AS float*)asw;
  auto ldx_own = [&](unsigned kb) {
    const u32x4 lo = ld16<0>(x_own_rs, x_own_lane, kb * 128);
    const u32x4 hi = ld16<0>(x_own_rs, x_own_lane + 64, kb * 128);
    return __builtin_shufflevector(lo, hi, 0, 1, 2, 3, 4, 5, 6, 7);
  };
  auto ldx_st = [&](int sv, unsigned kb) {
    const unsigned so = ((rank + sv) & (NR - 1)) * XSRC + kb * 32 * XFRAG;
    const u32x4 lo = ld16<AUX_SYS>(my_rs, x_st_lane, so);
    const u32x4 hi = ld16<AUX_SYS>(my_rs, x_st_lane + 1024, so);
    return __builtin_shufflevector(lo, hi, 0, 1, 2, 3, 4, 5, 6, 7);
  };
  const unsigned tr_lane = fg * 1024 + ((fg & 1) << 7) + fr * 8;

  f32x4 acc[5] = {};
  bool failed = false;
  auto step = [&](unsigned base, u32x8 xv) {
    const i32x8 bx = __builtin_bit_cast(i32x8, xv);
    unsigned tl = tr_lane;
    asm volatile("" : "+v"(tl));
    LDS_AS unsigned char* sl = (LDS_AS unsigned char*)lds + base;
    if (extra) {
      i32x8 aw;
      const unsigned te = tl - fg * 768 + 8192;
#pragma unroll
      for (int c = 0; c < 4; ++c) {
        const i32x2 r = __builtin_amdgcn_ds_read_tr8_b64_v2i32((LDS_AS i32x2*)(sl + (c >> 1) * 1024 + (te ^ ((c & 1) << 7))));
        aw[2 * c] = r[0]; aw[2 * c + 1] = r[1];
      }
      acc[4] = __builtin_amdgcn_mfma_scale_f32_16x16x128_f8f6f4(aw, bx, acc[4], 0, 0, 0, 0x7f7f7f7f, 0, 0x7f7f7f7f);
    }
#pragma unroll
    for (int q = 0; q < 4; ++q) {
      i32x8 aw;
#pragma unroll
      for (int c = 0; c < 4; ++c) {
        const i32x2 r = __builtin_amdgcn_ds_read_tr8_b64_v2i32((LDS_AS i32x2*)(sl + (c >> 1) * 4096 + q * 256 + (tl ^ ((c & 1) << 7))));
        aw[2 * c] = r[0]; aw[2 * c + 1] = r[1];
      }
      acc[q] = __builtin_amdgcn_mfma_scale_f32_16x16x128_f8f6f4(aw, bx, acc[q], 0, 0, 0, 0x7f7f7f7f, 0, 0x7f7f7f7f);
    }
  };
#pragma unroll
  for (unsigned kb = 0; kb < NKB; ++kb) {
    const u32x8 xv = ldx_own(kb);
    while (!rfl(ownrdy[kb])) __builtin_amdgcn_s_sleep(1);
    step(LDS_OWN + kb * SLOT, xv);
  }
  while (!((unsigned)__builtin_amdgcn_ballot_w64(ctl[lane & 15] != 0u) & 0x1feu)) __builtin_amdgcn_s_sleep(4);
  rezero3();
  unsigned kb = 0;
  for (unsigned stop = DRY ? NKB - 1 : NKB;; stop = NKB) {
    asm volatile("" : "+s"(stop));
    for (; kb < stop && !failed; ++kb) {
      u32x8 xf[NR];
      for (unsigned it = 1;; ++it) {
        unsigned m = 0xffffffffu;
#pragma unroll
        for (int sv = 1; sv < NR; ++sv) xf[sv] = ldx_st(sv, kb);
#pragma unroll
        for (int sv = 1; sv < NR; ++sv) m = min(m, min8(xf[sv]));
        if (__builtin_amdgcn_ballot_w64(m == 0) == 0) break;
        if ((it & 15) == 0 && expired(tmo, call, t0)) { failed = true; break; }
        __builtin_amdgcn_s_sleep(32);
      }
      unsigned pend = failed ? 0u : 0xfeu;
      while (pend) {
        unsigned rdy;
        for (;;) {
          const unsigned rv = ctl[lane & 15];
          rdy = (unsigned)__builtin_amdgcn_ballot_w64(rv > ((lane & 15) < 8 ? kb : 0u)) & (pend | 0x100u);
          if (rdy) break;
          __builtin_amdgcn_s_sleep(1);
        }
        pend &= ~rdy;
        if (rdy & 0x100u) { failed = true; pend = 0; rdy = 0; }
#pragma unroll
        for (int sv = 1; sv < NR; ++sv)
          if (rdy & (1u << sv)) {
            step((sv - 1) * SLOT, xf[sv]);
            asm volatile("s_waitcnt lgkmcnt(0)" ::: "memory");
            consumed[sv * 8 + wave] = kb + 1;
          }
      }
    }
    if (failed && lane == 0) *abortf = 1;

    STW0(2);
    const float sce = failed ? __builtin_nanf("") : sc;
    const auto o_rs = mk_rsrc((gchar*)aout, stop == NKB ? MS * N_ * 4 : 0);
    auto silu4 = [&](f32x4 a) {
      u32x4 o;
#pragma unroll
      for (int e = 0; e < 4; ++e) {
        const float y = a[e] * sce;
        const float z = __builtin_amdgcn_fmed3f(y, -60.0f, 60.0f);
        o[e] = __builtin_bit_cast(unsigned, y * __builtin_amdgcn_rcpf(1.0f + __expf(-z)));
      }
      return o;
    };
    const unsigned o_pair = ((unsigned)(mb * 16 + (fr & 7)) * N_ + (fr >> 3) * 16 + fg * 4 + tj * 64) * 4;
#pragma unroll
    for (int q = 0; q < 4; q += 2) {
      const u32x4 o0 = silu4(acc[q]), o1 = silu4(acc[q + 1]);
      u32x4 lo, hi;
#pragma unroll
      for (int e = 0; e < 4; ++e) {
        const unsigned snd = fr < 8 ? o1[e] : o0[e];
        const unsigned rcv = __builtin_amdgcn_update_dpp(0u, snd, 0x128  , 0xf, 0xf, false);
        lo[e] = fr < 8 ? o0[e] : rcv;
        hi[e] = fr < 8 ? rcv : o1[e];
      }
      st16(lo, o_rs, o_pair + q * 64);
      st16(hi, o_rs, o_pair + q * 64 + 8 * N_ * 4);
    }
    if (extra) st16(silu4(acc[4]), o_rs, ((unsigned)(mb * 16 + fr) * N_ + fg * 4 + xcd * 64 + tu * 16) * 4);
    if (stop == NKB) break;
  }
}

}

extern "C" void kernel_launch(int rank, int nranks, ncclComm_t comm,
                              hipStream_t stream,
                              void* const* d_in, const int* in_sizes, int n_in,
                              void* d_out, int out_size,
                              const int* in_dtypes, int out_dtype,
                              void* const* d_out_all,
                              void* const* d_ws_all, size_t ws_bytes) {
  if (nranks != NR || (unsigned)rank >= (unsigned)NR || ws_bytes < WS_NEED || n_in < 4) {
    fprintf(stderr, "kernel_launch: unexpected configuration\n");
    return;
  }
  Params p;
  p.x = (const uint8_t*)d_in[0];
  p.w = (const uint8_t*)d_in[1];
  p.sx = (const float*)d_in[2];
  p.sw = (const float*)d_in[3];
  p.out = (float*)d_out;
  for (int i = 0; i < NR; ++i) p.ws[i] = (char*)d_ws_all[(rank + i) & (NR - 1)];
  p.rank = rank;
  hipLaunchKernelGGL(gemm_rs_kernel, dim3(GRID), dim3(1024), 0, stream, p);
}
```

```cpp
#include <hip/hip_runtime.h>
#include <rccl/rccl.h>
#include <cstdio>
#include <cstdint>

namespace {

constexpr int NR = 8;
constexpr int M_ = 4096, K_ = 4096, N_ = 4096;
constexpr int MS = M_ / NR;
constexpr int KS = K_ / NR;
constexpr int NKB = KS / 128;
constexpr int NJ = N_ / 64;
#define XPUSH 108
#define OWN0 200
#define NPULL_ 36
#define SLOT_T 134
#define MONN 16
#define MON0 0
#define ARRC 28
#define RWARM 0
#define WDELAY 0
#define XDELAY 0
#define SNAP 110
#define FOFF 110
#define DRY 1
constexpr int ROTW = 16;
constexpr int GRID = 256;
constexpr int NCOMM = 32;
constexpr int NPULL = NPULL_;
constexpr int PWAVES = (NPULL * (NR - 1) + NCOMM - 1) / NCOMM;
constexpr int XU = 256 - XPUSH;
constexpr int NUNIT = XU + (NKB * NJ - ROTW) * 8;
constexpr unsigned NPAIR = NUNIT / 2;
static_assert(XU % 2 == 0 && XPUSH <= GRID && PWAVES <= 8, "");

constexpr unsigned WPIECE = 8192;
constexpr unsigned WSRC = NKB * NJ * WPIECE;
constexpr unsigned WSTAGE = NR * WSRC;
constexpr unsigned XFRAG = 2048;
constexpr unsigned XSRC = NKB * 32 * XFRAG;
constexpr unsigned XSTAGE = NR * XSRC;
constexpr unsigned OFF_W = 0;
constexpr unsigned OFF_X = OFF_W + 2 * WSTAGE;
constexpr unsigned OFF_WE = OFF_X + 4 * XSTAGE;
constexpr unsigned OFF_XE = OFF_WE + 4 * WSRC;
constexpr unsigned OFF_CNT = OFF_XE + 4 * XSTAGE;
constexpr unsigned OFF_TMO = OFF_CNT + GRID * 128;
constexpr unsigned OFF_TKT = OFF_TMO + 128;
constexpr unsigned WS_NEED = OFF_TKT + 32 * 128;

constexpr unsigned SLOT = WPIECE + 2048;
constexpr unsigned LDS_OWN = 7 * SLOT;
constexpr unsigned LDS_CTL = LDS_OWN + NKB * SLOT;

#define ST(i) do {} while (0)
#define STW0(i) do {} while (0)
constexpr unsigned long long DEADLINE_TICKS = 50000000ull;

typedef unsigned u32x4 __attribute__((ext_vector_type(4)));
typedef unsigned u32x8 __attribute__((ext_vector_type(8)));
typedef int i32x2 __attribute__((ext_vector_type(2)));
typedef int i32x8 __attribute__((ext_vector_type(8)));
typedef float f32x4 __attribute__((ext_vector_type(4)));
#define LDS_AS __attribute__((address_space(3)))
#define GLB_AS __attribute__((address_space(1)))
typedef unsigned long long u64;
typedef GLB_AS char gchar;
typedef GLB_AS unsigned gu32;
constexpr int AUX_SYS = 17;

struct Params {
  const uint8_t* x;
  const uint8_t* w;
  const float* sx;
  const float* sw;
  float* out;
  char* ws[NR];
  int rank;
};

__device__ __forceinline__ __amdgpu_buffer_rsrc_t mk_rsrc(const gchar* p, unsigned bytes) {
  return __builtin_amdgcn_make_buffer_rsrc((void*)(u64)p, 0, bytes, 0x00020000);
}
__device__ __forceinline__ void st16(u32x4 v, __amdgpu_buffer_rsrc_t rs, unsigned off) {
  __builtin_amdgcn_raw_buffer_store_b128(v, rs, off, 0, AUX_SYS);
}
template <int AUX> __device__ __forceinline__ u32x4 ld16(__amdgpu_buffer_rsrc_t rs, unsigned voff, unsigned soff) {
  return __builtin_amdgcn_raw_buffer_load_b128(rs, voff, soff, AUX);
}
__device__ __forceinline__ unsigned ld4(__amdgpu_buffer_rsrc_t rs, unsigned voff, unsigned soff) {
  return __builtin_amdgcn_raw_buffer_load_b32(rs, voff, soff, AUX_SYS);
}
template <int N> __device__ __forceinline__ void wait_vm() {
  asm volatile("s_waitcnt vmcnt(%0)" :: "n"(N) : "memory");
}
__device__ __forceinline__ gchar* sel8(int k, u64 a0, u64 a1, u64 a2, u64 a3, u64 a4, u64 a5, u64 a6, u64 a7) {
  const u64 b0 = (k & 1) ? a1 : a0, b1 = (k & 1) ? a3 : a2, b2 = (k & 1) ? a5 : a4, b3 = (k & 1) ? a7 : a6;
  const u64 c0 = (k & 2) ? b1 : b0, c1 = (k & 2) ? b3 : b2;
  return (gchar*)((k & 4) ? c1 : c0);
}
__device__ __forceinline__ unsigned rfl(unsigned v) { return __builtin_amdgcn_readfirstlane(v); }
__device__ __forceinline__ u32x4 nonzero(u32x4 v) {
#pragma unroll
  for (int e = 0; e < 4; ++e) v[e] = v[e] ? v[e] : 0x80u;
  return v;
}
__device__ __forceinline__ unsigned min4(u32x4 v) { return min(min(v[0], v[1]), min(v[2], v[3])); }
__device__ __forceinline__ unsigned min8(u32x8 v) {
  return min(min(min(v[0], v[1]), min(v[2], v[3])), min(min(v[4], v[5]), min(v[6], v[7])));
}
__device__ __forceinline__ void sleep_until(unsigned long long t0, unsigned ticks) {
  while (__builtin_amdgcn_s_memrealtime() - t0 < ticks) __builtin_amdgcn_s_sleep(2);
}
__device__ __forceinline__ bool expired(gu32* tmo, unsigned call, unsigned long long t0) {
  if (rfl(__hip_atomic_load(tmo, __ATOMIC_RELAXED, __HIP_MEMORY_SCOPE_AGENT)) == call ||
      __builtin_amdgcn_s_memrealtime() - t0 > DEADLINE_TICKS) {
    __hip_atomic_store(tmo, call, __ATOMIC_RELAXED, __HIP_MEMORY_SCOPE_AGENT);
    return true;
  }
  return false;
}

__global__ void __launch_bounds__(1024, 4) gemm_rs_kernel(Params pin) {
  u64 ax = (u64)pin.x, aw_ = (u64)pin.w, asx = (u64)pin.sx, asw = (u64)pin.sw, aout = (u64)pin.out;
  u64 w0 = (u64)pin.ws[0], w1 = (u64)pin.ws[1], w2 = (u64)pin.ws[2], w3 = (u64)pin.ws[3];
  u64 w4 = (u64)pin.ws[4], w5 = (u64)pin.ws[5], w6 = (u64)pin.ws[6], w7 = (u64)pin.ws[7];
  int rank = pin.rank;
  asm volatile("" : "+s"(ax), "+s"(aw_), "+s"(asx), "+s"(asw), "+s"(aout), "+s"(rank),
                    "+s"(w0), "+s"(w1), "+s"(w2), "+s"(w3), "+s"(w4), "+s"(w5), "+s"(w6), "+s"(w7));
#define ws_of(k) sel8((k), w0, w1, w2, w3, w4, w5, w6, w7)
  const gchar* const px = (const gchar*)ax;
  const gchar* const pw = (const gchar*)aw_;
  __shared__ __attribute__((aligned(16))) unsigned char lds[LDS_CTL + 384];
  const int tid = threadIdx.x;
  const int lane = tid & 63;
  const int wave = __builtin_amdgcn_readfirstlane(tid >> 6);
  const int t = blockIdx.x;
  gchar* const myws = (gchar*)w0;
  gu32* const cnt = (gu32*)(myws + OFF_CNT + t * 128);
  gu32* const tmo = (gu32*)(myws + OFF_TMO);
  const int fr = lane & 15, fg = lane >> 4;
  const int xs = (rank + wave) & (NR - 1);
  const bool comm = t < NCOMM;
  const int er = lane >> 3, ec = lane & 7;
  const int e = t - NCOMM;
  const bool isx = wave < 8;
  const int ek = 1 + (unsigned)(e + NCOMM + 3 + wave) % (NR - 1), ex = wave ? e : GRID - NCOMM + e / (NR - 1);
  const bool push = isx && ex < XPUSH;
  auto w_src = [&](int ew) {
    return pw + (size_t)(((ew >> 6) * 128 + ((ew & 1) * 4 + ((wave >> 1) & 3)) * 16 + (wave & 1) * 8) * N_ + (ew & 62) * 64);
  };
  auto w_img = [&](int ew) -> unsigned {
    return OFF_WE + (ew & ~1) * WPIECE + ((ew & 1) * 4 + ((wave >> 1) & 3)) * 1024 + ((wave ^ (wave >> 1)) & 1) * 128;
  };
  const gchar* esrc = isx ? px + (size_t)((((rank + ek) & (NR - 1)) * MS + (ex & 63) * 8) * KS + (ex >> 6) * 128) : w_src(e);
  u32x4 ev;
  if (!comm) ev = *(const GLB_AS u32x4*)(esrc + (unsigned)(er * (isx ? KS : N_) + ec * 16));
  const unsigned cnt_raw = __hip_atomic_load(cnt, __ATOMIC_RELAXED, __HIP_MEMORY_SCOPE_AGENT);
  const unsigned long long t0 = __builtin_amdgcn_s_memrealtime();
  const auto my_rs = mk_rsrc(myws, OFF_CNT);
  const unsigned e_lane = (ec >> 2) * (isx ? 1024 : WPIECE) + (ec & 3) * 256 + er * 16;
  unsigned e_img = e_lane + (isx ? (ex >> 1) * XFRAG + (ex & 1) * 128 + (push ? OFF_X + rank * XSRC : OFF_XE + ((rank + ek) & (NR - 1)) * XSRC)
                                 : w_img(e));
  u64 e_base = (u64)ws_of(push ? ek : 0);
  asm volatile("" : "+s"(e_base), "+v"(e_img));
  const unsigned call = rfl(cnt_raw) + 1;
  static_assert(XSTAGE == WSRC, "");
  if (!comm) {
    st16(nonzero(ev), mk_rsrc((gchar*)e_base, OFF_CNT), e_img + (call & 3) * XSTAGE);
    if (!isx && e < NCOMM) {
      const u32x4 ev2 = *(const GLB_AS u32x4*)(w_src(GRID - NCOMM + e) + (unsigned)(er * N_ + ec * 16));
      st16(nonzero(ev2), my_rs, e_lane + w_img(GRID - NCOMM + e) + (call & 3) * XSTAGE);
    }
  }

  const int xcd = t & 7, idx = (t - NCOMM) >> 3;
  const int ti = idx & 3, tu = idx >> 2;
  const int tl = 8 + tu * 8 + xcd;
  const int joff = ((rank & 3) + 1) * ROTW;
  const int tj = (tl + joff) & (NJ - 1);
  const int tx = (xcd + joff) & (NJ - 1);
  const bool extra = tu < 4;
  const bool win = tl >= NJ - ROTW;
  const int kbs = rank >> 2;
  auto kb_of = [&](int r) { return !win ? r : r == NKB - 1 ? kbs : r + (r >= kbs); };
  volatile LDS_AS unsigned* ctl = (volatile LDS_AS unsigned*)(lds + LDS_CTL);
  volatile LDS_AS unsigned* ready = ctl;
  volatile LDS_AS unsigned* abortf = ctl + 8;
  volatile LDS_AS unsigned* invdone = ctl + 9;
  volatile LDS_AS unsigned* ownrdy = ctl + 12;
  volatile LDS_AS unsigned* consumed = ctl + 16;
  volatile LDS_AS unsigned* sched = ctl + 80;

  if (wave == 8) {
    if (lane < 16) ctl[lane] = 0;
    consumed[lane] = 0;
    asm volatile("s_waitcnt lgkmcnt(0)" ::: "memory");
  }
  asm volatile("" :: "s"(call));
  __builtin_amdgcn_s_barrier();
  if (tid == 0) __hip_atomic_store(cnt, call, __ATOMIC_RELAXED, __HIP_MEMORY_SCOPE_AGENT);
  auto rezero3 = [&]() {
    const u32x4 z = {0u, 0u, 0u, 0u};
    st16(z, my_rs, OFF_X + ((call + 2) & 3) * XSTAGE + xs * XSRC + t * 1024 + lane * 16);
    st16(z, my_rs, OFF_XE + ((call + 2) & 3) * XSTAGE + xs * XSRC + t * 1024 + lane * 16);
    st16(z, my_rs, OFF_WE + ((call + 2) & 3) * WSRC + t * WPIECE + tid * 16);
  };

  if (comm) {
    if (wave < 8) {
      if (wave < 2) {
        const int mk = 1 + min(t * PWAVES + wave * (PWAVES - 1), NPULL * (NR - 1) - 1) / NPULL;
        const unsigned b0 = OFF_X + (call & 3) * XSTAGE + ((rank + mk) & (NR - 1)) * XSRC;
        const unsigned se = (((lane & (MONN - 1)) * 64 / MONN + t % (64 / MONN)) * XPUSH / 64);
        sleep_until(t0, MON0);
        for (unsigned n = 1;; ++n) {
          const unsigned f = ld4(my_rs, b0 + (se >> 1) * XFRAG + (se & 1) * 128, 0);
          if (__builtin_popcountll(__builtin_amdgcn_ballot_w64(f != 0)) >= ARRC) break;
          if ((n & 63) == 0 && expired(tmo, call, t0)) break;
        }
        if (lane == 0) ctl[mk] = 1;
      }
      __builtin_amdgcn_s_barrier();
      rezero3();
      if (t == 0 && wave == 0 && lane < 2 * NR)
        __hip_atomic_store((gu32*)(myws + OFF_TKT + ((call + 2) & 3) * 8 * 128 + (lane < NR ? lane * 128 : (lane - NR) * 4)),
                           0u, __ATOMIC_RELAXED, __HIP_MEMORY_SCOPE_AGENT);
      return;
    }
    const int pwv = wave - 8;
    const int g = t * PWAVES + pwv;
    if (pwv >= PWAVES || g >= NPULL * (NR - 1)) return;
    const int k = 1 + g / NPULL, q = g % NPULL;
    const int s = (rank + k) & (NR - 1);
    const u64 sp = (u64)ws_of(k);
    const u32x4 srd_s = {rfl((unsigned)sp), rfl((unsigned)(sp >> 32) & 0xffffu), OFF_CNT, 0x00020000u};
    const u32x4 srd_d = {rfl((unsigned)w0), rfl((unsigned)(w0 >> 32) & 0xffffu), OFF_CNT, 0x00020000u};
    constexpr unsigned INV = 1u << 20;
    u64 btab = 0;
#pragma unroll
    for (int b = 0; b < 15; ++b) {
      const int r = b >> 2, lg = b & 3;
      btab |= (u64)((lg == 3 ? r + (r >= kbs) : r) * 4 + ((lg + rank + 1) & 3)) << (4 * b);
    }
    unsigned status = 0;
    const unsigned sx0 = OFF_XE + (call & 3) * XSTAGE + rank * XSRC + XPUSH * 1024, sw0 = OFF_WE + (call & 3) * WSRC;
    const unsigned dx0 = OFF_X + (call & 3) * XSTAGE + s * XSRC + XPUSH * 1024, dw0 = OFF_W + (call & 1) * WSTAGE + s * WSRC;
    const unsigned tkt = OFF_TKT + ((call & 3) * 8 + k) * 128;
    while (!rfl(ctl[k])) __builtin_amdgcn_s_sleep(1);
    {
      unsigned cur, nxt, s1, s2;
      u64 tnow;
#define VERIFY(N, X0, X1, X2, X3, L) \
      "s_cmp_lt_u32 %[cur], %[npair]\n\t" \
      "s_cbranch_scc0 .Lok" L "_%=\n\t" \
      "s_waitcnt vmcnt(" N ")\n\t" \
      ".Lchk" L "_%=:\n\t" \
      "v_min_u32 v121, " X2 ", " X3 "\n\t" \
      "v_min3_u32 v121, " X0 ", " X1 ", v121\n\t" \
      "v_cmp_eq_u32 vcc, 0, v121\n\t" \
      "s_nop 0\n\t" \
      "s_cbranch_vccnz .Lretry" L "_%=\n\t" \
      ".Lok" L "_%=:\n\t"
#define RETRY(X, OFS, L) \
      ".Lretry" L "_%=:\n\t" \
      "s_sleep 8\n\t" \
      "s_memrealtime %[tnow]\n\t" \
      "s_waitcnt lgkmcnt(0)\n\t" \
      "v_mov_b64 v[122:123], %[tnow]\n\t" \
      "v_cmp_lt_u64 vcc, %[tend], v[122:123]\n\t" \
      "s_nop 0\n\t" \
      "s_cbranch_vccnz .Lfail_%=\n\t" \
      "buffer_load_dwordx4 " X ", v119, %[srs], 0 offen " OFS " sc0 sc1\n\t" \
      "s_waitcnt vmcnt(0)\n\t" \
      "s_branch .Lchk" L "_%=\n\t"
#define HALF(XA, XA0, XA1, XA2, XA3, XB, XB0, XB1, XB2, XB3, YA, YB, TIN, TOUT, LA, LB) \
      VERIFY("3", XA0, XA1, XA2, XA3, LA) \
      "v_readfirstlane_b32 %[nxt], " TIN "\n\t" \
      "s_mov_b64 exec, 1\n\t" \
      "global_atomic_add " TOUT ", %[tkt], %[inc], %[ws] sc0\n\t" \
      "s_mov_b64 exec, -1\n\t" \
      "s_add_u32 %[nxt], %[nxt], %[base]\n\t" \
      "s_sub_u32 %[s1], %[nxt], %[xpair]\n\t" \
      "s_add_u32 %[s2], %[s1], %[rot]\n\t" \
      "s_and_b32 %[s2], %[s2], %[rmask]\n\t" \
      "s_lshr_b32 %[s1], %[s1], 6\n\t" \
      "s_lshl_b32 %[s1], %[s1], 2\n\t" \
      "s_lshr_b64 vcc, %[btab], %[s1]\n\t" \
      "s_and_b32 %[s1], vcc_lo, 15\n\t" \
      "s_lshl_b32 %[s1], %[s1], 6\n\t" \
      "s_or_b32 %[s2], %[s2], %[s1]\n\t" \
      "s_cmp_lt_u32 %[nxt], %[xpair]\n\t" \
      "s_cselect_b32 %[s2], %[nxt], %[s2]\n\t" \
      "s_cselect_b32 %[s1], %[sx], %[sw]\n\t" \
      "s_lshl_b32 %[s2], %[s2], 11\n\t" \
      "s_add_u32 %[s1], %[s1], %[s2]\n\t" \
      "s_cmp_lt_u32 %[nxt], %[npair]\n\t" \
      "s_cselect_b32 %[s1], %[s1], 0x80000000\n\t" \
      "v_add_u32 v118, %[s1], %[lo]\n\t" \
      "buffer_load_dwordx4 " YA ", v118, %[srs], 0 offen sc0 sc1\n\t" \
      "buffer_store_dwordx4 " XA ", v120, %[srd], 0 offen sc0 sc1\n\t" \
      VERIFY("4", XB0, XB1, XB2, XB3, LB) \
      "buffer_load_dwordx4 " YB ", v118, %[srs], 0 offen offset:1024 sc0 sc1\n\t" \
      "buffer_store_dwordx4 " XB ", v120, %[srd], 0 offen offset:1024 sc0 sc1\n\t" \
      "s_cmp_lt_u32 %[nxt], %[xpair]\n\t" \
      "s_cselect_b32 %[s1], %[dx], %[dw]\n\t" \
      "s_add_u32 %[s1], %[s1], %[s2]\n\t" \
      "s_cmp_lt_u32 %[nxt], %[npair]\n\t" \
      "s_cselect_b32 %[s1], %[s1], 0x80000000\n\t" \
      "v_mov_b32 v119, v118\n\t" \
      "v_add_u32 v120, %[s1], %[lo]\n\t" \
      PSTAMP \
      "s_mov_b32 %[cur], %[nxt]\n\t"
#define PSTAMP
#define PSTAMP_OPS
      asm volatile(
          "s_mov_b32 %[cur], %[inv]\n\t"
          "v_mov_b32 v117, %[first]\n\t"
          "v_mov_b32 v120, 0x80000000\n\t"
          "s_nop 4\n\t"
          ".Lloop_%=:\n\t"
          HALF("v[100:103]", "v100", "v101", "v102", "v103", "v[104:107]", "v104", "v105", "v106", "v107",
               "v[108:111]", "v[112:115]", "v117", "v116", "a", "b")
          "s_cmp_ge_u32 %[cur], %[npair]\n\t"
          "s_cbranch_scc1 .Ldone_%=\n\t"
          HALF("v[108:111]", "v108", "v109", "v110", "v111", "v[112:115]", "v112", "v113", "v114", "v115",
               "v[100:103]", "v[104:107]", "v116", "v117", "c", "d")
          "s_cmp_lt_u32 %[cur], %[npair]\n\t"
          "s_cbranch_scc1 .Lloop_%=\n\t"
          "s_branch .Ldone_%=\n\t"
          RETRY("v[100:103]", "", "a")
          RETRY("v[104:107]", "offset:1024", "b")
          RETRY("v[108:111]", "", "c")
          RETRY("v[112:115]", "offset:1024", "d")
          ".Lfail_%=:\n\t"
          "s_mov_b32 %[status], 1\n\t"
          ".Ldone_%=:\n\t"
          "s_waitcnt vmcnt(0)"
          : [cur] "=&s"(cur), [nxt] "=&s"(nxt), [s1] "=&s"(s1), [s2] "=&s"(s2), [tnow] "=&s"(tnow), [status] "+s"(status) PSTAMP_OPS
          : [srs] "s"(srd_s), [srd] "s"(srd_d), [ws] "s"(w0), [tend] "s"(t0 + DEADLINE_TICKS),
            [sx] "s"(rfl(sx0)), [sw] "s"(rfl(sw0)), [dx] "s"(rfl(dx0)), [dw] "s"(rfl(dw0)),
            [base] "n"(NPULL), [first] "s"(rfl((unsigned)(q - NPULL))),
            [btab] "s"(btab), [rot] "s"(rfl(k * (ROTW / 8) * 4)), [rmask] "n"(ROTW * 4 - 1), [npair] "n"(NPAIR), [xpair] "n"(XU / 2), [inv] "n"(INV),
            [tkt] "v"(tkt), [inc] "v"(1u), [lo] "v"(lane * 16)
          : "memory", "vcc", "scc", "v100", "v101", "v102", "v103", "v104", "v105", "v106", "v107", "v108", "v109",
            "v110", "v111", "v112", "v113", "v114", "v115", "v116", "v117", "v118", "v119", "v120", "v121", "v122", "v123");
    }
    if (status) __hip_atomic_store(tmo, call, __ATOMIC_RELAXED, __HIP_MEMORY_SCOPE_AGENT);
    if (pwv < 8) ST(48 + pwv);
    return;
  }

  if (wave == 8) {
    sleep_until(t0, OWN0);
    const gchar* w_even = pw + (size_t)fr * N_ + fg * 16 + tj * 64;
    const gchar* w_odd = pw + (size_t)(fr ^ 8) * N_ + fg * 16 + tj * 64;
    const gchar* w_ex = pw + (size_t)(fg * 16 + (fr ^ ((fg & 1) << 3))) * N_ + tx * 64 + tu * 16;
    for (int kb = 0; kb < NKB; ++kb) {
#pragma unroll
      for (int v = 0; v < 8; ++v)
        __builtin_amdgcn_global_load_lds((const GLB_AS void*)(((v & 1) ? w_odd : w_even) + (size_t)(kb * 128 + v * 16) * N_),
                                         (LDS_AS void*)(lds + LDS_OWN + kb * SLOT + v * 1024), 16, 0, 0);
      if (extra) {
        __builtin_amdgcn_global_load_lds((const GLB_AS void*)(w_ex + (size_t)(kb * 128) * N_),
                                         (LDS_AS void*)(lds + LDS_OWN + kb * SLOT + 8192), 16, 0, 0);
        __builtin_amdgcn_global_load_lds((const GLB_AS void*)(w_ex + (size_t)(kb * 128 + 64) * N_),
                                         (LDS_AS void*)(lds + LDS_OWN + kb * SLOT + 9216), 16, 0, 0);
      }
      wait_vm<0>();
      if (lane == 0) ownrdy[kb] = 1;
    }
    gu32* const tk = (gu32*)(myws + OFF_TKT + (call & 3) * 8 * 128);
    if (win) {
      sleep_until(t0, 1000 + (t & 63) * 20);
      __builtin_amdgcn_fence(__ATOMIC_ACQUIRE, "");
      wait_vm<0>();
      for (unsigned it = 1;; ++it) {
        const unsigned f = __hip_atomic_load(tk + (lane & 7), __ATOMIC_RELAXED, __HIP_MEMORY_SCOPE_AGENT);
        if (__builtin_amdgcn_ballot_w64(f == 0) == 0) {
          sched[lane & 7] = f;
          asm volatile("s_waitcnt lgkmcnt(0)" ::: "memory");
          if (lane == 0) *invdone = 1;
          break;
        }
        if ((it & 63) == 0 && expired(tmo, call, t0)) { if (lane == 0) *abortf = 1; break; }
        __builtin_amdgcn_s_sleep(8);
      }
    } else if (t == NCOMM + 4 * 32) {
      for (unsigned it = 1;; ++it) {
        const unsigned ta = (unsigned)__builtin_amdgcn_s_memrealtime();
        const unsigned f = __hip_atomic_load(tk + (1 + lane % 7) * 32, __ATOMIC_RELAXED, __HIP_MEMORY_SCOPE_AGENT);
        unsigned mx = 0;
#pragma unroll
        for (int k = 0; k < NR - 1; ++k) mx = max(mx, (unsigned)__builtin_amdgcn_readlane(f, k));
        const unsigned tb = (unsigned)__builtin_amdgcn_s_memrealtime();
        if (mx + SNAP >= NPAIR - NPULL && (tb - ta < 80 || mx + SNAP / 2 >= NPAIR - NPULL)) {
          const int left = max((int)(NPAIR - NPULL) - (int)mx, 0);
          const unsigned fmin = ta + 25 + left * SLOT_T / 40 + FOFF;
          const unsigned lag = __builtin_amdgcn_update_dpp(0u, min((mx - f + 2) / 4, 200u) + 1, 0x111  , 0xf, 0xf, false);
          if (lane < NR) __hip_atomic_store(tk + lane, lane ? lag : fmin | 1u, __ATOMIC_RELAXED, __HIP_MEMORY_SCOPE_AGENT);
          break;
        }
        if ((it & 63) == 0 && expired(tmo, call, t0)) break;
        __builtin_amdgcn_s_sleep(8);
      }
    }
    return;
  }

  if (wave > 8) {
    const int sv = wave - 8;
    const int s = (rank + sv) & (NR - 1);
    const unsigned ex_lane = fg * 1024 + tu * 256 + fr * 16;
    sleep_until(t0, OWN0);
    const unsigned zo = OFF_W + ((call + 1) & 1) * WSTAGE + s * WSRC + ti * NJ * WPIECE;
    auto rezero = [&](int part) {
      const u32x4 z = {0u, 0u, 0u, 0u};
      st16(z, my_rs, zo + tj * WPIECE + lane * 16 + part * 3072);
      st16(z, my_rs, zo + tj * WPIECE + lane * 16 + part * 3072 + 1024);
      if (part < 2) {
        st16(z, my_rs, zo + tj * WPIECE + lane * 16 + part * 3072 + 2048);
        if (extra) st16(z, my_rs, zo + tx * WPIECE + ex_lane + part * 4096);
      }
    };
    LDS_AS unsigned char* slot = (LDS_AS unsigned char*)lds + (sv - 1) * SLOT + lane * 16;
    const unsigned sb = OFF_W + (call & 1) * WSTAGE + s * WSRC;
    for (int kb = 0; kb < NKB; ++kb) {
      const unsigned po = (kb_of(kb) * NJ + tj) * WPIECE;
      const unsigned so = sb + po, so_ex = sb + (kb * NJ + tx) * WPIECE;
      const bool direct = win && kb == NKB - 1;
      bool dead = false;
      for (unsigned it = 1; !direct; ++it) {
        if (rfl(ld4(my_rs, 0, so)) != 0) break;
        if ((it & 63) == 0 && expired(tmo, call, t0)) { dead = true; break; }
        __builtin_amdgcn_s_sleep(2);
      }
      u32x4 d[10];
      bool have = false;
      if (direct) {
        while (!rfl(*invdone) && !rfl(*abortf)) __builtin_amdgcn_s_sleep(2);
        if (rfl(*abortf)) return;
        const int J = ((tl - sv * (ROTW / 8)) & (ROTW - 1)) * (NR - 1) + sv - 1;
        const unsigned sc_l = sched[lane & 7];
        int c0 = 0, c1 = 0;
#pragma unroll
        for (int k = 1; k < NR; ++k) {
          const int ak = (int)__builtin_amdgcn_readlane(sc_l, k) - 1;
          c0 += max(lane - ak, 0);
          c1 += max(lane + 64 - ak, 0);
        }
        const int row = __builtin_popcountll(__builtin_amdgcn_ballot_w64(c0 <= J)) + __builtin_popcountll(__builtin_amdgcn_ballot_w64(c1 <= J)) - 1;
        int p = J - (int)(row < 64 ? __builtin_amdgcn_readlane(c0, row & 63) : __builtin_amdgcn_readlane(c1, row & 63));
        int lk = sv;
#pragma unroll
        for (int k = NR - 1; k >= 1; --k) {
          const unsigned act = (unsigned)__builtin_amdgcn_ballot_w64((int)sc_l - 1 <= row) & 0xfeu;
          if (((act >> k) & 1u) && __builtin_popcount(act & ((1u << k) - 1u)) == p) lk = k;
        }
        const unsigned fire = (unsigned)__builtin_amdgcn_readlane(sc_l, 0) + row * SLOT_T / 10;
        while ((int)(fire - (unsigned)__builtin_amdgcn_s_memrealtime()) > 0) __builtin_amdgcn_s_sleep(1);
        const auto src = mk_rsrc(ws_of(sv), OFF_CNT);
        const unsigned eo = OFF_WE + (call & 3) * WSRC + po;
        const auto via = mk_rsrc(ws_of(lk), OFF_CNT);
        unsigned m = 0xffffffffu;
#pragma unroll
        for (int v = 0; v < 8; ++v) d[v] = ld16<0>(via, lane * 16 + v * 1024, lk == sv ? eo : so);
#pragma unroll
        for (int v = 0; v < 8; ++v) m = min(m, min4(d[v]));
        for (unsigned it = 1; __builtin_amdgcn_ballot_w64(m == 0) != 0 && !dead; ++it) {
          m = 0xffffffffu;
#pragma unroll
          for (int v = 0; v < 8; ++v) d[v] = ld16<AUX_SYS>(src, lane * 16 + v * 1024, eo);
#pragma unroll
          for (int v = 0; v < 8; ++v) m = min(m, min4(d[v]));
          if ((it & 63) == 0 && expired(tmo, call, t0)) dead = true;
        }
        have = true;
      }
      for (unsigned it = 1; !dead && !have; ++it) {
        unsigned m = 0xffffffffu;
#pragma unroll
        for (int v = 0; v < 8; ++v) d[v] = ld16<AUX_SYS>(my_rs, lane * 16 + v * 1024, so);
        if (extra) {
          d[8] = ld16<AUX_SYS>(my_rs, ex_lane, so_ex);
          d[9] = ld16<AUX_SYS>(my_rs, ex_lane + 4096, so_ex);
          m = min(min4(d[8]), min4(d[9]));
        }
#pragma unroll
        for (int v = 0; v < 8; ++v) m = min(m, min4(d[v]));
        if (__builtin_amdgcn_ballot_w64(m == 0) == 0) break;
        if ((it & 63) == 0 && expired(tmo, call, t0)) dead = true;
      }
      if (dead) { if (lane == 0) *abortf = 1; return; }
      while (__builtin_amdgcn_ballot_w64(consumed[sv * 8 + (lane & 7)] < (unsigned)kb) && !rfl(*abortf))
        __builtin_amdgcn_s_sleep(2);
#pragma unroll
      for (int v = 0; v < 8; ++v) *(LDS_AS u32x4*)(slot + v * 1024) = d[v];
      if (extra) { *(LDS_AS u32x4*)(slot + 8192) = d[8]; *(LDS_AS u32x4*)(slot + 9216) = d[9]; }
      asm volatile("s_waitcnt lgkmcnt(0)" ::: "memory");
      if (lane == 0) ready[sv] = kb + 1;
      ST(4 + kb * 8 + sv);
      if (kb < NKB - 1) rezero(kb);
      if (kb == 1 && ti == 0) {
        const unsigned f = ld4(my_rs, lane * 128, sb + ((NKB - 1) * NJ + tj) * WPIECE);
        asm volatile("" :: "v"(f));
      }
    }
    return;
  }

  sleep_until(t0, OWN0);
  const int mb = ti * 8 + wave;
  const auto x_own_rs = mk_rsrc(px, M_ * KS);
  const unsigned x_own_lane = (unsigned)(rank * MS + mb * 16 + fr) * KS + fg * 16;
  const unsigned x_st_lane = OFF_X + (call & 3) * XSTAGE + mb * XFRAG + lane * 16;
  const float sc = *(const GLB_AS float*)asx * *(const GLB_AS float*)asw;
  auto ldx_own = [&](unsigned kb) {
    const u32x4 lo = ld16<0>(x_own_rs, x_own_lane, kb * 128);
    const u32x4 hi = ld16<0>(x_own_rs, x_own_lane + 64, kb * 128);
    return __builtin_shufflevector(lo, hi, 0, 1, 2, 3, 4, 5, 6, 7);
  };
  auto ldx_st = [&](int sv, unsigned kb) {
    const unsigned so = ((rank + sv) & (NR - 1)) * XSRC + kb * 32 * XFRAG;
    const u32x4 lo = ld16<AUX_SYS>(my_rs, x_st_lane, so);
    const u32x4 hi = ld16<AUX_SYS>(my_rs, x_st_lane + 1024, so);
    return __builtin_shufflevector(lo, hi, 0, 1, 2, 3, 4, 5, 6, 7);
  };
  const unsigned tr_lane = fg * 1024 + ((fg & 1) << 7) + fr * 8;

  f32x4 acc[5] = {};
  bool failed = false;
  auto step = [&](unsigned base, u32x8 xv) {
    const i32x8 bx = __builtin_bit_cast(i32x8, xv);
    unsigned tl = tr_lane;
    asm volatile("" : "+v"(tl));
    LDS_AS unsigned char* sl = (LDS_AS unsigned char*)lds + base;
    if (extra) {
      i32x8 aw;
      const unsigned te = tl - fg * 768 + 8192;
#pragma unroll
      for (int c = 0; c < 4; ++c) {
        const i32x2 r = __builtin_amdgcn_ds_read_tr8_b64_v2i32((LDS_AS i32x2*)(sl + (c >> 1) * 1024 + (te ^ ((c & 1) << 7))));
        aw[2 * c] = r[0]; aw[2 * c + 1] = r[1];
      }
      acc[4] = __builtin_amdgcn_mfma_scale_f32_16x16x128_f8f6f4(aw, bx, acc[4], 0, 0, 0, 0x7f7f7f7f, 0, 0x7f7f7f7f);
    }
#pragma unroll
    for (int q = 0; q < 4; ++q) {
      i32x8 aw;
#pragma unroll
      for (int c = 0; c < 4; ++c) {
        const i32x2 r = __builtin_amdgcn_ds_read_tr8_b64_v2i32((LDS_AS i32x2*)(sl + (c >> 1) * 4096 + q * 256 + (tl ^ ((c & 1) << 7))));
        aw[2 * c] = r[0]; aw[2 * c + 1] = r[1];
      }
      acc[q] = __builtin_amdgcn_mfma_scale_f32_16x16x128_f8f6f4(aw, bx, acc[q], 0, 0, 0, 0x7f7f7f7f, 0, 0x7f7f7f7f);
    }
  };
#pragma unroll
  for (unsigned kb = 0; kb < NKB; ++kb) {
    const u32x8 xv = ldx_own(kb);
    while (!rfl(ownrdy[kb])) __builtin_amdgcn_s_sleep(1);
    step(LDS_OWN + kb * SLOT, xv);
  }
  while (!((unsigned)__builtin_amdgcn_ballot_w64(ctl[lane & 15] != 0u) & 0x1feu)) __builtin_amdgcn_s_sleep(4);
  rezero3();
  unsigned kb = 0;
  for (unsigned stop = DRY ? NKB - 1 : NKB;; stop = NKB) {
    asm volatile("" : "+s"(stop));
    for (; kb < stop && !failed; ++kb) {
      u32x8 xf[NR];
      for (unsigned it = 1;; ++it) {
        unsigned m = 0xffffffffu;
#pragma unroll
        for (int sv = 1; sv < NR; ++sv) xf[sv] = ldx_st(sv, kb_of(kb));
#pragma unroll
        for (int sv = 1; sv < NR; ++sv) m = min(m, min8(xf[sv]));
        if (__builtin_amdgcn_ballot_w64(m == 0) == 0) break;
        if ((it & 15) == 0 && expired(tmo, call, t0)) { failed = true; break; }
        __builtin_amdgcn_s_sleep(32);
      }
      unsigned pend = failed ? 0u : 0xfeu;
      while (pend) {
        unsigned rdy;
        for (;;) {
          const unsigned rv = ctl[lane & 15];
          rdy = (unsigned)__builtin_amdgcn_ballot_w64(rv > ((lane & 15) < 8 ? kb : 0u)) & (pend | 0x100u);
          if (rdy) break;
          __builtin_amdgcn_s_sleep(1);
        }
        pend &= ~rdy;
        if (rdy & 0x100u) { failed = true; pend = 0; rdy = 0; }
#pragma unroll
        for (int sv = 1; sv < NR; ++sv)
          if (rdy & (1u << sv)) {
            step((sv - 1) * SLOT, xf[sv]);
            asm volatile("s_waitcnt lgkmcnt(0)" ::: "memory");
            consumed[sv * 8 + wave] = kb + 1;
          }
      }
    }
    if (failed && lane == 0) *abortf = 1;

    STW0(2);
    const float sce = failed ? __builtin_nanf("") : sc;
    const auto o_rs = mk_rsrc((gchar*)aout, stop == NKB ? MS * N_ * 4 : 0);
    auto silu4 = [&](f32x4 a) {
      u32x4 o;
#pragma unroll
      for (int e = 0; e < 4; ++e) {
        const float y = a[e] * sce;
        const float z = __builtin_amdgcn_fmed3f(y, -60.0f, 60.0f);
        o[e] = __builtin_bit_cast(unsigned, y * __builtin_amdgcn_rcpf(1.0f + __expf(-z)));
      }
      return o;
    };
    const unsigned o_pair = ((unsigned)(mb * 16 + (fr & 7)) * N_ + (fr >> 3) * 16 + fg * 4 + tj * 64) * 4;
#pragma unroll
    for (int q = 0; q < 4; q += 2) {
      const u32x4 o0 = silu4(acc[q]), o1 = silu4(acc[q + 1]);
      u32x4 lo, hi;
#pragma unroll
      for (int e = 0; e < 4; ++e) {
        const unsigned snd = fr < 8 ? o1[e] : o0[e];
        const unsigned rcv = __builtin_amdgcn_update_dpp(0u, snd, 0x128  , 0xf, 0xf, false);
        lo[e] = fr < 8 ? o0[e] : rcv;
        hi[e] = fr < 8 ? rcv : o1[e];
      }
      st16(lo, o_rs, o_pair + q * 64);
      st16(hi, o_rs, o_pair + q * 64 + 8 * N_ * 4);
    }
    if (extra) st16(silu4(acc[4]), o_rs, ((unsigned)(mb * 16 + fr) * N_ + fg * 4 + tx * 64 + tu * 16) * 4);
    if (stop == NKB) break;
  }
}

}

extern "C" void kernel_launch(int rank, int nranks, ncclComm_t comm,
                              hipStream_t stream,
                              void* const* d_in, const int* in_sizes, int n_in,
                              void* d_out, int out_size,
                              const int* in_dtypes, int out_dtype,
                              void* const* d_out_all,
                              void* const* d_ws_all, size_t ws_bytes) {
  if (nranks != NR || (unsigned)rank >= (unsigned)NR || ws_bytes < WS_NEED || n_in < 4) {
    fprintf(stderr, "kernel_launch: unexpected configuration\n");
    return;
  }
  Params p;
  p.x = (const uint8_t*)d_in[0];
  p.w = (const uint8_t*)d_in[1];
  p.sx = (const float*)d_in[2];
  p.sw = (const float*)d_in[3];
  p.out = (float*)d_out;
  for (int i = 0; i < NR; ++i) p.ws[i] = (char*)d_ws_all[(rank + i) & (NR - 1)];
  p.rank = rank;
  hipLaunchKernelGGL(gemm_rs_kernel, dim3(GRID), dim3(1024), 0, stream, p);
}
```

```cpp
#include <hip/hip_runtime.h>
#include <rccl/rccl.h>
#include <cstdio>
#include <cstdint>
#include <cstddef>

namespace {

constexpr int NR = 8;
constexpr int M_ = 4096, K_ = 4096, N_ = 4096;
constexpr int MS = M_ / NR;
constexpr int KS = K_ / NR;
constexpr int NKB = KS / 128;
constexpr int NJ = N_ / 64;
#define XPUSH 108
#define OWN0 200
#define NPULL_ 36
#define SLOT_T 134
#define MONN 16
#define ARRC 28
#define PRIO 1
#define ROTS 8
#define SNAP 110
#define FOFF 145
constexpr int ROTW = 16;
constexpr int GRID = 256;
constexpr int NCOMM = 32;
constexpr int NPULL = NPULL_;
constexpr int PWAVES = (NPULL * (NR - 1) + NCOMM - 1) / NCOMM;
constexpr int XU = 256 - XPUSH;
constexpr int NUNIT = XU + (NKB * NJ - ROTW) * 8;
constexpr unsigned NPAIR = NUNIT / 2;
static_assert(XU % 2 == 0 && XPUSH <= GRID && PWAVES <= 8, "");

constexpr unsigned WPIECE = 8192;
constexpr unsigned WSRC = NKB * NJ * WPIECE;
constexpr unsigned WSTAGE = NR * WSRC;
constexpr unsigned XFRAG = 2048;
constexpr unsigned XSRC = NKB * 32 * XFRAG;
constexpr unsigned XSTAGE = NR * XSRC;
constexpr unsigned OFF_W = 0;
constexpr unsigned OFF_X = OFF_W + 2 * WSTAGE;
constexpr unsigned OFF_WE = OFF_X + 4 * XSTAGE;
constexpr unsigned OFF_XE = OFF_WE + 4 * WSRC;
constexpr unsigned OFF_CNT = OFF_XE + 4 * XSTAGE;
constexpr unsigned OFF_TMO = OFF_CNT + GRID * 128;
constexpr unsigned OFF_TKT = OFF_TMO + 128;
constexpr unsigned WS_NEED = OFF_TKT + 32 * 128;

constexpr unsigned SLOT = WPIECE + 2048;
constexpr unsigned LDS_OWN = 7 * SLOT;
constexpr unsigned LDS_CTL = LDS_OWN + NKB * SLOT;

#define ST(i) do {} while (0)
#define STW0(i) do {} while (0)
constexpr unsigned long long DEADLINE_TICKS = 50000000ull;

typedef unsigned u32x4 __attribute__((ext_vector_type(4)));
typedef unsigned u32x8 __attribute__((ext_vector_type(8)));
typedef int i32x2 __attribute__((ext_vector_type(2)));
typedef int i32x8 __attribute__((ext_vector_type(8)));
typedef float f32x4 __attribute__((ext_vector_type(4)));
#define LDS_AS __attribute__((address_space(3)))
#define GLB_AS __attribute__((address_space(1)))
typedef unsigned long long u64;
typedef u64 u64x8 __attribute__((ext_vector_type(8)));
typedef u64 u64x4 __attribute__((ext_vector_type(4)));
typedef u64 u64x2 __attribute__((ext_vector_type(2)));
typedef GLB_AS char gchar;
typedef GLB_AS unsigned gu32;
constexpr int AUX_SYS = 17;

struct Params {
  const uint8_t* x;
  const uint8_t* w;
  const float* sx;
  const float* sw;
  float* out;
  char* ws[NR];
  int rank;
  int pad;
};
static_assert(sizeof(Params) == 0x70 && offsetof(Params, ws) == 0x28 && offsetof(Params, rank) == 0x68, "");

__device__ __forceinline__ __amdgpu_buffer_rsrc_t mk_rsrc(const gchar* p, unsigned bytes) {
  return __builtin_amdgcn_make_buffer_rsrc((void*)(u64)p, 0, bytes, 0x00020000);
}
__device__ __forceinline__ void st16(u32x4 v, __amdgpu_buffer_rsrc_t rs, unsigned off) {
  __builtin_amdgcn_raw_buffer_store_b128(v, rs, off, 0, AUX_SYS);
}
template <int AUX> __device__ __forceinline__ u32x4 ld16(__amdgpu_buffer_rsrc_t rs, unsigned voff, unsigned soff) {
  return __builtin_amdgcn_raw_buffer_load_b128(rs, voff, soff, AUX);
}
__device__ __forceinline__ unsigned ld4(__amdgpu_buffer_rsrc_t rs, unsigned voff, unsigned soff) {
  return __builtin_amdgcn_raw_buffer_load_b32(rs, voff, soff, AUX_SYS);
}
template <int N> __device__ __forceinline__ void wait_vm() {
  asm volatile("s_waitcnt vmcnt(%0)" :: "n"(N) : "memory");
}
__device__ __forceinline__ gchar* sel8(int k, u64 a0, u64 a1, u64 a2, u64 a3, u64 a4, u64 a5, u64 a6, u64 a7) {
  const u64 b0 = (k & 1) ? a1 : a0, b1 = (k & 1) ? a3 : a2, b2 = (k & 1) ? a5 : a4, b3 = (k & 1) ? a7 : a6;
  const u64 c0 = (k & 2) ? b1 : b0, c1 = (k & 2) ? b3 : b2;
  return (gchar*)((k & 4) ? c1 : c0);
}
__device__ __forceinline__ unsigned rfl(unsigned v) { return __builtin_amdgcn_readfirstlane(v); }
__device__ __forceinline__ u32x4 nonzero(u32x4 v) {
#pragma unroll
  for (int e = 0; e < 4; ++e) v[e] = v[e] ? v[e] : 0x80u;
  return v;
}
__device__ __forceinline__ unsigned min4(u32x4 v) { return min(min(v[0], v[1]), min(v[2], v[3])); }
__device__ __forceinline__ unsigned min8(u32x8 v) {
  return min(min(min(v[0], v[1]), min(v[2], v[3])), min(min(v[4], v[5]), min(v[6], v[7])));
}
__device__ __forceinline__ void sleep_until(unsigned long long t0, unsigned ticks) {
  while (__builtin_amdgcn_s_memrealtime() - t0 < ticks) __builtin_amdgcn_s_sleep(2);
}
__device__ __forceinline__ bool expired(gu32* tmo, unsigned call, unsigned long long t0) {
  if (rfl(__hip_atomic_load(tmo, __ATOMIC_RELAXED, __HIP_MEMORY_SCOPE_AGENT)) == call ||
      __builtin_amdgcn_s_memrealtime() - t0 > DEADLINE_TICKS) {
    __hip_atomic_store(tmo, call, __ATOMIC_RELAXED, __HIP_MEMORY_SCOPE_AGENT);
    return true;
  }
  return false;
}

__global__ void __launch_bounds__(1024, 4) gemm_rs_kernel(Params) {
  __shared__ __attribute__((aligned(16))) unsigned char lds[LDS_CTL + 384];
  u64x8 ka;
  u64x4 kb_;
  u64x2 kc;
  asm volatile("s_load_dwordx16 %0, %3, 0x0\n\ts_load_dwordx8 %1, %3, 0x40\n\ts_load_dwordx4 %2, %3, 0x60"
               : "=&s"(ka), "=&s"(kb_), "=&s"(kc) : "s"(__builtin_amdgcn_kernarg_segment_ptr()));
  const int tid = threadIdx.x;
  const int lane = tid & 63;
  const int wave = __builtin_amdgcn_readfirstlane(tid >> 6);
  const int t = blockIdx.x;
  const bool comm = t < NCOMM;
  const int er = lane >> 3, ec = lane & 7;
  const int e = t - NCOMM;
  const bool isx = wave < 8;
  if (isx) __builtin_amdgcn_s_setprio(2);
  const int ek = 1 + (unsigned)(e + NCOMM + 3 + wave) % (NR - 1), ex = wave ? e : GRID - NCOMM + e / (NR - 1);
  auto w_off = [&](int ew) -> unsigned {
    return ((ew >> 6) * 128 + ((ew & 1) * 4 + ((wave >> 1) & 3)) * 16 + (wave & 1) * 8 + er) * N_ + (ew & 62) * 64 + ec * 16;
  };
  auto w_img = [&](int ew) -> unsigned {
    return OFF_WE + (ew & ~1) * WPIECE + ((ew & 1) * 4 + ((wave >> 1) & 3)) * 1024 + ((wave ^ (wave >> 1)) & 1) * 128;
  };
  unsigned e_off = isx ? (ek * MS + (ex & 63) * 8 + er) * KS + (ex >> 6) * 128 + ec * 16 : w_off(e);
  unsigned c_off = OFF_CNT + t * 128;
  asm volatile("" : "+v"(e_off), "+v"(c_off));
  asm volatile("s_waitcnt lgkmcnt(0)" : "+s"(ka), "+s"(kb_), "+s"(kc));
  const u64 ax = ka[0], aw_ = ka[1], asx = ka[2], asw = ka[3], aout = ka[4];
  const u64 w0 = ka[5], w1 = ka[6], w2 = ka[7], w3 = kb_[0], w4 = kb_[1], w5 = kb_[2], w6 = kb_[3], w7 = kc[0];
  const int rank = (int)kc[1];
  u32x4 ev;
  if (!comm) ev = *(const GLB_AS u32x4*)((const gchar*)(isx ? ax : aw_) + ((e_off + (isx ? rank * (MS * KS) : 0)) & (M_ * KS - 1)));
  const unsigned cnt_raw = __hip_atomic_load((gu32*)((gchar*)w0 + c_off), __ATOMIC_RELAXED, __HIP_MEMORY_SCOPE_AGENT);
#define ws_of(k) sel8((k), w0, w1, w2, w3, w4, w5, w6, w7)
  const gchar* const px = (const gchar*)ax;
  const gchar* const pw = (const gchar*)aw_;
  gchar* const myws = (gchar*)w0;
  gu32* const cnt = (gu32*)(myws + OFF_CNT + t * 128);
  gu32* const tmo = (gu32*)(myws + OFF_TMO);
  const int fr = lane & 15, fg = lane >> 4;
  const int xs = (rank + wave) & (NR - 1);
  const bool push = isx && ex < XPUSH;
  const unsigned long long t0 = __builtin_amdgcn_s_memrealtime();
  const auto my_rs = mk_rsrc(myws, OFF_CNT);
  const unsigned e_lane = (ec >> 2) * (isx ? 1024 : WPIECE) + (ec & 3) * 256 + er * 16;
  unsigned e_img = e_lane + (isx ? (ex >> 1) * XFRAG + (ex & 1) * 128 + (push ? OFF_X + rank * XSRC : OFF_XE + ((rank + ek) & (NR - 1)) * XSRC)
                                 : w_img(e));
  u64 e_base = (u64)ws_of(push ? ek : 0);
  asm volatile("" : "+s"(e_base));
  asm volatile("" : "+v"(e_img));
  const unsigned call = rfl(cnt_raw) + 1;
  static_assert(XSTAGE == WSRC, "");
  if (!comm) {
    st16(nonzero(ev), mk_rsrc((gchar*)e_base, OFF_CNT), e_img + (call & 3) * XSTAGE);
    if (!isx && e < NCOMM) {
      const u32x4 ev2 = *(const GLB_AS u32x4*)(pw + w_off(GRID - NCOMM + e));
      st16(nonzero(ev2), my_rs, e_lane + w_img(GRID - NCOMM + e) + (call & 3) * XSTAGE);
    }
  }
  __builtin_amdgcn_s_setprio(0);

  const int xcd = t & 7, idx = (t - NCOMM) >> 3;
  const int ti = idx & 3, tu = idx >> 2;
  const int tl = 8 + tu * 8 + xcd;
  const int joff = ((rank & 3) + 1) * ROTW;
  const int tj = (tl + joff) & (NJ - 1);
  const int tx = (xcd + joff) & (NJ - 1);
  const bool extra = tu < 4;
  const bool win = tl >= NJ - ROTW;
  const int kbs = rank >> 2;
  auto kb_of = [&](int r) { return !win ? r : r == NKB - 1 ? kbs : r + (r >= kbs); };
  volatile LDS_AS unsigned* ctl = (volatile LDS_AS unsigned*)(lds + LDS_CTL);
  volatile LDS_AS unsigned* ready = ctl;
  volatile LDS_AS unsigned* abortf = ctl + 8;
  volatile LDS_AS unsigned* invdone = ctl + 9;
  volatile LDS_AS unsigned* ownrdy = ctl + 12;
  volatile LDS_AS unsigned* consumed = ctl + 16;
  volatile LDS_AS unsigned* sched = ctl + 80;

  if (wave == 8) {
    if (lane < 16) ctl[lane] = 0;
    consumed[lane] = 0;
    asm volatile("s_waitcnt lgkmcnt(0)" ::: "memory");
  }
  asm volatile("" :: "s"(call));
  __builtin_amdgcn_s_barrier();
  if (tid == 0) __hip_atomic_store(cnt, call, __ATOMIC_RELAXED, __HIP_MEMORY_SCOPE_AGENT);
  auto rezero3 = [&]() {
    const u32x4 z = {0u, 0u, 0u, 0u};
    st16(z, my_rs, OFF_X + ((call + 2) & 3) * XSTAGE + xs * XSRC + t * 1024 + lane * 16);
    st16(z, my_rs, OFF_XE + ((call + 2) & 3) * XSTAGE + xs * XSRC + t * 1024 + lane * 16);
    st16(z, my_rs, OFF_WE + ((call + 2) & 3) * WSRC + t * WPIECE + tid * 16);
  };

  if (comm) {
    if (wave < 8) {
      if (wave < 2) {
        const int mk = 1 + min(t * PWAVES + wave * (PWAVES - 1), NPULL * (NR - 1) - 1) / NPULL;
        const unsigned b0 = OFF_X + (call & 3) * XSTAGE + ((rank + mk) & (NR - 1)) * XSRC;
        const unsigned se = (((lane & (MONN - 1)) * 64 / MONN + t % (64 / MONN)) * XPUSH / 64);
        for (unsigned n = 1;; ++n) {
          const unsigned f = ld4(my_rs, b0 + (se >> 1) * XFRAG + (se & 1) * 128, 0);
          if (__builtin_popcountll(__builtin_amdgcn_ballot_w64(f != 0)) >= ARRC) break;
          if ((n & 63) == 0 && expired(tmo, call, t0)) break;
        }
        if (lane == 0) ctl[mk] = 1;
      }
      __builtin_amdgcn_s_barrier();
      rezero3();
      if (t == 0 && wave == 0 && lane < 2 * NR)
        __hip_atomic_store((gu32*)(myws + OFF_TKT + ((call + 2) & 3) * 8 * 128 + (lane < NR ? lane * 128 : (lane - NR) * 4)),
                           0u, __ATOMIC_RELAXED, __HIP_MEMORY_SCOPE_AGENT);
      return;
    }
    const int pwv = wave - 8;
    const int g = t * PWAVES + pwv;
    if (pwv >= PWAVES || g >= NPULL * (NR - 1)) return;
    const int k = 1 + g / NPULL, q = g % NPULL;
    const int s = (rank + k) & (NR - 1);
    const u64 sp = (u64)ws_of(k);
    const u32x4 srd_s = {rfl((unsigned)sp), rfl((unsigned)(sp >> 32) & 0xffffu), OFF_CNT, 0x00020000u};
    const u32x4 srd_d = {rfl((unsigned)w0), rfl((unsigned)(w0 >> 32) & 0xffffu), OFF_CNT, 0x00020000u};
    constexpr unsigned INV = 1u << 20;
    u64 btab = 0;
#pragma unroll
    for (int b = 0; b < 15; ++b) {
      const int r = b >> 2, lg = b & 3;
      btab |= (u64)((lg == 3 ? r + (r >= kbs) : r) * 4 + ((lg + rank + 1) & 3)) << (4 * b);
    }
    unsigned status = 0;
    const unsigned sx0 = OFF_XE + (call & 3) * XSTAGE + rank * XSRC + XPUSH * 1024, sw0 = OFF_WE + (call & 3) * WSRC;
    const unsigned dx0 = OFF_X + (call & 3) * XSTAGE + s * XSRC + XPUSH * 1024, dw0 = OFF_W + (call & 1) * WSTAGE + s * WSRC;
    const unsigned tkt = OFF_TKT + ((call & 3) * 8 + k) * 128;
    while (!rfl(ctl[k])) __builtin_amdgcn_s_sleep(1);
    {
      unsigned cur, nxt, s1, s2;
      u64 tnow;
#define VERIFY(N, X0, X1, X2, X3, L) \
      "s_cmp_lt_u32 %[cur], %[npair]\n\t" \
      "s_cbranch_scc0 .Lok" L "_%=\n\t" \
      "s_waitcnt vmcnt(" N ")\n\t" \
      ".Lchk" L "_%=:\n\t" \
      "v_min_u32 v121, " X2 ", " X3 "\n\t" \
      "v_min3_u32 v121, " X0 ", " X1 ", v121\n\t" \
      "v_cmp_eq_u32 vcc, 0, v121\n\t" \
      "s_nop 0\n\t" \
      "s_cbranch_vccnz .Lretry" L "_%=\n\t" \
      ".Lok" L "_%=:\n\t"
#define RETRY(X, OFS, L) \
      ".Lretry" L "_%=:\n\t" \
      "s_sleep 8\n\t" \
      "s_memrealtime %[tnow]\n\t" \
      "s_waitcnt lgkmcnt(0)\n\t" \
      "v_mov_b64 v[122:123], %[tnow]\n\t" \
      "v_cmp_lt_u64 vcc, %[tend], v[122:123]\n\t" \
      "s_nop 0\n\t" \
      "s_cbranch_vccnz .Lfail_%=\n\t" \
      "buffer_load_dwordx4 " X ", v119, %[srs], 0 offen " OFS " sc0 sc1\n\t" \
      "s_waitcnt vmcnt(0)\n\t" \
      "s_branch .Lchk" L "_%=\n\t"
#define HALF(XA, XA0, XA1, XA2, XA3, XB, XB0, XB1, XB2, XB3, YA, YB, TIN, TOUT, LA, LB) \
      VERIFY("3", XA0, XA1, XA2, XA3, LA) \
      "v_readfirstlane_b32 %[nxt], " TIN "\n\t" \
      "s_mov_b64 exec, 1\n\t" \
      "global_atomic_add " TOUT ", %[tkt], %[inc], %[ws] sc0\n\t" \
      "s_mov_b64 exec, -1\n\t" \
      "s_add_u32 %[nxt], %[nxt], %[base]\n\t" \
      "s_sub_u32 %[s1], %[nxt], %[xpair]\n\t" \
      "s_add_u32 %[s2], %[s1], %[rot]\n\t" \
      "s_and_b32 %[s2], %[s2], %[rmask]\n\t" \
      "s_and_b32 vcc_lo, %[s1], %[rkeep]\n\t" \
      "s_or_b32 %[s2], %[s2], vcc_lo\n\t" \
      "s_lshr_b32 %[s1], %[s1], 6\n\t" \
      "s_lshl_b32 %[s1], %[s1], 2\n\t" \
      "s_lshr_b64 vcc, %[btab], %[s1]\n\t" \
      "s_and_b32 %[s1], vcc_lo, 15\n\t" \
      "s_lshl_b32 %[s1], %[s1], 6\n\t" \
      "s_or_b32 %[s2], %[s2], %[s1]\n\t" \
      "s_cmp_lt_u32 %[nxt], %[xpair]\n\t" \
      "s_cselect_b32 %[s2], %[nxt], %[s2]\n\t" \
      "s_cselect_b32 %[s1], %[sx], %[sw]\n\t" \
      "s_lshl_b32 %[s2], %[s2], 11\n\t" \
      "s_add_u32 %[s1], %[s1], %[s2]\n\t" \
      "s_cmp_lt_u32 %[nxt], %[npair]\n\t" \
      "s_cselect_b32 %[s1], %[s1], 0x80000000\n\t" \
      "v_add_u32 v118, %[s1], %[lo]\n\t" \
      "buffer_load_dwordx4 " YA ", v118, %[srs], 0 offen sc0 sc1\n\t" \
      "buffer_store_dwordx4 " XA ", v120, %[srd], 0 offen sc0 sc1\n\t" \
      VERIFY("4", XB0, XB1, XB2, XB3, LB) \
      "buffer_load_dwordx4 " YB ", v118, %[srs], 0 offen offset:1024 sc0 sc1\n\t" \
      "buffer_store_dwordx4 " XB ", v120, %[srd], 0 offen offset:1024 sc0 sc1\n\t" \
      "s_cmp_lt_u32 %[nxt], %[xpair]\n\t" \
      "s_cselect_b32 %[s1], %[dx], %[dw]\n\t" \
      "s_add_u32 %[s1], %[s1], %[s2]\n\t" \
      "s_cmp_lt_u32 %[nxt], %[npair]\n\t" \
      "s_cselect_b32 %[s1], %[s1], 0x80000000\n\t" \
      "v_mov_b32 v119, v118\n\t" \
      "v_add_u32 v120, %[s1], %[lo]\n\t" \
      PSTAMP \
      "s_mov_b32 %[cur], %[nxt]\n\t"
#define PSTAMP
#define PSTAMP_OPS
      asm volatile(
          "s_mov_b32 %[cur], %[inv]\n\t"
          "v_mov_b32 v117, %[first]\n\t"
          "v_mov_b32 v120, 0x80000000\n\t"
          "s_nop 4\n\t"
          ".Lloop_%=:\n\t"
          HALF("v[100:103]", "v100", "v101", "v102", "v103", "v[104:107]", "v104", "v105", "v106", "v107",
               "v[108:111]", "v[112:115]", "v117", "v116", "a", "b")
          "s_cmp_ge_u32 %[cur], %[npair]\n\t"
          "s_cbranch_scc1 .Ldone_%=\n\t"
          HALF("v[108:111]", "v108", "v109", "v110", "v111", "v[112:115]", "v112", "v113", "v114", "v115",
               "v[100:103]", "v[104:107]", "v116", "v117", "c", "d")
          "s_cmp_lt_u32 %[cur], %[npair]\n\t"
          "s_cbranch_scc1 .Lloop_%=\n\t"
          "s_branch .Ldone_%=\n\t"
          RETRY("v[100:103]", "", "a")
          RETRY("v[104:107]", "offset:1024", "b")
          RETRY("v[108:111]", "", "c")
          RETRY("v[112:115]", "offset:1024", "d")
          ".Lfail_%=:\n\t"
          "s_mov_b32 %[status], 1\n\t"
          ".Ldone_%=:\n\t"
          "s_waitcnt vmcnt(0)"
          : [cur] "=&s"(cur), [nxt] "=&s"(nxt), [s1] "=&s"(s1), [s2] "=&s"(s2), [tnow] "=&s"(tnow), [status] "+s"(status) PSTAMP_OPS
          : [srs] "s"(srd_s), [srd] "s"(srd_d), [ws] "s"(w0), [tend] "s"(t0 + DEADLINE_TICKS),
            [sx] "s"(rfl(sx0)), [sw] "s"(rfl(sw0)), [dx] "s"(rfl(dx0)), [dw] "s"(rfl(dw0)),
            [base] "n"(NPULL), [first] "s"(rfl((unsigned)(q - NPULL))),
            [btab] "s"(btab), [rot] "s"(rfl(k * (ROTS / 8) * 4)), [rmask] "n"(ROTS * 4 - 1), [rkeep] "n"(64 - ROTS * 4), [npair] "n"(NPAIR), [xpair] "n"(XU / 2), [inv] "n"(INV),
            [tkt] "v"(tkt), [inc] "v"(1u), [lo] "v"(lane * 16)
          : "memory", "vcc", "scc", "v100", "v101", "v102", "v103", "v104", "v105", "v106", "v107", "v108", "v109",
            "v110", "v111", "v112", "v113", "v114", "v115", "v116", "v117", "v118", "v119", "v120", "v121", "v122", "v123");
    }
    if (status) __hip_atomic_store(tmo, call, __ATOMIC_RELAXED, __HIP_MEMORY_SCOPE_AGENT);
    if (pwv < 8) ST(48 + pwv);
    return;
  }

  if (wave == 8) {
    sleep_until(t0, OWN0);
    const gchar* w_even = pw + (size_t)fr * N_ + fg * 16 + tj * 64;
    const gchar* w_odd = pw + (size_t)(fr ^ 8) * N_ + fg * 16 + tj * 64;
    const gchar* w_ex = pw + (size_t)(fg * 16 + (fr ^ ((fg & 1) << 3))) * N_ + tx * 64 + tu * 16;
    for (int kb = 0; kb < NKB; ++kb) {
#pragma unroll
      for (int v = 0; v < 8; ++v)
        __builtin_amdgcn_global_load_lds((const GLB_AS void*)(((v & 1) ? w_odd : w_even) + (size_t)(kb * 128 + v * 16) * N_),
                                         (LDS_AS void*)(lds + LDS_OWN + kb * SLOT + v * 1024), 16, 0, 0);
      if (extra) {
        __builtin_amdgcn_global_load_lds((const GLB_AS void*)(w_ex + (size_t)(kb * 128) * N_),
                                         (LDS_AS void*)(lds + LDS_OWN + kb * SLOT + 8192), 16, 0, 0);
        __builtin_amdgcn_global_load_lds((const GLB_AS void*)(w_ex + (size_t)(kb * 128 + 64) * N_),
                                         (LDS_AS void*)(lds + LDS_OWN + kb * SLOT + 9216), 16, 0, 0);
      }
      wait_vm<0>();
      if (lane == 0) ownrdy[kb] = 1;
    }
    gu32* const tk = (gu32*)(myws + OFF_TKT + (call & 3) * 8 * 128);
    if (win) {
      sleep_until(t0, 1000 + (t & 63) * 20);
      __builtin_amdgcn_fence(__ATOMIC_ACQUIRE, "");
      wait_vm<0>();
      for (unsigned it = 1;; ++it) {
        const unsigned f = __hip_atomic_load(tk + (lane & 7), __ATOMIC_RELAXED, __HIP_MEMORY_SCOPE_AGENT);
        if (__builtin_amdgcn_ballot_w64(f == 0) == 0) {
          sched[lane & 7] = f;
          asm volatile("s_waitcnt lgkmcnt(0)" ::: "memory");
          if (lane == 0) *invdone = 1;
          break;
        }
        if ((it & 63) == 0 && expired(tmo, call, t0)) { if (lane == 0) *abortf = 1; break; }
        __builtin_amdgcn_s_sleep(8);
      }
    } else if (t == NCOMM + 4 * 32) {
      for (unsigned it = 1;; ++it) {
        const unsigned ta = (unsigned)__builtin_amdgcn_s_memrealtime();
        const unsigned f = __hip_atomic_load(tk + (1 + lane % 7) * 32, __ATOMIC_RELAXED, __HIP_MEMORY_SCOPE_AGENT);
        unsigned mx = 0;
#pragma unroll
        for (int k = 0; k < NR - 1; ++k) mx = max(mx, (unsigned)__builtin_amdgcn_readlane(f, k));
        const unsigned tb = (unsigned)__builtin_amdgcn_s_memrealtime();
        if (mx + SNAP >= NPAIR - NPULL && (tb - ta < 80 || mx + SNAP / 2 >= NPAIR - NPULL)) {
          const int left = max((int)(NPAIR - NPULL) - (int)mx, 0);
          const unsigned fmin = ta + 25 + left * SLOT_T / 40 + FOFF;
          const unsigned lag = __builtin_amdgcn_update_dpp(0u, f ? min((mx - f + 1) / 2, 90u) + 1 : 1000u, 0x111  , 0xf, 0xf, false);
          if (lane < NR) __hip_atomic_store(tk + lane, lane ? lag : fmin | 1u, __ATOMIC_RELAXED, __HIP_MEMORY_SCOPE_AGENT);
          break;
        }
        if ((it & 63) == 0 && expired(tmo, call, t0)) break;
        __builtin_amdgcn_s_sleep(8);
      }
    }
    return;
  }

  if (wave > 8) {
    const int sv = wave - 8;
    const int s = (rank + sv) & (NR - 1);
    const unsigned ex_lane = fg * 1024 + tu * 256 + fr * 16;
    sleep_until(t0, OWN0);
    const unsigned zo = OFF_W + ((call + 1) & 1) * WSTAGE + s * WSRC + ti * NJ * WPIECE;
    auto rezero = [&](int part) {
      const u32x4 z = {0u, 0u, 0u, 0u};
      st16(z, my_rs, zo + tj * WPIECE + lane * 16 + part * 3072);
      st16(z, my_rs, zo + tj * WPIECE + lane * 16 + part * 3072 + 1024);
      if (part < 2) {
        st16(z, my_rs, zo + tj * WPIECE + lane * 16 + part * 3072 + 2048);
        if (extra) st16(z, my_rs, zo + tx * WPIECE + ex_lane + part * 4096);
      }
    };
    LDS_AS unsigned char* slot = (LDS_AS unsigned char*)lds + (sv - 1) * SLOT + lane * 16;
    const unsigned sb = OFF_W + (call & 1) * WSTAGE + s * WSRC;
    auto slot_free = [&](unsigned kb) {
      while (__builtin_amdgcn_ballot_w64(consumed[sv * 8 + (lane & 7)] < kb) && !rfl(*abortf)) __builtin_amdgcn_s_sleep(2);
    };
    for (int kb = 0; kb < (win ? NKB - 1 : NKB); ++kb) {
      const unsigned so = sb + (kb_of(kb) * NJ + tj) * WPIECE, so_ex = sb + (kb * NJ + tx) * WPIECE;
      bool dead = false;
      for (unsigned it = 1;; ++it) {
        if (rfl(ld4(my_rs, 0, so)) != 0) break;
        if ((it & 63) == 0 && expired(tmo, call, t0)) { dead = true; break; }
        __builtin_amdgcn_s_sleep(2);
      }
      u32x4 d[10];
      for (unsigned it = 1; !dead; ++it) {
        unsigned m = 0xffffffffu;
#pragma unroll
        for (int v = 0; v < 8; ++v) d[v] = ld16<AUX_SYS>(my_rs, lane * 16 + v * 1024, so);
        if (extra) {
          d[8] = ld16<AUX_SYS>(my_rs, ex_lane, so_ex);
          d[9] = ld16<AUX_SYS>(my_rs, ex_lane + 4096, so_ex);
          m = min(min4(d[8]), min4(d[9]));
        }
#pragma unroll
        for (int v = 0; v < 8; ++v) m = min(m, min4(d[v]));
        if (__builtin_amdgcn_ballot_w64(m == 0) == 0) break;
        if ((it & 63) == 0 && expired(tmo, call, t0)) dead = true;
      }
      if (dead) { if (lane == 0) *abortf = 1; return; }
      slot_free(kb);
#pragma unroll
      for (int v = 0; v < 8; ++v) *(LDS_AS u32x4*)(slot + v * 1024) = d[v];
      if (extra) { *(LDS_AS u32x4*)(slot + 8192) = d[8]; *(LDS_AS u32x4*)(slot + 9216) = d[9]; }
      asm volatile("s_waitcnt lgkmcnt(0)" ::: "memory");
      if (lane == 0) ready[sv] = kb + 1;
      ST(4 + kb * 8 + sv);
      if (kb < NKB - 1) rezero(kb);
      if (kb == 1 && ti == 0) {
        const unsigned f = ld4(my_rs, lane * 128, sb + ((NKB - 1) * NJ + tj) * WPIECE);
        asm volatile("" :: "v"(f));
      }
    }
    if (!win) return;
    while (!rfl(*invdone) && !rfl(*abortf)) __builtin_amdgcn_s_sleep(2);
    if (rfl(*abortf)) return;
    const unsigned po = (kbs * NJ + tj) * WPIECE, eo = OFF_WE + (call & 3) * WSRC + po;
    const unsigned sc_l = sched[lane & 7];
    int c0 = 0, c1 = 0;
#pragma unroll
    for (int k = 1; k < NR; ++k) {
      const int ak = (int)__builtin_amdgcn_readlane(sc_l, k) - 1;
      c0 += max(lane - ak, 0);
      c1 += max(lane + 64 - ak, 0);
    }
    unsigned fire[2], voff[2];
    u64 via[2];
#pragma unroll
    for (int h = 0; h < 2; ++h) {
      const int J = (h * ROTW + ((tl - sv * (ROTW / 8)) & (ROTW - 1))) * (NR - 1) + sv - 1;
      const int row = __builtin_popcountll(__builtin_amdgcn_ballot_w64(c0 <= J)) + __builtin_popcountll(__builtin_amdgcn_ballot_w64(c1 <= J)) - 1;
      const int p = J - (int)(row < 64 ? __builtin_amdgcn_readlane(c0, row & 63) : __builtin_amdgcn_readlane(c1, row & 63));
      const unsigned act = (unsigned)__builtin_amdgcn_ballot_w64((int)sc_l - 1 <= row) & 0xfeu;
      int lk = sv;
#pragma unroll
      for (int k = NR - 1; k >= 1; --k)
        if (((act >> k) & 1u) && __builtin_popcount(act & ((1u << k) - 1u)) == p) lk = k;
      fire[h] = (unsigned)__builtin_amdgcn_readlane(sc_l, 0) + row * SLOT_T / 20;
      via[h] = (u64)ws_of(lk);
      voff[h] = (lk == sv ? eo : sb + po) + h * 512;
      asm volatile("" : "+s"(fire[h]), "+s"(via[h]), "+s"(voff[h]));
    }
    slot_free(NKB - 1);
    const unsigned hl = (lane >> 5) * 1024 + (lane & 31) * 16;
    LDS_AS unsigned char* hslot = (LDS_AS unsigned char*)lds + (sv - 1) * SLOT + hl;
#pragma nounroll
    for (unsigned h = 0; h < 2; ++h) {
      asm volatile("" : "+s"(h));
      const unsigned fi = h ? fire[1] : fire[0], vo = h ? voff[1] : voff[0];
      const auto vrs = mk_rsrc((gchar*)(h ? via[1] : via[0]), OFF_CNT);
      while ((int)(fi - (unsigned)__builtin_amdgcn_s_memrealtime()) > 0) __builtin_amdgcn_s_sleep(1);
      u32x4 d[4];
#pragma unroll
      for (int v = 0; v < 4; ++v) d[v] = ld16<0>(vrs, hl + v * 2048, vo);
      unsigned m = 0xffffffffu;
#pragma unroll
      for (int v = 0; v < 4; ++v) {
        *(LDS_AS u32x4*)(hslot + h * 512 + v * 2048) = d[v];
        m = min(m, min4(d[v]));
        __builtin_amdgcn_sched_barrier(0);
      }
      for (unsigned it = 1; __builtin_expect(__builtin_amdgcn_ballot_w64(m == 0) != 0, 0); ++it) {
        if ((it & 63) == 0 && expired(tmo, call, t0)) { if (lane == 0) *abortf = 1; return; }
        m = 0xffffffffu;
#pragma unroll
        for (int v = 0; v < 4; ++v) {
          const u32x4 r = ld16<AUX_SYS>(mk_rsrc(ws_of(sv), OFF_CNT), hl + v * 2048, eo + h * 512);
          *(LDS_AS u32x4*)(hslot + h * 512 + v * 2048) = r;
          m = min(m, min4(r));
        }
      }
      asm volatile("s_waitcnt lgkmcnt(0)" ::: "memory");
      if (lane == 0) ready[sv] = NKB + h;
    }
    ST(4 + 24 + sv);
    return;
  }

  sleep_until(t0, OWN0);
  const int mb = ti * 8 + wave;
  const auto x_own_rs = mk_rsrc(px, M_ * KS);
  const unsigned x_own_lane = (unsigned)(rank * MS + mb * 16 + fr) * KS + fg * 16;
  const unsigned x_st_lane = OFF_X + (call & 3) * XSTAGE + mb * XFRAG + lane * 16;
  float sc = __builtin_bit_cast(float, rfl(__builtin_bit_cast(unsigned, *(const GLB_AS float*)asx * *(const GLB_AS float*)asw)));
  asm volatile("" : "+s"(sc));
  auto ldx_own = [&](unsigned kb) {
    const u32x4 lo = ld16<0>(x_own_rs, x_own_lane, kb * 128);
    const u32x4 hi = ld16<0>(x_own_rs, x_own_lane + 64, kb * 128);
    return __builtin_shufflevector(lo, hi, 0, 1, 2, 3, 4, 5, 6, 7);
  };
  auto ldx_st = [&](int sv, unsigned kb) {
    const unsigned so = ((rank + sv) & (NR - 1)) * XSRC + kb * 32 * XFRAG;
    const u32x4 lo = ld16<AUX_SYS>(my_rs, x_st_lane, so);
    const u32x4 hi = ld16<AUX_SYS>(my_rs, x_st_lane + 1024, so);
    return __builtin_shufflevector(lo, hi, 0, 1, 2, 3, 4, 5, 6, 7);
  };
  const unsigned tr_lane = fg * 1024 + ((fg & 1) << 7) + fr * 8;

  f32x4 acc[5] = {};
  bool failed = false;
  auto step = [&](unsigned base, u32x8 xv) {
    const i32x8 bx = __builtin_bit_cast(i32x8, xv);
    unsigned tl = tr_lane;
    asm volatile("" : "+v"(tl));
    LDS_AS unsigned char* sl = (LDS_AS unsigned char*)lds + base;
    if (extra) {
      i32x8 aw;
      const unsigned te = tl - fg * 768 + 8192;
#pragma unroll
      for (int c = 0; c < 4; ++c) {
        const i32x2 r = __builtin_amdgcn_ds_read_tr8_b64_v2i32((LDS_AS i32x2*)(sl + (c >> 1) * 1024 + (te ^ ((c & 1) << 7))));
        aw[2 * c] = r[0]; aw[2 * c + 1] = r[1];
      }
      acc[4] = __builtin_amdgcn_mfma_scale_f32_16x16x128_f8f6f4(aw, bx, acc[4], 0, 0, 0, 0x7f7f7f7f, 0, 0x7f7f7f7f);
    }
#pragma unroll
    for (int q = 0; q < 4; ++q) {
      i32x8 aw;
#pragma unroll
      for (int c = 0; c < 4; ++c) {
        const i32x2 r = __builtin_amdgcn_ds_read_tr8_b64_v2i32((LDS_AS i32x2*)(sl + (c >> 1) * 4096 + q * 256 + (tl ^ ((c & 1) << 7))));
        aw[2 * c] = r[0]; aw[2 * c + 1] = r[1];
      }
      acc[q] = __builtin_amdgcn_mfma_scale_f32_16x16x128_f8f6f4(aw, bx, acc[q], 0, 0, 0, 0x7f7f7f7f, 0, 0x7f7f7f7f);
    }
  };
#pragma unroll
  for (unsigned kb = 0; kb < NKB; ++kb) {
    const u32x8 xv = ldx_own(kb);
    while (!rfl(ownrdy[kb])) __builtin_amdgcn_s_sleep(1);
    step(LDS_OWN + kb * SLOT, xv);
  }
  while (!((unsigned)__builtin_amdgcn_ballot_w64(ctl[lane & 15] != 0u) & 0x1feu)) __builtin_amdgcn_s_sleep(4);
  rezero3();
  u32x8 xf[NR];
  auto load_x = [&](unsigned kb) {
    for (unsigned it = 1;; ++it) {
      unsigned m = 0xffffffffu;
#pragma unroll
      for (int sv = 1; sv < NR; ++sv) xf[sv] = ldx_st(sv, kb);
#pragma unroll
      for (int sv = 1; sv < NR; ++sv) m = min(m, min8(xf[sv]));
      if (__builtin_amdgcn_ballot_w64(m == 0) == 0) break;
      if ((it & 15) == 0 && expired(tmo, call, t0)) { failed = true; break; }
      __builtin_amdgcn_s_sleep(32);
    }
  };
  auto next_ready = [&](unsigned& pend, unsigned level) {
    unsigned rdy;
    for (;;) {
      const unsigned rv = ctl[lane & 15];
      rdy = (unsigned)__builtin_amdgcn_ballot_w64(rv > ((lane & 15) < 8 ? level : 0u)) & (pend | 0x100u);
      if (rdy) break;
      __builtin_amdgcn_s_sleep(1);
    }
    pend &= ~rdy;
    if (rdy & 0x100u) { failed = true; pend = 0; rdy = 0; }
    return rdy;
  };
  for (unsigned kb = 0; kb < (win ? NKB - 1 : NKB) && !failed; ++kb) {
    load_x(kb_of(kb));
    for (unsigned pend = failed ? 0u : 0xfeu; pend;) {
      const unsigned rdy = next_ready(pend, kb);
#pragma unroll
      for (int sv = 1; sv < NR; ++sv)
        if (rdy & (1u << sv)) {
          step((sv - 1) * SLOT, xf[sv]);
          asm volatile("s_waitcnt lgkmcnt(0)" ::: "memory");
          consumed[sv * 8 + wave] = kb + 1;
        }
    }
  }
  if (win && !failed) load_x(kbs);
  const auto o_rs = mk_rsrc((gchar*)aout, MS * N_ * 4);
  constexpr float LOG2E = 1.4426950408889634f;
  auto silu4 = [&](f32x4 a) {
    const f32x4 y = a * (failed ? __builtin_nanf("") : sc);
    f32x4 z = y * -LOG2E;
#pragma unroll
    for (int e = 0; e < 4; ++e) z[e] = __builtin_amdgcn_exp2f(__builtin_amdgcn_fmed3f(z[e], -60.0f * LOG2E, 60.0f * LOG2E));
    z += 1.0f;
#pragma unroll
    for (int e = 0; e < 4; ++e) z[e] = __builtin_amdgcn_rcpf(z[e]);
    return __builtin_bit_cast(u32x4, y * z);
  };
  if (extra) st16(silu4(acc[4]), o_rs, ((unsigned)(mb * 16 + fr) * N_ + fg * 4 + tx * 64 + tu * 16) * 4);
#pragma nounroll
  for (unsigned h = 0; h < 2; ++h) {
    asm volatile("" : "+s"(h));
    for (unsigned pend = win && !failed ? 0xfeu : 0u; pend;) {
      const unsigned rdy = next_ready(pend, NKB - 1 + h);
#pragma unroll
      for (int sv = 1; sv < NR; ++sv)
        if (rdy & (1u << sv)) {
          const i32x8 bx = __builtin_bit_cast(i32x8, xf[sv]);
          unsigned tl2 = tr_lane + h * 512;
          asm volatile("" : "+v"(tl2));
          LDS_AS unsigned char* sl = (LDS_AS unsigned char*)lds + (sv - 1) * SLOT;
#pragma unroll
          for (int q = 0; q < 2; ++q) {
            i32x8 aw;
#pragma unroll
            for (int c = 0; c < 4; ++c) {
              const i32x2 r = __builtin_amdgcn_ds_read_tr8_b64_v2i32((LDS_AS i32x2*)(sl + (c >> 1) * 4096 + q * 256 + (tl2 ^ ((c & 1) << 7))));
              aw[2 * c] = r[0]; aw[2 * c + 1] = r[1];
            }
            acc[q] = __builtin_amdgcn_mfma_scale_f32_16x16x128_f8f6f4(aw, bx, acc[q], 0, 0, 0, 0x7f7f7f7f, 0, 0x7f7f7f7f);
          }
        }
    }
    if (failed && lane == 0) *abortf = 1;
    STW0(2);
    const unsigned o_pair = ((unsigned)(mb * 16 + (fr & 7)) * N_ + (fr >> 3) * 16 + fg * 4 + tj * 64 + h * 32) * 4;
    const u32x4 o0 = silu4(acc[0]), o1 = silu4(acc[1]);
    u32x4 lo, hi;
#pragma unroll
    for (int e = 0; e < 4; ++e) {
      lo[e] = __builtin_amdgcn_update_dpp(o0[e], o1[e], 0x128  , 0xf, 0xc, false);
      hi[e] = __builtin_amdgcn_update_dpp(o1[e], o0[e], 0x128  , 0xf, 0x3, false);
    }
    st16(lo, o_rs, o_pair);
    st16(hi, o_rs, o_pair + 8 * N_ * 4);
    acc[0] = acc[2];
    acc[1] = acc[3];
  }
}

}

extern "C" void kernel_launch(int rank, int nranks, ncclComm_t comm,
                              hipStream_t stream,
                              void* const* d_in, const int* in_sizes, int n_in,
                              void* d_out, int out_size,
                              const int* in_dtypes, int out_dtype,
                              void* const* d_out_all,
                              void* const* d_ws_all, size_t ws_bytes) {
  if (nranks != NR || (unsigned)rank >= (unsigned)NR || ws_bytes < WS_NEED || n_in < 4) {
    fprintf(stderr, "kernel_launch: unexpected configuration\n");
    return;
  }
  Params p;
  p.x = (const uint8_t*)d_in[0];
  p.w = (const uint8_t*)d_in[1];
  p.sx = (const float*)d_in[2];
  p.sw = (const float*)d_in[3];
  p.out = (float*)d_out;
  for (int i = 0; i < NR; ++i) p.ws[i] = (char*)d_ws_all[(rank + i) & (NR - 1)];
  p.rank = rank;
  p.pad = 0;
  hipLaunchKernelGGL(gemm_rs_kernel, dim3(GRID), dim3(1024), 0, stream, p);
}
```

```cpp
#include <hip/hip_runtime.h>
#include <rccl/rccl.h>
#include <cstdio>
#include <cstdint>
#include <cstddef>

namespace {

constexpr int NR = 8;
constexpr int M_ = 4096, K_ = 4096, N_ = 4096;
constexpr int MS = M_ / NR;
constexpr int KS = K_ / NR;
constexpr int NKB = KS / 128;
constexpr int NJ = N_ / 64;
#define XPUSH 120
#define OWN0 200
#define NPULL_ 36
#define SLOT_T 134
#define MONN 16
#define ARRC 32
#define PRIO 1
#define WIN_ 24
#define SNAP 130
#define FOFF 130
constexpr int WIN = WIN_;
constexpr int GRID = 256;
constexpr int NCOMM = 32;
constexpr int NPULL = NPULL_;
constexpr int PWAVES = (NPULL * (NR - 1) + NCOMM - 1) / NCOMM;
constexpr int XU = 256 - XPUSH;
constexpr int NUNIT = XU + (NKB * NJ - WIN) * 8;
constexpr unsigned NPAIR = NUNIT / 2;
static_assert(XU % 2 == 0 && XPUSH <= GRID && PWAVES <= 8, "");

constexpr unsigned WPIECE = 8192;
constexpr unsigned WSRC = NKB * NJ * WPIECE;
constexpr unsigned WSTAGE = NR * WSRC;
constexpr unsigned XFRAG = 2048;
constexpr unsigned XSRC = NKB * 32 * XFRAG;
constexpr unsigned XSTAGE = NR * XSRC;
constexpr unsigned OFF_W = 0;
constexpr unsigned OFF_X = OFF_W + 2 * WSTAGE;
constexpr unsigned OFF_WE = OFF_X + 4 * XSTAGE;
constexpr unsigned OFF_XE = OFF_WE + 4 * WSRC;
constexpr unsigned OFF_CNT = OFF_XE + 4 * XSTAGE;
constexpr unsigned OFF_TMO = OFF_CNT + GRID * 128;
constexpr unsigned OFF_TKT = OFF_TMO + 128;
constexpr unsigned WS_NEED = OFF_TKT + 32 * 128;

constexpr unsigned SLOT = WPIECE + 2048;
constexpr unsigned LDS_OWN = 7 * SLOT;
constexpr unsigned LDS_CTL = LDS_OWN + NKB * SLOT;

#define ST(i) do {} while (0)
#define STW0(i) do {} while (0)
constexpr unsigned long long DEADLINE_TICKS = 50000000ull;

typedef unsigned u32x4 __attribute__((ext_vector_type(4)));
typedef unsigned u32x8 __attribute__((ext_vector_type(8)));
typedef int i32x2 __attribute__((ext_vector_type(2)));
typedef int i32x8 __attribute__((ext_vector_type(8)));
typedef float f32x4 __attribute__((ext_vector_type(4)));
#define LDS_AS __attribute__((address_space(3)))
#define GLB_AS __attribute__((address_space(1)))
typedef unsigned long long u64;
typedef u64 u64x8 __attribute__((ext_vector_type(8)));
typedef u64 u64x4 __attribute__((ext_vector_type(4)));
typedef u64 u64x2 __attribute__((ext_vector_type(2)));
typedef GLB_AS char gchar;
typedef GLB_AS unsigned gu32;
constexpr int AUX_SYS = 17;

struct Params {
  const uint8_t* x;
  const uint8_t* w;
  const float* sx;
  const float* sw;
  float* out;
  char* ws[NR];
  int rank;
  int pad;
};
static_assert(sizeof(Params) == 0x70 && offsetof(Params, ws) == 0x28 && offsetof(Params, rank) == 0x68, "");

__device__ __forceinline__ __amdgpu_buffer_rsrc_t mk_rsrc(const gchar* p, unsigned bytes) {
  return __builtin_amdgcn_make_buffer_rsrc((void*)(u64)p, 0, bytes, 0x00020000);
}
__device__ __forceinline__ void st16(u32x4 v, __amdgpu_buffer_rsrc_t rs, unsigned off) {
  __builtin_amdgcn_raw_buffer_store_b128(v, rs, off, 0, AUX_SYS);
}
template <int AUX> __device__ __forceinline__ u32x4 ld16(__amdgpu_buffer_rsrc_t rs, unsigned voff, unsigned soff) {
  return __builtin_amdgcn_raw_buffer_load_b128(rs, voff, soff, AUX);
}
__device__ __forceinline__ unsigned ld4(__amdgpu_buffer_rsrc_t rs, unsigned voff, unsigned soff) {
  return __builtin_amdgcn_raw_buffer_load_b32(rs, voff, soff, AUX_SYS);
}
template <int N> __device__ __forceinline__ void wait_vm() {
  asm volatile("s_waitcnt vmcnt(%0)" :: "n"(N) : "memory");
}
__device__ __forceinline__ gchar* sel8(int k, u64 a0, u64 a1, u64 a2, u64 a3, u64 a4, u64 a5, u64 a6, u64 a7) {
  const u64 b0 = (k & 1) ? a1 : a0, b1 = (k & 1) ? a3 : a2, b2 = (k & 1) ? a5 : a4, b3 = (k & 1) ? a7 : a6;
  const u64 c0 = (k & 2) ? b1 : b0, c1 = (k & 2) ? b3 : b2;
  return (gchar*)((k & 4) ? c1 : c0);
}
__device__ __forceinline__ unsigned rfl(unsigned v) { return __builtin_amdgcn_readfirstlane(v); }
__device__ __forceinline__ u32x4 nonzero(u32x4 v) {
#pragma unroll
  for (int e = 0; e < 4; ++e) v[e] = v[e] ? v[e] : 0x80u;
  return v;
}
__device__ __forceinline__ unsigned min4(u32x4 v) { return min(min(v[0], v[1]), min(v[2], v[3])); }
__device__ __forceinline__ unsigned min8(u32x8 v) {
  return min(min(min(v[0], v[1]), min(v[2], v[3])), min(min(v[4], v[5]), min(v[6], v[7])));
}
__device__ __forceinline__ void sleep_until(unsigned long long t0, unsigned ticks) {
  while (__builtin_amdgcn_s_memrealtime() - t0 < ticks) __builtin_amdgcn_s_sleep(2);
}
__device__ __forceinline__ bool expired(gu32* tmo, unsigned call, unsigned long long t0) {
  if (rfl(__hip_atomic_load(tmo, __ATOMIC_RELAXED, __HIP_MEMORY_SCOPE_AGENT)) == call ||
      __builtin_amdgcn_s_memrealtime() - t0 > DEADLINE_TICKS) {
    __hip_atomic_store(tmo, call, __ATOMIC_RELAXED, __HIP_MEMORY_SCOPE_AGENT);
    return true;
  }
  return false;
}

__global__ void __launch_bounds__(1024, 4) gemm_rs_kernel(Params) {
  __shared__ __attribute__((aligned(16))) unsigned char lds[LDS_CTL + 384];
  u64x8 ka;
  u64x4 kb_;
  u64x2 kc;
  asm volatile("s_load_dwordx16 %0, %3, 0x0\n\ts_load_dwordx8 %1, %3, 0x40\n\ts_load_dwordx4 %2, %3, 0x60"
               : "=&s"(ka), "=&s"(kb_), "=&s"(kc) : "s"(__builtin_amdgcn_kernarg_segment_ptr()));
  const int tid = threadIdx.x;
  const int lane = tid & 63;
  const int wave = __builtin_amdgcn_readfirstlane(tid >> 6);
  const int t = blockIdx.x;
  const bool comm = t < NCOMM;
  const int er = lane >> 3, ec = lane & 7;
  const int e = t - NCOMM;
  const bool isx = wave < 8;
  if (isx) __builtin_amdgcn_s_setprio(2);
  const int ek = 1 + (unsigned)(e + NCOMM + 3 + wave) % (NR - 1), ex = wave ? e : GRID - NCOMM + e / (NR - 1);
  auto w_off = [&](int ew) -> unsigned {
    return ((ew >> 6) * 128 + ((ew & 1) * 4 + ((wave >> 1) & 3)) * 16 + (wave & 1) * 8 + er) * N_ + (ew & 62) * 64 + ec * 16;
  };
  auto w_img = [&](int ew) -> unsigned {
    return OFF_WE + (ew & ~1) * WPIECE + ((ew & 1) * 4 + ((wave >> 1) & 3)) * 1024 + ((wave ^ (wave >> 1)) & 1) * 128;
  };
  unsigned e_off = isx ? (ek * MS + (ex & 63) * 8 + er) * KS + (ex >> 6) * 128 + ec * 16 : w_off(e);
  unsigned c_off = OFF_CNT + t * 128;
  asm volatile("" : "+v"(e_off), "+v"(c_off));
  asm volatile("s_waitcnt lgkmcnt(0)" : "+s"(ka), "+s"(kb_), "+s"(kc));
  const u64 ax = ka[0], aw_ = ka[1], asx = ka[2], asw = ka[3], aout = ka[4];
  const u64 w0 = ka[5], w1 = ka[6], w2 = ka[7], w3 = kb_[0], w4 = kb_[1], w5 = kb_[2], w6 = kb_[3], w7 = kc[0];
  const int rank = (int)kc[1];
  u32x4 ev;
  if (!comm) ev = *(const GLB_AS u32x4*)((const gchar*)(isx ? ax : aw_) + ((e_off + (isx ? rank * (MS * KS) : 0)) & (M_ * KS - 1)));
  const unsigned cnt_raw = __hip_atomic_load((gu32*)((gchar*)w0 + c_off), __ATOMIC_RELAXED, __HIP_MEMORY_SCOPE_AGENT);
#define ws_of(k) sel8((k), w0, w1, w2, w3, w4, w5, w6, w7)
  const gchar* const px = (const gchar*)ax;
  const gchar* const pw = (const gchar*)aw_;
  gchar* const myws = (gchar*)w0;
  gu32* const cnt = (gu32*)(myws + OFF_CNT + t * 128);
  gu32* const tmo = (gu32*)(myws + OFF_TMO);
  const int fr = lane & 15, fg = lane >> 4;
  const int xs = (rank + wave) & (NR - 1);
  const bool push = isx && ex < XPUSH;
  const unsigned long long t0 = __builtin_amdgcn_s_memrealtime();
  const auto my_rs = mk_rsrc(myws, OFF_CNT);
  const unsigned e_lane = (ec >> 2) * (isx ? 1024 : WPIECE) + (ec & 3) * 256 + er * 16;
  unsigned e_img = e_lane + (isx ? (ex >> 1) * XFRAG + (ex & 1) * 128 + (push ? OFF_X + rank * XSRC : OFF_XE + ((rank + ek) & (NR - 1)) * XSRC)
                                 : w_img(e));
  u64 e_base = (u64)ws_of(push ? ek : 0);
  asm volatile("" : "+s"(e_base));
  asm volatile("" : "+v"(e_img));
  const unsigned call = rfl(cnt_raw) + 1;
  static_assert(XSTAGE == WSRC, "");
  if (!comm) {
    st16(nonzero(ev), mk_rsrc((gchar*)e_base, OFF_CNT), e_img + (call & 3) * XSTAGE);
    if (!isx && e < NCOMM) {
      const u32x4 ev2 = *(const GLB_AS u32x4*)(pw + w_off(GRID - NCOMM + e));
      st16(nonzero(ev2), my_rs, e_lane + w_img(GRID - NCOMM + e) + (call & 3) * XSTAGE);
    }
  }
  __builtin_amdgcn_s_setprio(0);

  const int xcd = t & 7, idx = (t - NCOMM) >> 3;
  const int ti = idx & 3, tu = idx >> 2;
  const int tl = 8 + tu * 8 + xcd;
  const int joff = (rank & 3) * 16 + (rank >> 2) * 8;
  const int tj = (tl + joff) & (NJ - 1);
  const int tx = (xcd + joff) & (NJ - 1);
  const bool extra = tu < 4;
  const bool win = tl >= NJ - WIN;
  const int kbs = tu == 4 ? 2 : rank >> 2;
  auto kb_of = [&](int r) { return !win ? r : r == NKB - 1 ? kbs : r + (r >= kbs); };
  volatile LDS_AS unsigned* ctl = (volatile LDS_AS unsigned*)(lds + LDS_CTL);
  volatile LDS_AS unsigned* ready = ctl;
  volatile LDS_AS unsigned* abortf = ctl + 8;
  volatile LDS_AS unsigned* invdone = ctl + 9;
  volatile LDS_AS unsigned* ownrdy = ctl + 12;
  volatile LDS_AS unsigned* consumed = ctl + 16;
  volatile LDS_AS unsigned* sched = ctl + 80;

  if (wave == 8) {
    if (lane < 16) ctl[lane] = 0;
    consumed[lane] = 0;
    asm volatile("s_waitcnt lgkmcnt(0)" ::: "memory");
  }
  asm volatile("" :: "s"(call));
  __builtin_amdgcn_s_barrier();
  if (tid == 0) __hip_atomic_store(cnt, call, __ATOMIC_RELAXED, __HIP_MEMORY_SCOPE_AGENT);
  auto rezero3 = [&]() {
    const u32x4 z = {0u, 0u, 0u, 0u};
    st16(z, my_rs, OFF_X + ((call + 2) & 3) * XSTAGE + xs * XSRC + t * 1024 + lane * 16);
    st16(z, my_rs, OFF_XE + ((call + 2) & 3) * XSTAGE + xs * XSRC + t * 1024 + lane * 16);
    st16(z, my_rs, OFF_WE + ((call + 2) & 3) * WSRC + t * WPIECE + tid * 16);
  };

  if (comm) {
    if (wave < 8) {
      if (wave < 2) {
        const int mk = 1 + min(t * PWAVES + wave * (PWAVES - 1), NPULL * (NR - 1) - 1) / NPULL;
        const unsigned b0 = OFF_X + (call & 3) * XSTAGE + ((rank + mk) & (NR - 1)) * XSRC;
        const unsigned se = (((lane & (MONN - 1)) * 64 / MONN + t % (64 / MONN)) * XPUSH / 64);
        for (unsigned n = 1;; ++n) {
          const unsigned f = ld4(my_rs, b0 + (se >> 1) * XFRAG + (se & 1) * 128, 0);
          if (__builtin_popcountll(__builtin_amdgcn_ballot_w64(f != 0)) >= ARRC) break;
          if ((n & 63) == 0 && expired(tmo, call, t0)) break;
        }
        if (lane == 0) ctl[mk] = 1;
      }
      __builtin_amdgcn_s_barrier();
      rezero3();
      if (t == 0 && wave == 0 && lane < 2 * NR)
        __hip_atomic_store((gu32*)(myws + OFF_TKT + ((call + 2) & 3) * 8 * 128 + (lane < NR ? lane * 128 : (lane - NR) * 4)),
                           0u, __ATOMIC_RELAXED, __HIP_MEMORY_SCOPE_AGENT);
      return;
    }
    const int pwv = wave - 8;
    const int g = t * PWAVES + pwv;
    if (pwv >= PWAVES || g >= NPULL * (NR - 1)) return;
    const int k = 1 + g / NPULL, q = g % NPULL;
    const int s = (rank + k) & (NR - 1);
    const u64 sp = (u64)ws_of(k);
    const u32x4 srd_s = {rfl((unsigned)sp), rfl((unsigned)(sp >> 32) & 0xffffu), OFF_CNT, 0x00020000u};
    const u32x4 srd_d = {rfl((unsigned)w0), rfl((unsigned)(w0 >> 32) & 0xffffu), OFF_CNT, 0x00020000u};
    constexpr unsigned INV = 1u << 20;
    u64 ktab = 0;
#pragma unroll
    for (int b = 0; b < 32 - WIN / 8; ++b) {
      const int r = b >> 3, lh = b & 7;
      ktab |= (u64)(lh >= 6 ? r + (r >= (rank >> 2)) : lh * 8 >= NJ - WIN ? r + (r >= 2) : r) << (2 * b);
    }
    unsigned status = 0;
    const unsigned sx0 = OFF_XE + (call & 3) * XSTAGE + rank * XSRC + XPUSH * 1024, sw0 = OFF_WE + (call & 3) * WSRC;
    const unsigned dx0 = OFF_X + (call & 3) * XSTAGE + s * XSRC + XPUSH * 1024, dw0 = OFF_W + (call & 1) * WSTAGE + s * WSRC;
    const unsigned tkt = OFF_TKT + ((call & 3) * 8 + k) * 128;
    while (!rfl(ctl[k])) __builtin_amdgcn_s_sleep(1);
    {
      unsigned cur, nxt, s1, s2;
      u64 tnow;
#define VERIFY(N, X0, X1, X2, X3, L) \
      "s_cmp_lt_u32 %[cur], %[npair]\n\t" \
      "s_cbranch_scc0 .Lok" L "_%=\n\t" \
      "s_waitcnt vmcnt(" N ")\n\t" \
      ".Lchk" L "_%=:\n\t" \
      "v_min_u32 v121, " X2 ", " X3 "\n\t" \
      "v_min3_u32 v121, " X0 ", " X1 ", v121\n\t" \
      "v_cmp_eq_u32 vcc, 0, v121\n\t" \
      "s_nop 0\n\t" \
      "s_cbranch_vccnz .Lretry" L "_%=\n\t" \
      ".Lok" L "_%=:\n\t"
#define RETRY(X, OFS, L) \
      ".Lretry" L "_%=:\n\t" \
      "s_sleep 8\n\t" \
      "s_memrealtime %[tnow]\n\t" \
      "s_waitcnt lgkmcnt(0)\n\t" \
      "v_mov_b64 v[122:123], %[tnow]\n\t" \
      "v_cmp_lt_u64 vcc, %[tend], v[122:123]\n\t" \
      "s_nop 0\n\t" \
      "s_cbranch_vccnz .Lfail_%=\n\t" \
      "buffer_load_dwordx4 " X ", v119, %[srs], 0 offen " OFS " sc0 sc1\n\t" \
      "s_waitcnt vmcnt(0)\n\t" \
      "s_branch .Lchk" L "_%=\n\t"
#define HALF(XA, XA0, XA1, XA2, XA3, XB, XB0, XB1, XB2, XB3, YA, YB, TIN, TOUT, LA, LB) \
      VERIFY("3", XA0, XA1, XA2, XA3, LA) \
      "v_readfirstlane_b32 %[nxt], " TIN "\n\t" \
      "s_mov_b64 exec, 1\n\t" \
      "global_atomic_add " TOUT ", %[tkt], %[inc], %[ws] sc0\n\t" \
      "s_mov_b64 exec, -1\n\t" \
      "s_add_u32 %[nxt], %[nxt], %[base]\n\t" \
      "s_sub_u32 %[s1], %[nxt], %[xpair]\n\t" \
      "s_add_u32 %[s2], %[s1], %[rot]\n\t" \
      "s_and_b32 %[s2], %[s2], 31\n\t" \
      "s_add_u32 vcc_lo, %[s1], %[joff]\n\t" \
      "s_and_b32 vcc_lo, vcc_lo, 0xe0\n\t" \
      "s_or_b32 %[s2], %[s2], vcc_lo\n\t" \
      "s_lshr_b32 %[s1], %[s1], 5\n\t" \
      "s_lshl_b32 %[s1], %[s1], 1\n\t" \
      "s_lshr_b64 vcc, %[ktab], %[s1]\n\t" \
      "s_and_b32 %[s1], vcc_lo, 3\n\t" \
      "s_lshl_b32 %[s1], %[s1], 8\n\t" \
      "s_or_b32 %[s2], %[s2], %[s1]\n\t" \
      "s_cmp_lt_u32 %[nxt], %[xpair]\n\t" \
      "s_cselect_b32 %[s2], %[nxt], %[s2]\n\t" \
      "s_cselect_b32 %[s1], %[sx], %[sw]\n\t" \
      "s_lshl_b32 %[s2], %[s2], 11\n\t" \
      "s_add_u32 %[s1], %[s1], %[s2]\n\t" \
      "s_cmp_lt_u32 %[nxt], %[npair]\n\t" \
      "s_cselect_b32 %[s1], %[s1], 0x80000000\n\t" \
      "v_add_u32 v118, %[s1], %[lo]\n\t" \
      "buffer_load_dwordx4 " YA ", v118, %[srs], 0 offen sc0 sc1\n\t" \
      "buffer_store_dwordx4 " XA ", v120, %[srd], 0 offen sc0 sc1\n\t" \
      VERIFY("4", XB0, XB1, XB2, XB3, LB) \
      "buffer_load_dwordx4 " YB ", v118, %[srs], 0 offen offset:1024 sc0 sc1\n\t" \
      "buffer_store_dwordx4 " XB ", v120, %[srd], 0 offen offset:1024 sc0 sc1\n\t" \
      "s_cmp_lt_u32 %[nxt], %[xpair]\n\t" \
      "s_cselect_b32 %[s1], %[dx], %[dw]\n\t" \
      "s_add_u32 %[s1], %[s1], %[s2]\n\t" \
      "s_cmp_lt_u32 %[nxt], %[npair]\n\t" \
      "s_cselect_b32 %[s1], %[s1], 0x80000000\n\t" \
      "v_mov_b32 v119, v118\n\t" \
      "v_add_u32 v120, %[s1], %[lo]\n\t" \
      PSTAMP \
      "s_mov_b32 %[cur], %[nxt]\n\t"
#define PSTAMP
#define PSTAMP_OPS
      asm volatile(
          "s_mov_b32 %[cur], %[inv]\n\t"
          "v_mov_b32 v117, %[first]\n\t"
          "v_mov_b32 v120, 0x80000000\n\t"
          "s_nop 4\n\t"
          ".Lloop_%=:\n\t"
          HALF("v[100:103]", "v100", "v101", "v102", "v103", "v[104:107]", "v104", "v105", "v106", "v107",
               "v[108:111]", "v[112:115]", "v117", "v116", "a", "b")
          "s_cmp_ge_u32 %[cur], %[npair]\n\t"
          "s_cbranch_scc1 .Ldone_%=\n\t"
          HALF("v[108:111]", "v108", "v109", "v110", "v111", "v[112:115]", "v112", "v113", "v114", "v115",
               "v[100:103]", "v[104:107]", "v116", "v117", "c", "d")
          "s_cmp_lt_u32 %[cur], %[npair]\n\t"
          "s_cbranch_scc1 .Lloop_%=\n\t"
          "s_branch .Ldone_%=\n\t"
          RETRY("v[100:103]", "", "a")
          RETRY("v[104:107]", "offset:1024", "b")
          RETRY("v[108:111]", "", "c")
          RETRY("v[112:115]", "offset:1024", "d")
          ".Lfail_%=:\n\t"
          "s_mov_b32 %[status], 1\n\t"
          ".Ldone_%=:\n\t"
          "s_waitcnt vmcnt(0)"
          : [cur] "=&s"(cur), [nxt] "=&s"(nxt), [s1] "=&s"(s1), [s2] "=&s"(s2), [tnow] "=&s"(tnow), [status] "+s"(status) PSTAMP_OPS
          : [srs] "s"(srd_s), [srd] "s"(srd_d), [ws] "s"(w0), [tend] "s"(t0 + DEADLINE_TICKS),
            [sx] "s"(rfl(sx0)), [sw] "s"(rfl(sw0)), [dx] "s"(rfl(dx0)), [dw] "s"(rfl(dw0)),
            [base] "n"(NPULL), [first] "s"(rfl((unsigned)(q - NPULL))),
            [ktab] "s"(ktab), [rot] "s"(rfl(k * 4)), [joff] "s"(rfl(joff * 4)), [npair] "n"(NPAIR), [xpair] "n"(XU / 2), [inv] "n"(INV),
            [tkt] "v"(tkt), [inc] "v"(1u), [lo] "v"(lane * 16)
          : "memory", "vcc", "scc", "v100", "v101", "v102", "v103", "v104", "v105", "v106", "v107", "v108", "v109",
            "v110", "v111", "v112", "v113", "v114", "v115", "v116", "v117", "v118", "v119", "v120", "v121", "v122", "v123");
    }
    if (status) __hip_atomic_store(tmo, call, __ATOMIC_RELAXED, __HIP_MEMORY_SCOPE_AGENT);
    if (pwv < 8) ST(48 + pwv);
    return;
  }

  if (wave == 8) {
    sleep_until(t0, OWN0);
    const gchar* w_even = pw + (size_t)fr * N_ + fg * 16 + tj * 64;
    const gchar* w_odd = pw + (size_t)(fr ^ 8) * N_ + fg * 16 + tj * 64;
    const gchar* w_ex = pw + (size_t)(fg * 16 + (fr ^ ((fg & 1) << 3))) * N_ + tx * 64 + tu * 16;
    for (int kb = 0; kb < NKB; ++kb) {
#pragma unroll
      for (int v = 0; v < 8; ++v)
        __builtin_amdgcn_global_load_lds((const GLB_AS void*)(((v & 1) ? w_odd : w_even) + (size_t)(kb * 128 + v * 16) * N_),
                                         (LDS_AS void*)(lds + LDS_OWN + kb * SLOT + v * 1024), 16, 0, 0);
      if (extra) {
        __builtin_amdgcn_global_load_lds((const GLB_AS void*)(w_ex + (size_t)(kb * 128) * N_),
                                         (LDS_AS void*)(lds + LDS_OWN + kb * SLOT + 8192), 16, 0, 0);
        __builtin_amdgcn_global_load_lds((const GLB_AS void*)(w_ex + (size_t)(kb * 128 + 64) * N_),
                                         (LDS_AS void*)(lds + LDS_OWN + kb * SLOT + 9216), 16, 0, 0);
      }
      wait_vm<0>();
      if (lane == 0) ownrdy[kb] = 1;
    }
    gu32* const tk = (gu32*)(myws + OFF_TKT + (call & 3) * 8 * 128);
    if (win) {
      sleep_until(t0, 1000 + (GRID - 1 - t) * 15);
      __builtin_amdgcn_fence(__ATOMIC_ACQUIRE, "");
      wait_vm<0>();
      for (unsigned it = 1;; ++it) {
        const unsigned f = __hip_atomic_load(tk + (lane & 7), __ATOMIC_RELAXED, __HIP_MEMORY_SCOPE_AGENT);
        if (__builtin_amdgcn_ballot_w64(f == 0) == 0) {
          sched[lane & 7] = f;
          asm volatile("s_waitcnt lgkmcnt(0)" ::: "memory");
          if (lane == 0) *invdone = 1;
          break;
        }
        if ((it & 63) == 0 && expired(tmo, call, t0)) { if (lane == 0) *abortf = 1; break; }
        __builtin_amdgcn_s_sleep(8);
      }
    } else if (t == NCOMM + 3 * 32) {
      for (unsigned it = 1;; ++it) {
        const unsigned ta = (unsigned)__builtin_amdgcn_s_memrealtime();
        const unsigned f = __hip_atomic_load(tk + (1 + lane % 7) * 32, __ATOMIC_RELAXED, __HIP_MEMORY_SCOPE_AGENT);
        unsigned mx = 0;
#pragma unroll
        for (int k = 0; k < NR - 1; ++k) mx = max(mx, (unsigned)__builtin_amdgcn_readlane(f, k));
        const unsigned tb = (unsigned)__builtin_amdgcn_s_memrealtime();
        if (mx + SNAP >= NPAIR - NPULL && (tb - ta < 80 || mx + SNAP / 2 >= NPAIR - NPULL)) {
          const int left = max((int)(NPAIR - NPULL) - (int)mx, 0);
          const unsigned fmin = ta + 25 + left * SLOT_T / 40 + FOFF;
          const unsigned lag = __builtin_amdgcn_update_dpp(0u, f ? min((mx - f + 1) / 2, 80u) + 1 : 1000u, 0x111  , 0xf, 0xf, false);
          if (lane < NR) __hip_atomic_store(tk + lane, lane ? lag : fmin | 1u, __ATOMIC_RELAXED, __HIP_MEMORY_SCOPE_AGENT);
          break;
        }
        if ((it & 63) == 0 && expired(tmo, call, t0)) break;
        __builtin_amdgcn_s_sleep(8);
      }
    }
    return;
  }

  if (wave > 8) {
    const int sv = wave - 8;
    const int s = (rank + sv) & (NR - 1);
    const unsigned ex_lane = fg * 1024 + tu * 256 + fr * 16;
    sleep_until(t0, OWN0);
    const unsigned zo = OFF_W + ((call + 1) & 1) * WSTAGE + s * WSRC + ti * NJ * WPIECE;
    auto rezero = [&](int part) {
      const u32x4 z = {0u, 0u, 0u, 0u};
      st16(z, my_rs, zo + tj * WPIECE + lane * 16 + part * 3072);
      st16(z, my_rs, zo + tj * WPIECE + lane * 16 + part * 3072 + 1024);
      if (part < 2) {
        st16(z, my_rs, zo + tj * WPIECE + lane * 16 + part * 3072 + 2048);
        if (extra) st16(z, my_rs, zo + tx * WPIECE + ex_lane + part * 4096);
      }
    };
    LDS_AS unsigned char* slot = (LDS_AS unsigned char*)lds + (sv - 1) * SLOT + lane * 16;
    const unsigned sb = OFF_W + (call & 1) * WSTAGE + s * WSRC;
    auto slot_free = [&](unsigned kb) {
      while (__builtin_amdgcn_ballot_w64(consumed[sv * 8 + (lane & 7)] < kb) && !rfl(*abortf)) __builtin_amdgcn_s_sleep(2);
    };
    for (int kb = 0; kb < (win ? NKB - 1 : NKB); ++kb) {
      const unsigned so = sb + (kb_of(kb) * NJ + tj) * WPIECE, so_ex = sb + (kb * NJ + tx) * WPIECE;
      bool dead = false;
      for (unsigned it = 1;; ++it) {
        if (rfl(ld4(my_rs, 0, so)) != 0) break;
        if ((it & 63) == 0 && expired(tmo, call, t0)) { dead = true; break; }
        __builtin_amdgcn_s_sleep(2);
      }
      u32x4 d[10];
      for (unsigned it = 1; !dead; ++it) {
        unsigned m = 0xffffffffu;
#pragma unroll
        for (int v = 0; v < 8; ++v) d[v] = ld16<AUX_SYS>(my_rs, lane * 16 + v * 1024, so);
        if (extra) {
          d[8] = ld16<AUX_SYS>(my_rs, ex_lane, so_ex);
          d[9] = ld16<AUX_SYS>(my_rs, ex_lane + 4096, so_ex);
          m = min(min4(d[8]), min4(d[9]));
        }
#pragma unroll
        for (int v = 0; v < 8; ++v) m = min(m, min4(d[v]));
        if (__builtin_amdgcn_ballot_w64(m == 0) == 0) break;
        if ((it & 63) == 0 && expired(tmo, call, t0)) dead = true;
      }
      if (dead) { if (lane == 0) *abortf = 1; return; }
      slot_free(kb);
#pragma unroll
      for (int v = 0; v < 8; ++v) *(LDS_AS u32x4*)(slot + v * 1024) = d[v];
      if (extra) { *(LDS_AS u32x4*)(slot + 8192) = d[8]; *(LDS_AS u32x4*)(slot + 9216) = d[9]; }
      asm volatile("s_waitcnt lgkmcnt(0)" ::: "memory");
      if (lane == 0) ready[sv] = kb + 1;
      ST(4 + kb * 8 + sv);
      if (kb < NKB - 1) rezero(kb);
      if (kb == 1 && ti == 0) {
        const unsigned f = ld4(my_rs, lane * 128, sb + ((NKB - 1) * NJ + tj) * WPIECE);
        asm volatile("" :: "v"(f));
      }
    }
    if (!win) return;
    while (!rfl(*invdone) && !rfl(*abortf)) __builtin_amdgcn_s_sleep(2);
    if (rfl(*abortf)) return;
    const unsigned po = (kbs * NJ + tj) * WPIECE, eo = OFF_WE + (call & 3) * WSRC + po;
    const unsigned sc_l = sched[lane & 7];
    int c0 = 0, c1 = 0;
#pragma unroll
    for (int k = 1; k < NR; ++k) {
      const int ak = (int)__builtin_amdgcn_readlane(sc_l, k) - 1;
      c0 += max(lane - ak, 0);
      c1 += max(lane + 64 - ak, 0);
    }
    unsigned fire[2], voff[2];
    u64 via[2];
#pragma unroll
    for (int h = 0; h < 2; ++h) {
      const int J = (h * WIN + (unsigned)(tl - sv * (WIN / 8)) % WIN) * (NR - 1) + sv - 1;
      const int row = __builtin_popcountll(__builtin_amdgcn_ballot_w64(c0 <= J)) + __builtin_popcountll(__builtin_amdgcn_ballot_w64(c1 <= J)) - 1;
      const int p = J - (int)(row < 64 ? __builtin_amdgcn_readlane(c0, row & 63) : __builtin_amdgcn_readlane(c1, row & 63));
      const unsigned act = (unsigned)__builtin_amdgcn_ballot_w64((int)sc_l - 1 <= row) & 0xfeu;
      int lk = sv;
#pragma unroll
      for (int k = NR - 1; k >= 1; --k)
        if (((act >> k) & 1u) && __builtin_popcount(act & ((1u << k) - 1u)) == p) lk = k;
      fire[h] = (unsigned)__builtin_amdgcn_readlane(sc_l, 0) + row * SLOT_T / 20;
      via[h] = (u64)ws_of(lk);
      voff[h] = (lk == sv ? eo : sb + po) + h * 512;
      asm volatile("" : "+s"(fire[h]), "+s"(via[h]), "+s"(voff[h]));
    }
    slot_free(NKB - 1);
    const unsigned hl = (lane >> 5) * 1024 + (lane & 31) * 16;
    LDS_AS unsigned char* hslot = (LDS_AS unsigned char*)lds + (sv - 1) * SLOT + hl;
#pragma nounroll
    for (unsigned h = 0; h < 2; ++h) {
      asm volatile("" : "+s"(h));
      const unsigned fi = h ? fire[1] : fire[0], vo = h ? voff[1] : voff[0];
      const auto vrs = mk_rsrc((gchar*)(h ? via[1] : via[0]), OFF_CNT);
      while ((int)(fi - (unsigned)__builtin_amdgcn_s_memrealtime()) > 0) __builtin_amdgcn_s_sleep(1);
      u32x4 d[4];
#pragma unroll
      for (int v = 0; v < 4; ++v) d[v] = ld16<0>(vrs, hl + v * 2048, vo);
      unsigned m = 0xffffffffu;
#pragma unroll
      for (int v = 0; v < 4; ++v) {
        *(LDS_AS u32x4*)(hslot + h * 512 + v * 2048) = d[v];
        m = min(m, min4(d[v]));
        __builtin_amdgcn_sched_barrier(0);
      }
      for (unsigned it = 1; __builtin_expect(__builtin_amdgcn_ballot_w64(m == 0) != 0, 0); ++it) {
        if ((it & 63) == 0 && expired(tmo, call, t0)) { if (lane == 0) *abortf = 1; return; }
        m = 0xffffffffu;
#pragma unroll
        for (int v = 0; v < 4; ++v) {
          const u32x4 r = ld16<AUX_SYS>(mk_rsrc(ws_of(sv), OFF_CNT), hl + v * 2048, eo + h * 512);
          *(LDS_AS u32x4*)(hslot + h * 512 + v * 2048) = r;
          m = min(m, min4(r));
        }
      }
      asm volatile("s_waitcnt lgkmcnt(0)" ::: "memory");
      if (lane == 0) ready[sv] = NKB + h;
    }
    ST(4 + 24 + sv);
    return;
  }

  sleep_until(t0, OWN0);
  const int mb = ti * 8 + wave;
  const auto x_own_rs = mk_rsrc(px, M_ * KS);
  const unsigned x_own_lane = (unsigned)(rank * MS + mb * 16 + fr) * KS + fg * 16;
  const unsigned x_st_lane = OFF_X + (call & 3) * XSTAGE + mb * XFRAG + lane * 16;
  float sc = __builtin_bit_cast(float, rfl(__builtin_bit_cast(unsigned, *(const GLB_AS float*)asx * *(const GLB_AS float*)asw)));
  asm volatile("" : "+s"(sc));
  auto ldx_own = [&](unsigned kb) {
    const u32x4 lo = ld16<0>(x_own_rs, x_own_lane, kb * 128);
    const u32x4 hi = ld16<0>(x_own_rs, x_own_lane + 64, kb * 128);
    return __builtin_shufflevector(lo, hi, 0, 1, 2, 3, 4, 5, 6, 7);
  };
  auto ldx_st = [&](int sv, unsigned kb) {
    const unsigned so = ((rank + sv) & (NR - 1)) * XSRC + kb * 32 * XFRAG;
    const u32x4 lo = ld16<AUX_SYS>(my_rs, x_st_lane, so);
    const u32x4 hi = ld16<AUX_SYS>(my_rs, x_st_lane + 1024, so);
    return __builtin_shufflevector(lo, hi, 0, 1, 2, 3, 4, 5, 6, 7);
  };
  const unsigned tr_lane = fg * 1024 + ((fg & 1) << 7) + fr * 8;

  f32x4 acc[5] = {};
  bool failed = false;
  auto step = [&](unsigned base, u32x8 xv) {
    const i32x8 bx = __builtin_bit_cast(i32x8, xv);
    unsigned tl = tr_lane;
    asm volatile("" : "+v"(tl));
    LDS_AS unsigned char* sl = (LDS_AS unsigned char*)lds + base;
    if (extra) {
      i32x8 aw;
      const unsigned te = tl - fg * 768 + 8192;
#pragma unroll
      for (int c = 0; c < 4; ++c) {
        const i32x2 r = __builtin_amdgcn_ds_read_tr8_b64_v2i32((LDS_AS i32x2*)(sl + (c >> 1) * 1024 + (te ^ ((c & 1) << 7))));
        aw[2 * c] = r[0]; aw[2 * c + 1] = r[1];
      }
      acc[4] = __builtin_amdgcn_mfma_scale_f32_16x16x128_f8f6f4(aw, bx, acc[4], 0, 0, 0, 0x7f7f7f7f, 0, 0x7f7f7f7f);
    }
#pragma unroll
    for (int q = 0; q < 4; ++q) {
      i32x8 aw;
#pragma unroll
      for (int c = 0; c < 4; ++c) {
        const i32x2 r = __builtin_amdgcn_ds_read_tr8_b64_v2i32((LDS_AS i32x2*)(sl + (c >> 1) * 4096 + q * 256 + (tl ^ ((c & 1) << 7))));
        aw[2 * c] = r[0]; aw[2 * c + 1] = r[1];
      }
      acc[q] = __builtin_amdgcn_mfma_scale_f32_16x16x128_f8f6f4(aw, bx, acc[q], 0, 0, 0, 0x7f7f7f7f, 0, 0x7f7f7f7f);
    }
  };
#pragma unroll
  for (unsigned kb = 0; kb < NKB; ++kb) {
    const u32x8 xv = ldx_own(kb);
    while (!rfl(ownrdy[kb])) __builtin_amdgcn_s_sleep(1);
    step(LDS_OWN + kb * SLOT, xv);
  }
  while (!((unsigned)__builtin_amdgcn_ballot_w64(ctl[lane & 15] != 0u) & 0x1feu)) __builtin_amdgcn_s_sleep(4);
  rezero3();
  u32x8 xf[NR];
  auto load_x = [&](unsigned kb) {
    for (unsigned it = 1;; ++it) {
      unsigned m = 0xffffffffu;
#pragma unroll
      for (int sv = 1; sv < NR; ++sv) xf[sv] = ldx_st(sv, kb);
#pragma unroll
      for (int sv = 1; sv < NR; ++sv) m = min(m, min8(xf[sv]));
      if (__builtin_amdgcn_ballot_w64(m == 0) == 0) break;
      if ((it & 15) == 0 && expired(tmo, call, t0)) { failed = true; break; }
      __builtin_amdgcn_s_sleep(32);
    }
  };
  auto next_ready = [&](unsigned& pend, unsigned level) {
    unsigned rdy;
    for (;;) {
      const unsigned rv = ctl[lane & 15];
      rdy = (unsigned)__builtin_amdgcn_ballot_w64(rv > ((lane & 15) < 8 ? level : 0u)) & (pend | 0x100u);
      if (rdy) break;
      __builtin_amdgcn_s_sleep(1);
    }
    pend &= ~rdy;
    if (rdy & 0x100u) { failed = true; pend = 0; rdy = 0; }
    return rdy;
  };
  for (unsigned kb = 0; kb < (win ? NKB - 1 : NKB) && !failed; ++kb) {
    load_x(kb_of(kb));
    for (unsigned pend = failed ? 0u : 0xfeu; pend;) {
      const unsigned rdy = next_ready(pend, kb);
#pragma unroll
      for (int sv = 1; sv < NR; ++sv)
        if (rdy & (1u << sv)) {
          step((sv - 1) * SLOT, xf[sv]);
          asm volatile("s_waitcnt lgkmcnt(0)" ::: "memory");
          consumed[sv * 8 + wave] = kb + 1;
        }
    }
  }
  if (win && !failed) load_x(kbs);
  const auto o_rs = mk_rsrc((gchar*)aout, MS * N_ * 4);
  constexpr float LOG2E = 1.4426950408889634f;
  auto silu4 = [&](f32x4 a) {
    const f32x4 y = a * (failed ? __builtin_nanf("") : sc);
    f32x4 z = y * -LOG2E;
#pragma unroll
    for (int e = 0; e < 4; ++e) z[e] = __builtin_amdgcn_exp2f(__builtin_amdgcn_fmed3f(z[e], -60.0f * LOG2E, 60.0f * LOG2E));
    z += 1.0f;
#pragma unroll
    for (int e = 0; e < 4; ++e) z[e] = __builtin_amdgcn_rcpf(z[e]);
    return __builtin_bit_cast(u32x4, y * z);
  };
  if (extra) st16(silu4(acc[4]), o_rs, ((unsigned)(mb * 16 + fr) * N_ + fg * 4 + tx * 64 + tu * 16) * 4);
#pragma nounroll
  for (unsigned h = 0; h < 2; ++h) {
    asm volatile("" : "+s"(h));
    for (unsigned pend = win && !failed ? 0xfeu : 0u; pend;) {
      const unsigned rdy = next_ready(pend, NKB - 1 + h);
#pragma unroll
      for (int sv = 1; sv < NR; ++sv)
        if (rdy & (1u << sv)) {
          const i32x8 bx = __builtin_bit_cast(i32x8, xf[sv]);
          unsigned tl2 = tr_lane + h * 512;
          asm volatile("" : "+v"(tl2));
          LDS_AS unsigned char* sl = (LDS_AS unsigned char*)lds + (sv - 1) * SLOT;
#pragma unroll
          for (int q = 0; q < 2; ++q) {
            i32x8 aw;
#pragma unroll
            for (int c = 0; c < 4; ++c) {
              const i32x2 r = __builtin_amdgcn_ds_read_tr8_b64_v2i32((LDS_AS i32x2*)(sl + (c >> 1) * 4096 + q * 256 + (tl2 ^ ((c & 1) << 7))));
              aw[2 * c] = r[0]; aw[2 * c + 1] = r[1];
            }
            acc[q] = __builtin_amdgcn_mfma_scale_f32_16x16x128_f8f6f4(aw, bx, acc[q], 0, 0, 0, 0x7f7f7f7f, 0, 0x7f7f7f7f);
          }
        }
    }
    if (failed && lane == 0) *abortf = 1;
    STW0(2);
    const unsigned o_pair = ((unsigned)(mb * 16 + (fr & 7)) * N_ + (fr >> 3) * 16 + fg * 4 + tj * 64 + h * 32) * 4;
    const u32x4 o0 = silu4(acc[0]), o1 = silu4(acc[1]);
    u32x4 lo, hi;
#pragma unroll
    for (int e = 0; e < 4; ++e) {
      lo[e] = __builtin_amdgcn_update_dpp(o0[e], o1[e], 0x128  , 0xf, 0xc, false);
      hi[e] = __builtin_amdgcn_update_dpp(o1[e], o0[e], 0x128  , 0xf, 0x3, false);
    }
    st16(lo, o_rs, o_pair);
    st16(hi, o_rs, o_pair + 8 * N_ * 4);
    acc[0] = acc[2];
    acc[1] = acc[3];
  }
}

}

extern "C" void kernel_launch(int rank, int nranks, ncclComm_t comm,
                              hipStream_t stream,
                              void* const* d_in, const int* in_sizes, int n_in,
                              void* d_out, int out_size,
                              const int* in_dtypes, int out_dtype,
                              void* const* d_out_all,
                              void* const* d_ws_all, size_t ws_bytes) {
  if (nranks != NR || (unsigned)rank >= (unsigned)NR || ws_bytes < WS_NEED || n_in < 4) {
    fprintf(stderr, "kernel_launch: unexpected configuration\n");
    return;
  }
  Params p;
  p.x = (const uint8_t*)d_in[0];
  p.w = (const uint8_t*)d_in[1];
  p.sx = (const float*)d_in[2];
  p.sw = (const float*)d_in[3];
  p.out = (float*)d_out;
  for (int i = 0; i < NR; ++i) p.ws[i] = (char*)d_ws_all[(rank + i) & (NR - 1)];
  p.rank = rank;
  p.pad = 0;
  hipLaunchKernelGGL(gemm_rs_kernel, dim3(GRID), dim3(1024), 0, stream, p);
}
```

```cpp
#include <hip/hip_runtime.h>
#include <rccl/rccl.h>
#include <cstdio>
#include <cstdint>
#include <cstddef>

namespace {

constexpr int NR = 8;
constexpr int M_ = 4096, K_ = 4096, N_ = 4096;
constexpr int MS = M_ / NR;
constexpr int KS = K_ / NR;
constexpr int NKB = KS / 128;
constexpr int NJ = N_ / 64;
#define XPUSH 120
#define OWN0 200
#define NPULL_ 36
#define SLOT_T 134
#define MONN 16
#define ARRC 32
#define PRIO 1
#define WIN_ 24
#define ORIGIN 1
#define XONE 1
#define SNAP 130
#define FOFF 130
constexpr int WIN = WIN_;
constexpr int GRID = 256;
constexpr int NCOMM = 32;
constexpr int NPULL = NPULL_;
constexpr int PWAVES = (NPULL * (NR - 1) + NCOMM - 1) / NCOMM;
constexpr int XU = 256 - XPUSH;
constexpr int NUNIT = XU + (NKB * NJ - WIN) * 8;
constexpr unsigned NPAIR = NUNIT / 2;
static_assert(XU % 2 == 0 && XPUSH <= GRID && PWAVES <= 8, "");

constexpr unsigned WPIECE = 8192;
constexpr unsigned WSRC = NKB * NJ * WPIECE;
constexpr unsigned WSTAGE = NR * WSRC;
constexpr unsigned XFRAG = 2048;
constexpr unsigned XSRC = NKB * 32 * XFRAG;
constexpr unsigned XPEER = 8 * XSRC;
constexpr unsigned OFF_W = 0;
constexpr unsigned OFF_X = OFF_W + 2 * WSTAGE;
constexpr unsigned OFF_XE = OFF_X + 4 * XSRC;
constexpr unsigned OFF_WE = OFF_X + NR * XPEER;
constexpr unsigned OFF_CNT = OFF_WE + 4 * WSRC;
constexpr unsigned OFF_TMO = OFF_CNT + GRID * 128;
constexpr unsigned OFF_TKT = OFF_TMO + 128;
constexpr unsigned WS_NEED = OFF_TKT + 32 * 128;

constexpr unsigned SLOT = WPIECE + 2048;
constexpr unsigned LDS_OWN = 7 * SLOT;
constexpr unsigned LDS_CTL = LDS_OWN + NKB * SLOT;

#define ST(i) do {} while (0)
#define STW0(i) do {} while (0)
constexpr unsigned long long DEADLINE_TICKS = 50000000ull;

typedef unsigned u32x4 __attribute__((ext_vector_type(4)));
typedef unsigned u32x8 __attribute__((ext_vector_type(8)));
typedef int i32x2 __attribute__((ext_vector_type(2)));
typedef int i32x8 __attribute__((ext_vector_type(8)));
typedef float f32x4 __attribute__((ext_vector_type(4)));
#define LDS_AS __attribute__((address_space(3)))
#define GLB_AS __attribute__((address_space(1)))
typedef unsigned long long u64;
typedef u64 u64x8 __attribute__((ext_vector_type(8)));
typedef u64 u64x4 __attribute__((ext_vector_type(4)));
typedef u64 u64x2 __attribute__((ext_vector_type(2)));
typedef GLB_AS char gchar;
typedef GLB_AS unsigned gu32;
constexpr int AUX_SYS = 17;

struct Params {
  const uint8_t* x;
  const uint8_t* w;
  const float* sx;
  const float* sw;
  float* out;
  char* ws[NR];
  int rank;
  int pad;
};
static_assert(sizeof(Params) == 0x70 && offsetof(Params, ws) == 0x28 && offsetof(Params, rank) == 0x68, "");

__device__ __forceinline__ __amdgpu_buffer_rsrc_t mk_rsrc(const gchar* p, unsigned bytes) {
  return __builtin_amdgcn_make_buffer_rsrc((void*)(u64)p, 0, bytes, 0x00020000);
}
__device__ __forceinline__ void st16(u32x4 v, __amdgpu_buffer_rsrc_t rs, unsigned off) {
  __builtin_amdgcn_raw_buffer_store_b128(v, rs, off, 0, AUX_SYS);
}
template <int AUX> __device__ __forceinline__ u32x4 ld16(__amdgpu_buffer_rsrc_t rs, unsigned voff, unsigned soff) {
  return __builtin_amdgcn_raw_buffer_load_b128(rs, voff, soff, AUX);
}
__device__ __forceinline__ unsigned ld4(__amdgpu_buffer_rsrc_t rs, unsigned voff, unsigned soff) {
  return __builtin_amdgcn_raw_buffer_load_b32(rs, voff, soff, AUX_SYS);
}
template <int N> __device__ __forceinline__ void wait_vm() {
  asm volatile("s_waitcnt vmcnt(%0)" :: "n"(N) : "memory");
}
__device__ __forceinline__ gchar* sel8(int k, u64 a0, u64 a1, u64 a2, u64 a3, u64 a4, u64 a5, u64 a6, u64 a7) {
  const u64 b0 = (k & 1) ? a1 : a0, b1 = (k & 1) ? a3 : a2, b2 = (k & 1) ? a5 : a4, b3 = (k & 1) ? a7 : a6;
  const u64 c0 = (k & 2) ? b1 : b0, c1 = (k & 2) ? b3 : b2;
  return (gchar*)((k & 4) ? c1 : c0);
}
__device__ __forceinline__ unsigned rfl(unsigned v) { return __builtin_amdgcn_readfirstlane(v); }
__device__ __forceinline__ u32x4 nonzero(u32x4 v) {
#pragma unroll
  for (int e = 0; e < 4; ++e) v[e] = v[e] ? v[e] : 0x80u;
  return v;
}
__device__ __forceinline__ unsigned min4(u32x4 v) { return min(min(v[0], v[1]), min(v[2], v[3])); }
__device__ __forceinline__ unsigned min8(u32x8 v) {
  return min(min(min(v[0], v[1]), min(v[2], v[3])), min(min(v[4], v[5]), min(v[6], v[7])));
}
__device__ __forceinline__ void sleep_until(unsigned long long t0, unsigned ticks) {
  while (__builtin_amdgcn_s_memrealtime() - t0 < ticks) __builtin_amdgcn_s_sleep(2);
}
__device__ __forceinline__ bool expired(gu32* tmo, unsigned call, unsigned long long t0) {
  if (rfl(__hip_atomic_load(tmo, __ATOMIC_RELAXED, __HIP_MEMORY_SCOPE_AGENT)) == call ||
      __builtin_amdgcn_s_memrealtime() - t0 > DEADLINE_TICKS) {
    __hip_atomic_store(tmo, call, __ATOMIC_RELAXED, __HIP_MEMORY_SCOPE_AGENT);
    return true;
  }
  return false;
}

__global__ void __launch_bounds__(1024, 4) gemm_rs_kernel(Params) {
  __shared__ __attribute__((aligned(16))) unsigned char lds[LDS_CTL + 384];
  u64x8 ka;
  u64x4 kb_;
  u64x2 kc;
  asm volatile("s_load_dwordx16 %0, %3, 0x0\n\ts_load_dwordx8 %1, %3, 0x40\n\ts_load_dwordx4 %2, %3, 0x60"
               : "=&s"(ka), "=&s"(kb_), "=&s"(kc) : "s"(__builtin_amdgcn_kernarg_segment_ptr()));
  const int tid = threadIdx.x;
  const int lane = tid & 63;
  const int wave = __builtin_amdgcn_readfirstlane(tid >> 6);
  const int t = blockIdx.x;
  const bool comm = t < NCOMM;
  const int er = lane >> 3, ec = lane & 7;
  const int e = t - NCOMM;
  const bool isx = wave < 8;
  if (isx) __builtin_amdgcn_s_setprio(2);
  const int ek = 1 + (unsigned)e % (NR - 1), ex = (unsigned)e / (NR - 1) * 8 + wave;
  auto w_off = [&](int ew) -> unsigned {
    return ((ew >> 6) * 128 + ((ew & 1) * 4 + ((wave >> 1) & 3)) * 16 + (wave & 1) * 8 + er) * N_ + (ew & 62) * 64 + ec * 16;
  };
  auto w_img = [&](int ew) -> unsigned {
    return OFF_WE + (ew & ~1) * WPIECE + ((ew & 1) * 4 + ((wave >> 1) & 3)) * 1024 + ((wave ^ (wave >> 1)) & 1) * 128;
  };
  unsigned e_off = isx ? (ek * MS + (ex & 63) * 8 + er) * KS + (ex >> 6) * 128 + ec * 16 : w_off(e);
  unsigned c_off = OFF_CNT + t * 128;
  asm volatile("" : "+v"(e_off), "+v"(c_off));
  asm volatile("s_waitcnt lgkmcnt(0)" : "+s"(ka), "+s"(kb_), "+s"(kc));
  const u64 ax = ka[0], aw_ = ka[1], asx = ka[2], asw = ka[3], aout = ka[4];
  const u64 w0 = ka[5], w1 = ka[6], w2 = ka[7], w3 = kb_[0], w4 = kb_[1], w5 = kb_[2], w6 = kb_[3], w7 = kc[0];
  const int rank = (int)kc[1];
  u32x4 ev;
  if (!comm) ev = *(const GLB_AS u32x4*)((const gchar*)(isx ? ax : aw_) + ((e_off + (isx ? rank * (MS * KS) : 0)) & (M_ * KS - 1)));
  const unsigned cnt_raw = __hip_atomic_load((gu32*)((gchar*)w0 + c_off), __ATOMIC_RELAXED, __HIP_MEMORY_SCOPE_AGENT);
#define ws_of(k) sel8((k), w0, w1, w2, w3, w4, w5, w6, w7)
  unsigned warm = 0;
  if (comm && isx)
    warm = ld4(mk_rsrc(ws_of(wave), OFF_CNT), wave ? (lane ? 0x80000000u : OFF_XE + rank * XPEER + t * 128)
                                                   : (lane < NR ? OFF_XE + lane * XPEER + t * 128 : 0x80000000u), 0);
  const gchar* const px = (const gchar*)ax;
  const gchar* const pw = (const gchar*)aw_;
  gchar* const myws = (gchar*)w0;
  gu32* const cnt = (gu32*)(myws + OFF_CNT + t * 128);
  gu32* const tmo = (gu32*)(myws + OFF_TMO);
  const int fr = lane & 15, fg = lane >> 4;
  const int xs = (rank + wave) & (NR - 1);
  const bool push = isx && ex < XPUSH;
  const unsigned long long t0 = __builtin_amdgcn_s_memrealtime();
  const auto my_rs = mk_rsrc(myws, OFF_CNT);
  const unsigned e_lane = (ec >> 2) * (isx ? 1024 : WPIECE) + (ec & 3) * 256 + er * 16;
  unsigned e_img = e_lane + (isx ? (ex >> 1) * XFRAG + (ex & 1) * 128 + (push ? OFF_X + rank * XPEER : OFF_XE + ((rank + ek) & (NR - 1)) * XPEER)
                                 : w_img(e));
  u64 e_base = (u64)ws_of(push ? ek : 0);
  asm volatile("" : "+s"(e_base));
  asm volatile("" : "+v"(e_img));
  const unsigned call = rfl(cnt_raw) + 1;
  if (!comm) {
    st16(nonzero(ev), mk_rsrc((gchar*)e_base, OFF_CNT), e_img + (call & 3) * (isx ? XSRC : WSRC));
    if (!isx && e < NCOMM) {
      const u32x4 ev2 = *(const GLB_AS u32x4*)(pw + w_off(GRID - NCOMM + e));
      st16(nonzero(ev2), my_rs, e_lane + w_img(GRID - NCOMM + e) + (call & 3) * WSRC);
    }
  }
  __builtin_amdgcn_s_setprio(0);

  const int xcd = t & 7, idx = (t - NCOMM) >> 3;
  const int ti = idx & 3, tu = idx >> 2;
  const int tl = 8 + tu * 8 + xcd;
  const int joff = (rank & 3) * 16 + (rank >> 2) * 8;
  const int tj = (tl + joff) & (NJ - 1);
  const int tx = (xcd + joff) & (NJ - 1);
  const bool extra = tu < 4;
  const bool win = tl >= NJ - WIN;
  const int kbs = tu == 4 ? 2 : rank >> 2;
  auto kb_of = [&](int r) { return !win ? r : r == NKB - 1 ? kbs : r + (r >= kbs); };
  volatile LDS_AS unsigned* ctl = (volatile LDS_AS unsigned*)(lds + LDS_CTL);
  volatile LDS_AS unsigned* ready = ctl;
  volatile LDS_AS unsigned* abortf = ctl + 8;
  volatile LDS_AS unsigned* invdone = ctl + 9;
  volatile LDS_AS unsigned* ownrdy = ctl + 12;
  volatile LDS_AS unsigned* consumed = ctl + 16;
  volatile LDS_AS unsigned* sched = ctl + 80;

  if (wave == 8) {
    if (lane < 16) ctl[lane] = 0;
    consumed[lane] = 0;
    asm volatile("s_waitcnt lgkmcnt(0)" ::: "memory");
  }
  asm volatile("" :: "s"(call));
  __builtin_amdgcn_s_barrier();
  if (tid == 0) __hip_atomic_store(cnt, call, __ATOMIC_RELAXED, __HIP_MEMORY_SCOPE_AGENT);
  auto rezero3 = [&]() {
    const u32x4 z = {0u, 0u, 0u, 0u};
    st16(z, my_rs, OFF_X + ((call + 2) & 3) * XSRC + xs * XPEER + t * 1024 + lane * 16);
    st16(z, my_rs, OFF_XE + ((call + 2) & 3) * XSRC + xs * XPEER + t * 1024 + lane * 16);
    st16(z, my_rs, OFF_WE + ((call + 2) & 3) * WSRC + t * WPIECE + tid * 16);
  };

  if (comm) {
    if (wave < 8) {
      asm volatile("" :: "v"(warm));
      if (wave < 2) {
        const int mk = 1 + min(t * PWAVES + wave * (PWAVES - 1), NPULL * (NR - 1) - 1) / NPULL;
        const unsigned b0 = OFF_X + (call & 3) * XSRC + ((rank + mk) & (NR - 1)) * XPEER;
        const unsigned se = (((lane & (MONN - 1)) * 64 / MONN + t % (64 / MONN)) * XPUSH / 64);
        for (unsigned n = 1;; ++n) {
          const unsigned f = ld4(my_rs, b0 + (se >> 1) * XFRAG + (se & 1) * 128, 0);
          if (__builtin_popcountll(__builtin_amdgcn_ballot_w64(f != 0)) >= ARRC) break;
          if ((n & 63) == 0 && expired(tmo, call, t0)) break;
        }
        if (lane == 0) ctl[mk] = 1;
      }
      __builtin_amdgcn_s_barrier();
      rezero3();
      if (t == 0 && wave == 0 && lane < 2 * NR)
        __hip_atomic_store((gu32*)(myws + OFF_TKT + ((call + 2) & 3) * 8 * 128 + (lane < NR ? lane * 128 : (lane - NR) * 4)),
                           0u, __ATOMIC_RELAXED, __HIP_MEMORY_SCOPE_AGENT);
      return;
    }
    const int pwv = wave - 8;
    const int g = t * PWAVES + pwv;
    if (pwv >= PWAVES || g >= NPULL * (NR - 1)) return;
    const int k = 1 + g / NPULL, q = g % NPULL;
    const int s = (rank + k) & (NR - 1);
    const u64 sp = (u64)ws_of(k);
    const u32x4 srd_s = {rfl((unsigned)sp), rfl((unsigned)(sp >> 32) & 0xffffu), OFF_CNT, 0x00020000u};
    const u32x4 srd_d = {rfl((unsigned)w0), rfl((unsigned)(w0 >> 32) & 0xffffu), OFF_CNT, 0x00020000u};
    constexpr unsigned INV = 1u << 20;
    u64 ktab = 0;
#pragma unroll
    for (int b = 0; b < 32 - WIN / 8; ++b) {
      const int r = b >> 3, lh = b & 7;
      ktab |= (u64)(lh >= 6 ? r + (r >= (rank >> 2)) : lh * 8 >= NJ - WIN ? r + (r >= 2) : r) << (2 * b);
    }
    unsigned status = 0;
    const unsigned sx0 = OFF_XE + (call & 3) * XSRC + rank * XPEER + XPUSH * 1024, sw0 = OFF_WE + (call & 3) * WSRC;
    const unsigned dx0 = OFF_X + (call & 3) * XSRC + s * XPEER + XPUSH * 1024, dw0 = OFF_W + (call & 1) * WSTAGE + s * WSRC;
    const unsigned tkt = OFF_TKT + ((call & 3) * 8 + k) * 128;
    while (!rfl(ctl[k])) __builtin_amdgcn_s_sleep(1);
    {
      unsigned cur, nxt, s1, s2;
      u64 tnow;
#define VERIFY(N, X0, X1, X2, X3, L) \
      "s_cmp_lt_u32 %[cur], %[npair]\n\t" \
      "s_cbranch_scc0 .Lok" L "_%=\n\t" \
      "s_waitcnt vmcnt(" N ")\n\t" \
      ".Lchk" L "_%=:\n\t" \
      "v_min_u32 v121, " X2 ", " X3 "\n\t" \
      "v_min3_u32 v121, " X0 ", " X1 ", v121\n\t" \
      "v_cmp_eq_u32 vcc, 0, v121\n\t" \
      "s_nop 0\n\t" \
      "s_cbranch_vccnz .Lretry" L "_%=\n\t" \
      ".Lok" L "_%=:\n\t"
#define RETRY(X, OFS, L) \
      ".Lretry" L "_%=:\n\t" \
      "s_sleep 8\n\t" \
      "s_memrealtime %[tnow]\n\t" \
      "s_waitcnt lgkmcnt(0)\n\t" \
      "v_mov_b64 v[122:123], %[tnow]\n\t" \
      "v_cmp_lt_u64 vcc, %[tend], v[122:123]\n\t" \
      "s_nop 0\n\t" \
      "s_cbranch_vccnz .Lfail_%=\n\t" \
      "buffer_load_dwordx4 " X ", v119, %[srs], 0 offen " OFS " sc0 sc1\n\t" \
      "s_waitcnt vmcnt(0)\n\t" \
      "s_branch .Lchk" L "_%=\n\t"
#define HALF(XA, XA0, XA1, XA2, XA3, XB, XB0, XB1, XB2, XB3, YA, YB, TIN, TOUT, LA, LB) \
      VERIFY("3", XA0, XA1, XA2, XA3, LA) \
      "v_readfirstlane_b32 %[nxt], " TIN "\n\t" \
      "s_mov_b64 exec, 1\n\t" \
      "global_atomic_add " TOUT ", %[tkt], %[inc], %[ws] sc0\n\t" \
      "s_mov_b64 exec, -1\n\t" \
      "s_add_u32 %[nxt], %[nxt], %[base]\n\t" \
      "s_sub_u32 %[s1], %[nxt], %[xpair]\n\t" \
      "s_add_u32 %[s2], %[s1], %[rot]\n\t" \
      "s_and_b32 %[s2], %[s2], 31\n\t" \
      "s_add_u32 vcc_lo, %[s1], %[joff]\n\t" \
      "s_and_b32 vcc_lo, vcc_lo, 0xe0\n\t" \
      "s_or_b32 %[s2], %[s2], vcc_lo\n\t" \
      "s_lshr_b32 %[s1], %[s1], 5\n\t" \
      "s_lshl_b32 %[s1], %[s1], 1\n\t" \
      "s_lshr_b64 vcc, %[ktab], %[s1]\n\t" \
      "s_and_b32 %[s1], vcc_lo, 3\n\t" \
      "s_lshl_b32 %[s1], %[s1], 8\n\t" \
      "s_or_b32 %[s2], %[s2], %[s1]\n\t" \
      "s_cmp_lt_u32 %[nxt], %[xpair]\n\t" \
      "s_cselect_b32 %[s2], %[nxt], %[s2]\n\t" \
      "s_cselect_b32 %[s1], %[sx], %[sw]\n\t" \
      "s_lshl_b32 %[s2], %[s2], 11\n\t" \
      "s_add_u32 %[s1], %[s1], %[s2]\n\t" \
      "s_cmp_lt_u32 %[nxt], %[npair]\n\t" \
      "s_cselect_b32 %[s1], %[s1], 0x80000000\n\t" \
      "v_add_u32 v118, %[s1], %[lo]\n\t" \
      "buffer_load_dwordx4 " YA ", v118, %[srs], 0 offen sc0 sc1\n\t" \
      "buffer_store_dwordx4 " XA ", v120, %[srd], 0 offen sc0 sc1\n\t" \
      VERIFY("4", XB0, XB1, XB2, XB3, LB) \
      "buffer_load_dwordx4 " YB ", v118, %[srs], 0 offen offset:1024 sc0 sc1\n\t" \
      "buffer_store_dwordx4 " XB ", v120, %[srd], 0 offen offset:1024 sc0 sc1\n\t" \
      "s_cmp_lt_u32 %[nxt], %[xpair]\n\t" \
      "s_cselect_b32 %[s1], %[dx], %[dw]\n\t" \
      "s_add_u32 %[s1], %[s1], %[s2]\n\t" \
      "s_cmp_lt_u32 %[nxt], %[npair]\n\t" \
      "s_cselect_b32 %[s1], %[s1], 0x80000000\n\t" \
      "v_mov_b32 v119, v118\n\t" \
      "v_add_u32 v120, %[s1], %[lo]\n\t" \
      PSTAMP \
      "s_mov_b32 %[cur], %[nxt]\n\t"
#define PSTAMP
#define PSTAMP_OPS
      asm volatile(
          "s_mov_b32 %[cur], %[inv]\n\t"
          "v_mov_b32 v117, %[first]\n\t"
          "v_mov_b32 v120, 0x80000000\n\t"
          "s_nop 4\n\t"
          ".Lloop_%=:\n\t"
          HALF("v[100:103]", "v100", "v101", "v102", "v103", "v[104:107]", "v104", "v105", "v106", "v107",
               "v[108:111]", "v[112:115]", "v117", "v116", "a", "b")
          "s_cmp_ge_u32 %[cur], %[npair]\n\t"
          "s_cbranch_scc1 .Ldone_%=\n\t"
          HALF("v[108:111]", "v108", "v109", "v110", "v111", "v[112:115]", "v112", "v113", "v114", "v115",
               "v[100:103]", "v[104:107]", "v116", "v117", "c", "d")
          "s_cmp_lt_u32 %[cur], %[npair]\n\t"
          "s_cbranch_scc1 .Lloop_%=\n\t"
          "s_branch .Ldone_%=\n\t"
          RETRY("v[100:103]", "", "a")
          RETRY("v[104:107]", "offset:1024", "b")
          RETRY("v[108:111]", "", "c")
          RETRY("v[112:115]", "offset:1024", "d")
          ".Lfail_%=:\n\t"
          "s_mov_b32 %[status], 1\n\t"
          ".Ldone_%=:\n\t"
          "s_waitcnt vmcnt(0)"
          : [cur] "=&s"(cur), [nxt] "=&s"(nxt), [s1] "=&s"(s1), [s2] "=&s"(s2), [tnow] "=&s"(tnow), [status] "+s"(status) PSTAMP_OPS
          : [srs] "s"(srd_s), [srd] "s"(srd_d), [ws] "s"(w0), [tend] "s"(t0 + DEADLINE_TICKS),
            [sx] "s"(rfl(sx0)), [sw] "s"(rfl(sw0)), [dx] "s"(rfl(dx0)), [dw] "s"(rfl(dw0)),
            [base] "n"(NPULL), [first] "s"(rfl((unsigned)(q - NPULL))),
            [ktab] "s"(ktab), [rot] "s"(rfl(k * 4)), [joff] "s"(rfl(joff * 4)), [npair] "n"(NPAIR), [xpair] "n"(XU / 2), [inv] "n"(INV),
            [tkt] "v"(tkt), [inc] "v"(1u), [lo] "v"(lane * 16)
          : "memory", "vcc", "scc", "v100", "v101", "v102", "v103", "v104", "v105", "v106", "v107", "v108", "v109",
            "v110", "v111", "v112", "v113", "v114", "v115", "v116", "v117", "v118", "v119", "v120", "v121", "v122", "v123");
    }
    if (status) __hip_atomic_store(tmo, call, __ATOMIC_RELAXED, __HIP_MEMORY_SCOPE_AGENT);
    if (pwv < 8) ST(48 + pwv);
    return;
  }

  if (wave == 8) {
    sleep_until(t0, OWN0);
    const gchar* w_even = pw + (size_t)fr * N_ + fg * 16 + tj * 64;
    const gchar* w_odd = pw + (size_t)(fr ^ 8) * N_ + fg * 16 + tj * 64;
    const gchar* w_ex = pw + (size_t)(fg * 16 + (fr ^ ((fg & 1) << 3))) * N_ + tx * 64 + tu * 16;
    for (int kb = 0; kb < NKB; ++kb) {
#pragma unroll
      for (int v = 0; v < 8; ++v)
        __builtin_amdgcn_global_load_lds((const GLB_AS void*)(((v & 1) ? w_odd : w_even) + (size_t)(kb * 128 + v * 16) * N_),
                                         (LDS_AS void*)(lds + LDS_OWN + kb * SLOT + v * 1024), 16, 0, 0);
      if (extra) {
        __builtin_amdgcn_global_load_lds((const GLB_AS void*)(w_ex + (size_t)(kb * 128) * N_),
                                         (LDS_AS void*)(lds + LDS_OWN + kb * SLOT + 8192), 16, 0, 0);
        __builtin_amdgcn_global_load_lds((const GLB_AS void*)(w_ex + (size_t)(kb * 128 + 64) * N_),
                                         (LDS_AS void*)(lds + LDS_OWN + kb * SLOT + 9216), 16, 0, 0);
      }
      wait_vm<0>();
      if (lane == 0) ownrdy[kb] = 1;
    }
    gu32* const tk = (gu32*)(myws + OFF_TKT + (call & 3) * 8 * 128);
    if (win) {
      sleep_until(t0, 1000 + (GRID - 1 - t) * 15);
      __builtin_amdgcn_fence(__ATOMIC_ACQUIRE, "");
      wait_vm<0>();
      for (unsigned it = 1;; ++it) {
        const unsigned f = __hip_atomic_load(tk + (lane & 7), __ATOMIC_RELAXED, __HIP_MEMORY_SCOPE_AGENT);
        if (__builtin_amdgcn_ballot_w64(f == 0) == 0) {
          sched[lane & 7] = f;
          asm volatile("s_waitcnt lgkmcnt(0)" ::: "memory");
          if (lane == 0) *invdone = 1;
          break;
        }
        if ((it & 63) == 0 && expired(tmo, call, t0)) { if (lane == 0) *abortf = 1; break; }
        __builtin_amdgcn_s_sleep(8);
      }
    } else if (t == NCOMM + 3 * 32) {
      for (unsigned it = 1;; ++it) {
        const unsigned ta = (unsigned)__builtin_amdgcn_s_memrealtime();
        const unsigned f = __hip_atomic_load(tk + (1 + lane % 7) * 32, __ATOMIC_RELAXED, __HIP_MEMORY_SCOPE_AGENT);
        unsigned mx = 0;
#pragma unroll
        for (int k = 0; k < NR - 1; ++k) mx = max(mx, (unsigned)__builtin_amdgcn_readlane(f, k));
        const unsigned tb = (unsigned)__builtin_amdgcn_s_memrealtime();
        if (mx + SNAP >= NPAIR - NPULL && (tb - ta < 80 || mx + SNAP / 2 >= NPAIR - NPULL)) {
          const int left = max((int)(NPAIR - NPULL) - (int)mx, 0);
          const unsigned fmin = ta + 25 + left * SLOT_T / 40 + FOFF;
          const unsigned lag = __builtin_amdgcn_update_dpp(0u, f ? min((mx - f + 1) / 2, 80u) + 1 : 1000u, 0x111  , 0xf, 0xf, false);
          if (lane < NR) __hip_atomic_store(tk + lane, lane ? lag : fmin | 1u, __ATOMIC_RELAXED, __HIP_MEMORY_SCOPE_AGENT);
          break;
        }
        if ((it & 63) == 0 && expired(tmo, call, t0)) break;
        __builtin_amdgcn_s_sleep(8);
      }
    }
    return;
  }

  if (wave > 8) {
    const int sv = wave - 8;
    const int s = (rank + sv) & (NR - 1);
    const unsigned ex_lane = fg * 1024 + tu * 256 + fr * 16;
    sleep_until(t0, OWN0);
    const unsigned zo = OFF_W + ((call + 1) & 1) * WSTAGE + s * WSRC + ti * NJ * WPIECE;
    auto rezero = [&](int part) {
      const u32x4 z = {0u, 0u, 0u, 0u};
      st16(z, my_rs, zo + tj * WPIECE + lane * 16 + part * 3072);
      st16(z, my_rs, zo + tj * WPIECE + lane * 16 + part * 3072 + 1024);
      if (part < 2) {
        st16(z, my_rs, zo + tj * WPIECE + lane * 16 + part * 3072 + 2048);
        if (extra) st16(z, my_rs, zo + tx * WPIECE + ex_lane + part * 4096);
      }
    };
    LDS_AS unsigned char* slot = (LDS_AS unsigned char*)lds + (sv - 1) * SLOT + lane * 16;
    const unsigned sb = OFF_W + (call & 1) * WSTAGE + s * WSRC;
    auto slot_free = [&](unsigned kb) {
      while (__builtin_amdgcn_ballot_w64(consumed[sv * 8 + (lane & 7)] < kb) && !rfl(*abortf)) __builtin_amdgcn_s_sleep(2);
    };
    for (int kb = 0; kb < (win ? NKB - 1 : NKB); ++kb) {
      const unsigned so = sb + (kb_of(kb) * NJ + tj) * WPIECE, so_ex = sb + (kb * NJ + tx) * WPIECE;
      bool dead = false;
      for (unsigned it = 1;; ++it) {
        if (rfl(ld4(my_rs, 0, so)) != 0) break;
        if ((it & 63) == 0 && expired(tmo, call, t0)) { dead = true; break; }
        __builtin_amdgcn_s_sleep(2);
      }
      u32x4 d[10];
      for (unsigned it = 1; !dead; ++it) {
        unsigned m = 0xffffffffu;
#pragma unroll
        for (int v = 0; v < 8; ++v) d[v] = ld16<AUX_SYS>(my_rs, lane * 16 + v * 1024, so);
        if (extra) {
          d[8] = ld16<AUX_SYS>(my_rs, ex_lane, so_ex);
          d[9] = ld16<AUX_SYS>(my_rs, ex_lane + 4096, so_ex);
          m = min(min4(d[8]), min4(d[9]));
        }
#pragma unroll
        for (int v = 0; v < 8; ++v) m = min(m, min4(d[v]));
        if (__builtin_amdgcn_ballot_w64(m == 0) == 0) break;
        if ((it & 63) == 0 && expired(tmo, call, t0)) dead = true;
      }
      if (dead) { if (lane == 0) *abortf = 1; return; }
      slot_free(kb);
#pragma unroll
      for (int v = 0; v < 8; ++v) *(LDS_AS u32x4*)(slot + v * 1024) = d[v];
      if (extra) { *(LDS_AS u32x4*)(slot + 8192) = d[8]; *(LDS_AS u32x4*)(slot + 9216) = d[9]; }
      asm volatile("s_waitcnt lgkmcnt(0)" ::: "memory");
      if (lane == 0) ready[sv] = kb + 1;
      ST(4 + kb * 8 + sv);
      if (kb < NKB - 1) rezero(kb);
      if (kb == 1 && ti == 0) {
        const unsigned f = ld4(my_rs, lane * 128, sb + ((NKB - 1) * NJ + tj) * WPIECE);
        asm volatile("" :: "v"(f));
      }
    }
    if (!win) return;
    while (!rfl(*invdone) && !rfl(*abortf)) __builtin_amdgcn_s_sleep(2);
    if (rfl(*abortf)) return;
    const unsigned po = (kbs * NJ + tj) * WPIECE, eo = OFF_WE + (call & 3) * WSRC + po;
    const unsigned sc_l = sched[lane & 7];
    int c0 = 0, c1 = 0;
#pragma unroll
    for (int k = 1; k < NR; ++k) {
      const int ak = (int)__builtin_amdgcn_readlane(sc_l, k) - 1;
      c0 += max(lane - ak, 0);
      c1 += max(lane + 64 - ak, 0);
    }
    constexpr int NJOB = 2 * WIN;
    const int base = __builtin_popcountll(__builtin_amdgcn_ballot_w64(c0 < NJOB * (NR - 1))) +
                     __builtin_popcountll(__builtin_amdgcn_ballot_w64(c1 < NJOB * (NR - 1))) - NJOB;
    int shed = 0;
#pragma unroll
    for (int k = 1; k < NR - 1; ++k)
      if (k < sv) shed += min(max((int)__builtin_amdgcn_readlane(sc_l, k) - 1 - base, 0), NJOB);
    unsigned fire[2], voff[2];
    u64 via[2];
#pragma unroll
    for (int h = 0; h < 2; ++h) {
      const int i = h * WIN + (unsigned)(tl - sv * (WIN / 8)) % WIN;
      int row = base + i, lk = sv;
      if (ORIGIN == 0 || row < (int)__builtin_amdgcn_readlane(sc_l, sv) - 1) {
        const int J = ORIGIN ? shed + i : i * (NR - 1) + sv - 1;
        row = __builtin_popcountll(__builtin_amdgcn_ballot_w64(c0 <= J)) + __builtin_popcountll(__builtin_amdgcn_ballot_w64(c1 <= J)) - 1;
        const int p = J - (int)(row < 64 ? __builtin_amdgcn_readlane(c0, row & 63) : __builtin_amdgcn_readlane(c1, row & 63));
        const unsigned act = (unsigned)__builtin_amdgcn_ballot_w64((int)sc_l - 1 <= row) & 0xfeu;
#pragma unroll
        for (int k = NR - 1; k >= 1; --k)
          if (((act >> k) & 1u) && __builtin_popcount(act & ((1u << k) - 1u)) == p) lk = k;
      }
      fire[h] = (unsigned)__builtin_amdgcn_readlane(sc_l, 0) + row * SLOT_T / 20;
      via[h] = (u64)ws_of(lk);
      voff[h] = (lk == sv ? eo : sb + po) + h * 512;
      asm volatile("" : "+s"(fire[h]), "+s"(via[h]), "+s"(voff[h]));
    }
    slot_free(NKB - 1);
    const unsigned hl = (lane >> 5) * 1024 + (lane & 31) * 16;
    LDS_AS unsigned char* hslot = (LDS_AS unsigned char*)lds + (sv - 1) * SLOT + hl;
#pragma nounroll
    for (unsigned h = 0; h < 2; ++h) {
      asm volatile("" : "+s"(h));
      const unsigned fi = h ? fire[1] : fire[0], vo = h ? voff[1] : voff[0];
      const auto vrs = mk_rsrc((gchar*)(h ? via[1] : via[0]), OFF_CNT);
      while ((int)(fi - (unsigned)__builtin_amdgcn_s_memrealtime()) > 0) __builtin_amdgcn_s_sleep(1);
      u32x4 d[4];
#pragma unroll
      for (int v = 0; v < 4; ++v) d[v] = ld16<0>(vrs, hl + v * 2048, vo);
      unsigned m = 0xffffffffu;
#pragma unroll
      for (int v = 0; v < 4; ++v) {
        *(LDS_AS u32x4*)(hslot + h * 512 + v * 2048) = d[v];
        m = min(m, min4(d[v]));
        __builtin_amdgcn_sched_barrier(0);
      }
      for (unsigned it = 1; __builtin_expect(__builtin_amdgcn_ballot_w64(m == 0) != 0, 0); ++it) {
        if ((it & 63) == 0 && expired(tmo, call, t0)) { if (lane == 0) *abortf = 1; return; }
        m = 0xffffffffu;
#pragma unroll
        for (int v = 0; v < 4; ++v) {
          const u32x4 r = ld16<AUX_SYS>(mk_rsrc(ws_of(sv), OFF_CNT), hl + v * 2048, eo + h * 512);
          *(LDS_AS u32x4*)(hslot + h * 512 + v * 2048) = r;
          m = min(m, min4(r));
        }
      }
      asm volatile("s_waitcnt lgkmcnt(0)" ::: "memory");
      if (lane == 0) ready[sv] = NKB + h;
    }
    ST(4 + 24 + sv);
    return;
  }

  sleep_until(t0, OWN0);
  const int mb = ti * 8 + wave;
  const auto x_own_rs = mk_rsrc(px, M_ * KS);
  const unsigned x_own_lane = (unsigned)(rank * MS + mb * 16 + fr) * KS + fg * 16;
  const unsigned x_st_lane = OFF_X + (call & 3) * XSRC + mb * XFRAG + lane * 16;
  float sc = __builtin_bit_cast(float, rfl(__builtin_bit_cast(unsigned, *(const GLB_AS float*)asx * *(const GLB_AS float*)asw)));
  asm volatile("" : "+s"(sc));
  auto ldx_own = [&](unsigned kb) {
    const u32x4 lo = ld16<0>(x_own_rs, x_own_lane, kb * 128);
    const u32x4 hi = ld16<0>(x_own_rs, x_own_lane + 64, kb * 128);
    return __builtin_shufflevector(lo, hi, 0, 1, 2, 3, 4, 5, 6, 7);
  };
  auto ldx_st = [&](int sv, unsigned kb) {
    const unsigned so = ((rank + sv) & (NR - 1)) * XPEER + kb * 32 * XFRAG;
    const u32x4 lo = ld16<AUX_SYS>(my_rs, x_st_lane, so);
    const u32x4 hi = ld16<AUX_SYS>(my_rs, x_st_lane + 1024, so);
    return __builtin_shufflevector(lo, hi, 0, 1, 2, 3, 4, 5, 6, 7);
  };
  const unsigned tr_lane = fg * 1024 + ((fg & 1) << 7) + fr * 8;

  f32x4 acc[5] = {};
  bool failed = false;
  auto step = [&](unsigned base, u32x8 xv) {
    const i32x8 bx = __builtin_bit_cast(i32x8, xv);
    unsigned tl = tr_lane;
    asm volatile("" : "+v"(tl));
    LDS_AS unsigned char* sl = (LDS_AS unsigned char*)lds + base;
    if (extra) {
      i32x8 aw;
      const unsigned te = tl - fg * 768 + 8192;
#pragma unroll
      for (int c = 0; c < 4; ++c) {
        const i32x2 r = __builtin_amdgcn_ds_read_tr8_b64_v2i32((LDS_AS i32x2*)(sl + (c >> 1) * 1024 + (te ^ ((c & 1) << 7))));
        aw[2 * c] = r[0]; aw[2 * c + 1] = r[1];
      }
      acc[4] = __builtin_amdgcn_mfma_scale_f32_16x16x128_f8f6f4(aw, bx, acc[4], 0, 0, 0, 0x7f7f7f7f, 0, 0x7f7f7f7f);
    }
#pragma unroll
    for (int q = 0; q < 4; ++q) {
      i32x8 aw;
#pragma unroll
      for (int c = 0; c < 4; ++c) {
        const i32x2 r = __builtin_amdgcn_ds_read_tr8_b64_v2i32((LDS_AS i32x2*)(sl + (c >> 1) * 4096 + q * 256 + (tl ^ ((c & 1) << 7))));
        aw[2 * c] = r[0]; aw[2 * c + 1] = r[1];
      }
      acc[q] = __builtin_amdgcn_mfma_scale_f32_16x16x128_f8f6f4(aw, bx, acc[q], 0, 0, 0, 0x7f7f7f7f, 0, 0x7f7f7f7f);
    }
  };
#pragma unroll
  for (unsigned kb = 0; kb < NKB; ++kb) {
    const u32x8 xv = ldx_own(kb);
    while (!rfl(ownrdy[kb])) __builtin_amdgcn_s_sleep(1);
    step(LDS_OWN + kb * SLOT, xv);
  }
  while (!((unsigned)__builtin_amdgcn_ballot_w64(ctl[lane & 15] != 0u) & 0x1feu)) __builtin_amdgcn_s_sleep(4);
  rezero3();
  u32x8 xf[NR];
  auto load_x = [&](unsigned kb) {
    for (unsigned it = 1;; ++it) {
      unsigned m = 0xffffffffu;
#pragma unroll
      for (int sv = 1; sv < NR; ++sv) xf[sv] = ldx_st(sv, kb);
#pragma unroll
      for (int sv = 1; sv < NR; ++sv) m = min(m, min8(xf[sv]));
      if (__builtin_amdgcn_ballot_w64(m == 0) == 0) break;
      if ((it & 15) == 0 && expired(tmo, call, t0)) { failed = true; break; }
      __builtin_amdgcn_s_sleep(32);
    }
  };
  auto next_ready = [&](unsigned& pend, unsigned level) {
    unsigned rdy;
    for (;;) {
      const unsigned rv = ctl[lane & 15];
      rdy = (unsigned)__builtin_amdgcn_ballot_w64(rv > ((lane & 15) < 8 ? level : 0u)) & (pend | 0x100u);
      if (rdy) break;
      __builtin_amdgcn_s_sleep(1);
    }
    pend &= ~rdy;
    if (rdy & 0x100u) { failed = true; pend = 0; rdy = 0; }
    return rdy;
  };
  for (unsigned kb = 0; kb < (win ? NKB - 1 : NKB) && !failed; ++kb) {
    load_x(kb_of(kb));
    for (unsigned pend = failed ? 0u : 0xfeu; pend;) {
      const unsigned rdy = next_ready(pend, kb);
#pragma unroll
      for (int sv = 1; sv < NR; ++sv)
        if (rdy & (1u << sv)) {
          step((sv - 1) * SLOT, xf[sv]);
          asm volatile("s_waitcnt lgkmcnt(0)" ::: "memory");
          consumed[sv * 8 + wave] = kb + 1;
        }
    }
  }
  if (win && !failed) load_x(kbs);
  const auto o_rs = mk_rsrc((gchar*)aout, MS * N_ * 4);
  constexpr float LOG2E = 1.4426950408889634f;
  auto silu4 = [&](f32x4 a) {
    const f32x4 y = a * (failed ? __builtin_nanf("") : sc);
    f32x4 z = y * -LOG2E;
#pragma unroll
    for (int e = 0; e < 4; ++e) z[e] = __builtin_amdgcn_exp2f(__builtin_amdgcn_fmed3f(z[e], -60.0f * LOG2E, 60.0f * LOG2E));
    z += 1.0f;
#pragma unroll
    for (int e = 0; e < 4; ++e) z[e] = __builtin_amdgcn_rcpf(z[e]);
    return __builtin_bit_cast(u32x4, y * z);
  };
  if (extra) st16(silu4(acc[4]), o_rs, ((unsigned)(mb * 16 + fr) * N_ + fg * 4 + tx * 64 + tu * 16) * 4);
#pragma nounroll
  for (unsigned h = 0; h < 2; ++h) {
    asm volatile("" : "+s"(h));
    for (unsigned pend = win && !failed ? 0xfeu : 0u; pend;) {
      const unsigned rdy = next_ready(pend, NKB - 1 + h);
#pragma unroll
      for (int sv = 1; sv < NR; ++sv)
        if (rdy & (1u << sv)) {
          const i32x8 bx = __builtin_bit_cast(i32x8, xf[sv]);
          unsigned tl2 = tr_lane + h * 512;
          asm volatile("" : "+v"(tl2));
          LDS_AS unsigned char* sl = (LDS_AS unsigned char*)lds + (sv - 1) * SLOT;
#pragma unroll
          for (int q = 0; q < 2; ++q) {
            i32x8 aw;
#pragma unroll
            for (int c = 0; c < 4; ++c) {
              const i32x2 r = __builtin_amdgcn_ds_read_tr8_b64_v2i32((LDS_AS i32x2*)(sl + (c >> 1) * 4096 + q * 256 + (tl2 ^ ((c & 1) << 7))));
              aw[2 * c] = r[0]; aw[2 * c + 1] = r[1];
            }
            acc[q] = __builtin_amdgcn_mfma_scale_f32_16x16x128_f8f6f4(aw, bx, acc[q], 0, 0, 0, 0x7f7f7f7f, 0, 0x7f7f7f7f);
          }
        }
    }
    if (failed && lane == 0) *abortf = 1;
    STW0(2);
    const unsigned o_pair = ((unsigned)(mb * 16 + (fr & 7)) * N_ + (fr >> 3) * 16 + fg * 4 + tj * 64 + h * 32) * 4;
    const u32x4 o0 = silu4(acc[0]), o1 = silu4(acc[1]);
    u32x4 lo, hi;
#pragma unroll
    for (int e = 0; e < 4; ++e) {
      lo[e] = __builtin_amdgcn_update_dpp(o0[e], o1[e], 0x128  , 0xf, 0xc, false);
      hi[e] = __builtin_amdgcn_update_dpp(o1[e], o0[e], 0x128  , 0xf, 0x3, false);
    }
    st16(lo, o_rs, o_pair);
    st16(hi, o_rs, o_pair + 8 * N_ * 4);
    acc[0] = acc[2];
    acc[1] = acc[3];
  }
}

}

extern "C" void kernel_launch(int rank, int nranks, ncclComm_t comm,
                              hipStream_t stream,
                              void* const* d_in, const int* in_sizes, int n_in,
                              void* d_out, int out_size,
                              const int* in_dtypes, int out_dtype,
                              void* const* d_out_all,
                              void* const* d_ws_all, size_t ws_bytes) {
  if (nranks != NR || (unsigned)rank >= (unsigned)NR || ws_bytes < WS_NEED || n_in < 4) {
    fprintf(stderr, "kernel_launch: unexpected configuration\n");
    return;
  }
  Params p;
  p.x = (const uint8_t*)d_in[0];
  p.w = (const uint8_t*)d_in[1];
  p.sx = (const float*)d_in[2];
  p.sw = (const float*)d_in[3];
  p.out = (float*)d_out;
  for (int i = 0; i < NR; ++i) p.ws[i] = (char*)d_ws_all[(rank + i) & (NR - 1)];
  p.rank = rank;
  p.pad = 0;
  hipLaunchKernelGGL(gemm_rs_kernel, dim3(GRID), dim3(1024), 0, stream, p);
}
```

```cpp
#include <hip/hip_runtime.h>
#include <rccl/rccl.h>
#include <cstdio>
#include <cstdint>
#include <cstddef>

namespace {

constexpr int NR = 8;
constexpr int M_ = 4096, K_ = 4096, N_ = 4096;
constexpr int MS = M_ / NR;
constexpr int KS = K_ / NR;
constexpr int NKB = KS / 128;
constexpr int NJ = N_ / 64;
constexpr int XPUSH = 120;
constexpr int ARRC = 32;
constexpr int OWN0 = 200;
constexpr int WIN = 24;
constexpr int SLOT_T = 134;
constexpr int SNAP = 130;
constexpr int FOFF = 130;
constexpr int GRID = 256;
constexpr int NCOMM = 32;
constexpr int NPULL = 36;
constexpr int PWAVES = (NPULL * (NR - 1) + NCOMM - 1) / NCOMM;
constexpr int XU = 256 - XPUSH;
constexpr int NUNIT = XU + (NKB * NJ - WIN) * 8;
constexpr unsigned NPAIR = NUNIT / 2;
static_assert(XU % 2 == 0 && XPUSH <= GRID && PWAVES <= 8, "");

constexpr unsigned WPIECE = 8192;
constexpr unsigned WSRC = NKB * NJ * WPIECE;
constexpr unsigned WSTAGE = NR * WSRC;
constexpr unsigned XFRAG = 2048;
constexpr unsigned XSRC = NKB * 32 * XFRAG;
constexpr unsigned XPEER = 8 * XSRC;
constexpr unsigned OFF_W = 0;
constexpr unsigned OFF_X = OFF_W + 2 * WSTAGE;
constexpr unsigned OFF_XE = OFF_X + 4 * XSRC;
constexpr unsigned OFF_WE = OFF_X + NR * XPEER;
constexpr unsigned OFF_CNT = OFF_WE + 4 * WSRC;
constexpr unsigned OFF_TMO = OFF_CNT + GRID * 128;
constexpr unsigned OFF_TKT = OFF_TMO + 128;
constexpr unsigned WS_NEED = OFF_TKT + 32 * 128;

constexpr unsigned SLOT = WPIECE + 2048;
constexpr unsigned LDS_OWN = 7 * SLOT;
constexpr unsigned LDS_CTL = LDS_OWN + NKB * SLOT;

constexpr unsigned long long DEADLINE_TICKS = 50000000ull;

typedef unsigned u32x4 __attribute__((ext_vector_type(4)));
typedef unsigned u32x8 __attribute__((ext_vector_type(8)));
typedef int i32x2 __attribute__((ext_vector_type(2)));
typedef int i32x8 __attribute__((ext_vector_type(8)));
typedef float f32x4 __attribute__((ext_vector_type(4)));
#define LDS_AS __attribute__((address_space(3)))
#define GLB_AS __attribute__((address_space(1)))
typedef unsigned long long u64;
typedef u64 u64x8 __attribute__((ext_vector_type(8)));
typedef u64 u64x4 __attribute__((ext_vector_type(4)));
typedef u64 u64x2 __attribute__((ext_vector_type(2)));
typedef GLB_AS char gchar;
typedef GLB_AS unsigned gu32;
constexpr int AUX_SYS = 17;

struct Params {
  const uint8_t* x;
  const uint8_t* w;
  const float* sx;
  const float* sw;
  float* out;
  char* ws[NR];
  int rank;
  int pad;
};
static_assert(sizeof(Params) == 0x70 && offsetof(Params, ws) == 0x28 && offsetof(Params, rank) == 0x68, "");

__device__ __forceinline__ __amdgpu_buffer_rsrc_t mk_rsrc(const gchar* p, unsigned bytes) {
  return __builtin_amdgcn_make_buffer_rsrc((void*)(u64)p, 0, bytes, 0x00020000);
}
__device__ __forceinline__ void st16(u32x4 v, __amdgpu_buffer_rsrc_t rs, unsigned off) {
  __builtin_amdgcn_raw_buffer_store_b128(v, rs, off, 0, AUX_SYS);
}
template <int AUX> __device__ __forceinline__ u32x4 ld16(__amdgpu_buffer_rsrc_t rs, unsigned voff, unsigned soff) {
  return __builtin_amdgcn_raw_buffer_load_b128(rs, voff, soff, AUX);
}
__device__ __forceinline__ unsigned ld4(__amdgpu_buffer_rsrc_t rs, unsigned voff, unsigned soff) {
  asm volatile("" ::: "memory");
  return __builtin_amdgcn_raw_buffer_load_b32(rs, voff, soff, AUX_SYS);
}
template <int N> __device__ __forceinline__ void wait_vm() {
  asm volatile("s_waitcnt vmcnt(%0)" :: "n"(N) : "memory");
}
__device__ __forceinline__ gchar* sel8(int k, u64 a0, u64 a1, u64 a2, u64 a3, u64 a4, u64 a5, u64 a6, u64 a7) {
  const u64 b0 = (k & 1) ? a1 : a0, b1 = (k & 1) ? a3 : a2, b2 = (k & 1) ? a5 : a4, b3 = (k & 1) ? a7 : a6;
  const u64 c0 = (k & 2) ? b1 : b0, c1 = (k & 2) ? b3 : b2;
  return (gchar*)((k & 4) ? c1 : c0);
}
__device__ __forceinline__ unsigned rfl(unsigned v) { return __builtin_amdgcn_readfirstlane(v); }
__device__ __forceinline__ u32x4 nonzero(u32x4 v) {
#pragma unroll
  for (int e = 0; e < 4; ++e) v[e] = v[e] ? v[e] : 0x80u;
  return v;
}
__device__ __forceinline__ unsigned min4(u32x4 v) { return min(min(v[0], v[1]), min(v[2], v[3])); }
__device__ __forceinline__ unsigned min8(u32x8 v) {
  return min(min(min(v[0], v[1]), min(v[2], v[3])), min(min(v[4], v[5]), min(v[6], v[7])));
}
__device__ __forceinline__ void sleep_until(unsigned long long t0, unsigned ticks) {
  while (__builtin_amdgcn_s_memrealtime() - t0 < ticks) __builtin_amdgcn_s_sleep(2);
}
__device__ __forceinline__ bool expired(gu32* tmo, unsigned call, unsigned long long t0) {
  if (rfl(__hip_atomic_load(tmo, __ATOMIC_RELAXED, __HIP_MEMORY_SCOPE_AGENT)) == call ||
      __builtin_amdgcn_s_memrealtime() - t0 > DEADLINE_TICKS) {
    __hip_atomic_store(tmo, call, __ATOMIC_RELAXED, __HIP_MEMORY_SCOPE_AGENT);
    return true;
  }
  return false;
}

__global__ void __launch_bounds__(1024, 4) gemm_rs_kernel(Params) {
  __shared__ __attribute__((aligned(16))) unsigned char lds[LDS_CTL + 384];
  u64x8 ka;
  u64x4 kb_;
  u64x2 kc;
  asm volatile("s_load_dwordx16 %0, %3, 0x0\n\ts_load_dwordx8 %1, %3, 0x40\n\ts_load_dwordx4 %2, %3, 0x60"
               : "=&s"(ka), "=&s"(kb_), "=&s"(kc) : "s"(__builtin_amdgcn_kernarg_segment_ptr()));
  const int tid = threadIdx.x;
  const int lane = tid & 63;
  const int wave = __builtin_amdgcn_readfirstlane(tid >> 6);
  const int t = blockIdx.x;
  const bool comm = t < NCOMM;
  const int er = lane >> 3, ec = lane & 7;
  const int e = t - NCOMM;
  const bool isx = wave < 8;
  if (isx) __builtin_amdgcn_s_setprio(2);
  const int ek = 1 + (unsigned)e % (NR - 1), ex = (unsigned)e / (NR - 1) * 8 + wave;
  auto w_off = [&](int ew) -> unsigned {
    return ((ew >> 6) * 128 + ((ew & 1) * 4 + ((wave >> 1) & 3)) * 16 + (wave & 1) * 8 + er) * N_ + (ew & 62) * 64 + ec * 16;
  };
  auto w_img = [&](int ew) -> unsigned {
    return OFF_WE + (ew & ~1) * WPIECE + ((ew & 1) * 4 + ((wave >> 1) & 3)) * 1024 + ((wave ^ (wave >> 1)) & 1) * 128;
  };
  unsigned e_off = isx ? (ek * MS + (ex & 63) * 8 + er) * KS + (ex >> 6) * 128 + ec * 16 : w_off(GRID - 1 - t);
  unsigned c_off = OFF_CNT + t * 128;
  asm volatile("" : "+v"(e_off), "+v"(c_off));
  asm volatile("s_waitcnt lgkmcnt(0)" : "+s"(ka), "+s"(kb_), "+s"(kc));
  const u64 ax = ka[0], aw_ = ka[1], asx = ka[2], asw = ka[3], aout = ka[4];
  const u64 w0 = ka[5], w1 = ka[6], w2 = ka[7], w3 = kb_[0], w4 = kb_[1], w5 = kb_[2], w6 = kb_[3], w7 = kc[0];
  const int rank = (int)kc[1];
  u32x4 ev;
  if (!comm) ev = *(const GLB_AS u32x4*)((const gchar*)(isx ? ax : aw_) + ((e_off + (isx ? rank * (MS * KS) : 0)) & (M_ * KS - 1)));
  const unsigned cnt_raw = __hip_atomic_load((gu32*)((gchar*)w0 + c_off), __ATOMIC_RELAXED, __HIP_MEMORY_SCOPE_AGENT);
#define ws_of(k) sel8((k), w0, w1, w2, w3, w4, w5, w6, w7)
  unsigned warm = 0;
  if (comm && isx)
    warm = ld4(mk_rsrc(ws_of(wave), OFF_CNT), wave ? (lane ? 0x80000000u : OFF_XE + rank * XPEER + t * 128)
                                                   : (lane < NR ? OFF_XE + lane * XPEER + t * 128 : 0x80000000u), 0);
  const gchar* const px = (const gchar*)ax;
  const gchar* const pw = (const gchar*)aw_;
  gchar* const myws = (gchar*)w0;
  gu32* const cnt = (gu32*)(myws + OFF_CNT + t * 128);
  gu32* const tmo = (gu32*)(myws + OFF_TMO);
  const int fr = lane & 15, fg = lane >> 4;
  const int xs = (rank + wave) & (NR - 1);
  const bool push = isx && ex < XPUSH;
  const unsigned long long t0 = __builtin_amdgcn_s_memrealtime();
  const auto my_rs = mk_rsrc(myws, OFF_CNT);
  const unsigned e_lane = (ec >> 2) * (isx ? 1024 : WPIECE) + (ec & 3) * 256 + er * 16;
  unsigned e_img = e_lane + (isx ? (ex >> 1) * XFRAG + (ex & 1) * 128 + (push ? OFF_X + rank * XPEER : OFF_XE + ((rank + ek) & (NR - 1)) * XPEER)
                                 : w_img(GRID - 1 - t));
  u64 e_base = (u64)ws_of(push ? ek : 0);
  asm volatile("" : "+s"(e_base));
  asm volatile("" : "+v"(e_img));
  const unsigned call = rfl(cnt_raw) + 1;
  if (!comm) {
    st16(nonzero(ev), mk_rsrc((gchar*)e_base, OFF_CNT), e_img + (call & 3) * (isx ? XSRC : WSRC));
    if (!isx && e < NCOMM) {
      const u32x4 ev2 = *(const GLB_AS u32x4*)(pw + w_off(GRID - NCOMM + e));
      st16(nonzero(ev2), my_rs, e_lane + w_img(GRID - NCOMM + e) + (call & 3) * WSRC);
    }
  }
  __builtin_amdgcn_s_setprio(0);

  const int xcd = t & 7, idx = (t - NCOMM) >> 3;
  const int ti = idx & 3, tu = idx >> 2;
  const int tl = 8 + tu * 8 + xcd;
  const int joff = (rank & 3) * 16 + (rank >> 2) * 8;
  const int tj = (tl + joff) & (NJ - 1);
  const int tx = (xcd + joff) & (NJ - 1);
  const bool extra = tu < 4;
  const bool win = tl >= NJ - WIN;
  const int kbs = tu == 4 ? 2 : rank >> 2;
  auto kb_of = [&](int r) { return !win ? r : r == NKB - 1 ? kbs : r + (r >= kbs); };
  volatile LDS_AS unsigned* ctl = (volatile LDS_AS unsigned*)(lds + LDS_CTL);
  volatile LDS_AS unsigned* ready = ctl;
  volatile LDS_AS unsigned* abortf = ctl + 8;
  volatile LDS_AS unsigned* invdone = ctl + 9;
  volatile LDS_AS unsigned* ownrdy = ctl + 12;
  volatile LDS_AS unsigned* consumed = ctl + 16;
  volatile LDS_AS unsigned* sched = ctl + 80;

  if (wave == 8) {
    if (lane < 16) ctl[lane] = 0;
    consumed[lane] = 0;
    asm volatile("s_waitcnt lgkmcnt(0)" ::: "memory");
  }
  asm volatile("" :: "s"(call));
  __builtin_amdgcn_s_barrier();
  if (tid == 0) __hip_atomic_store(cnt, call, __ATOMIC_RELAXED, __HIP_MEMORY_SCOPE_AGENT);
  auto rezero3 = [&]() {
    const u32x4 z = {0u, 0u, 0u, 0u};
    st16(z, my_rs, OFF_X + ((call + 2) & 3) * XSRC + xs * XPEER + t * 1024 + lane * 16);
    st16(z, my_rs, OFF_XE + ((call + 2) & 3) * XSRC + xs * XPEER + t * 1024 + lane * 16);
    st16(z, my_rs, OFF_WE + ((call + 2) & 3) * WSRC + t * WPIECE + tid * 16);
  };

  if (comm) {
    if (wave < 8) {
      asm volatile("" :: "v"(warm));
      if (wave < 2) {
        const int mk = 1 + min(t * PWAVES + wave * (PWAVES - 1), NPULL * (NR - 1) - 1) / NPULL;
        const unsigned b0 = OFF_X + (call & 3) * XSRC + ((rank + mk) & (NR - 1)) * XPEER;
        const unsigned se = ((lane & 15) * 4 + (t & 3)) * XPUSH / 64;
        for (unsigned n = 1;; ++n) {
          const unsigned f = ld4(my_rs, b0 + (se >> 1) * XFRAG + (se & 1) * 128, 0);
          if (__builtin_popcountll(__builtin_amdgcn_ballot_w64(f != 0)) >= ARRC) break;
          if ((n & 63) == 0 && expired(tmo, call, t0)) break;
        }
        if (lane == 0) ctl[mk] = 1;
      }
      __builtin_amdgcn_s_barrier();
      rezero3();
      if (t == 0 && wave == 0 && lane < 2 * NR)
        __hip_atomic_store((gu32*)(myws + OFF_TKT + ((call + 2) & 3) * 8 * 128 + (lane < NR ? lane * 128 : (lane - NR) * 4)),
                           0u, __ATOMIC_RELAXED, __HIP_MEMORY_SCOPE_AGENT);
      return;
    }
    const int pwv = wave - 8;
    const int g = t * PWAVES + pwv;
    if (pwv >= PWAVES || g >= NPULL * (NR - 1)) return;
    const int k = 1 + g / NPULL, q = g % NPULL;
    const int s = (rank + k) & (NR - 1);
    const u64 sp = (u64)ws_of(k);
    const u32x4 srd_s = {rfl((unsigned)sp), rfl((unsigned)(sp >> 32) & 0xffffu), OFF_CNT, 0x00020000u};
    const u32x4 srd_d = {rfl((unsigned)w0), rfl((unsigned)(w0 >> 32) & 0xffffu), OFF_CNT, 0x00020000u};
    constexpr unsigned INV = 1u << 20;
    u64 ktab = 0;
#pragma unroll
    for (int b = 0; b < 32 - WIN / 8; ++b) {
      const int r = b >> 3, lh = b & 7;
      ktab |= (u64)(lh >= 6 ? r + (r >= (rank >> 2)) : lh * 8 >= NJ - WIN ? r + (r >= 2) : r) << (2 * b);
    }
    unsigned status = 0;
    const unsigned sx0 = OFF_XE + (call & 3) * XSRC + rank * XPEER + XPUSH * 1024, sw0 = OFF_WE + (call & 3) * WSRC;
    const unsigned dx0 = OFF_X + (call & 3) * XSRC + s * XPEER + XPUSH * 1024, dw0 = OFF_W + (call & 1) * WSTAGE + s * WSRC;
    const unsigned tkt = OFF_TKT + ((call & 3) * 8 + k) * 128;
    while (!rfl(ctl[k])) __builtin_amdgcn_s_sleep(1);
    {
      unsigned cur, nxt, s1, s2;
      u64 tnow;
#define VERIFY(N, X0, X1, X2, X3, L) \
      "s_cmp_lt_u32 %[cur], %[npair]\n\t" \
      "s_cbranch_scc0 .Lok" L "_%=\n\t" \
      "s_waitcnt vmcnt(" N ")\n\t" \
      ".Lchk" L "_%=:\n\t" \
      "v_min_u32 v121, " X2 ", " X3 "\n\t" \
      "v_min3_u32 v121, " X0 ", " X1 ", v121\n\t" \
      "v_cmp_eq_u32 vcc, 0, v121\n\t" \
      "s_nop 0\n\t" \
      "s_cbranch_vccnz .Lretry" L "_%=\n\t" \
      ".Lok" L "_%=:\n\t"
#define RETRY(X, OFS, L) \
      ".Lretry" L "_%=:\n\t" \
      "s_sleep 8\n\t" \
      "s_memrealtime %[tnow]\n\t" \
      "s_waitcnt lgkmcnt(0)\n\t" \
      "v_mov_b64 v[122:123], %[tnow]\n\t" \
      "v_cmp_lt_u64 vcc, %[tend], v[122:123]\n\t" \
      "s_nop 0\n\t" \
      "s_cbranch_vccnz .Lfail_%=\n\t" \
      "buffer_load_dwordx4 " X ", v119, %[srs], 0 offen " OFS " sc0 sc1\n\t" \
      "s_waitcnt vmcnt(0)\n\t" \
      "s_branch .Lchk" L "_%=\n\t"
#define HALF(XA, XA0, XA1, XA2, XA3, XB, XB0, XB1, XB2, XB3, YA, YB, TIN, TOUT, LA, LB) \
      VERIFY("3", XA0, XA1, XA2, XA3, LA) \
      "v_readfirstlane_b32 %[nxt], " TIN "\n\t" \
      "s_mov_b64 exec, 1\n\t" \
      "global_atomic_add " TOUT ", %[tkt], %[inc], %[ws] sc0\n\t" \
      "s_mov_b64 exec, -1\n\t" \
      "s_add_u32 %[nxt], %[nxt], %[base]\n\t" \
      "s_sub_u32 %[s1], %[nxt], %[xpair]\n\t" \
      "s_add_u32 %[s2], %[s1], %[rot]\n\t" \
      "s_and_b32 %[s2], %[s2], 31\n\t" \
      "s_add_u32 vcc_lo, %[s1], %[joff]\n\t" \
      "s_and_b32 vcc_lo, vcc_lo, 0xe0\n\t" \
      "s_or_b32 %[s2], %[s2], vcc_lo\n\t" \
      "s_lshr_b32 %[s1], %[s1], 5\n\t" \
      "s_lshl_b32 %[s1], %[s1], 1\n\t" \
      "s_lshr_b64 vcc, %[ktab], %[s1]\n\t" \
      "s_and_b32 %[s1], vcc_lo, 3\n\t" \
      "s_lshl_b32 %[s1], %[s1], 8\n\t" \
      "s_or_b32 %[s2], %[s2], %[s1]\n\t" \
      "s_cmp_lt_u32 %[nxt], %[xpair]\n\t" \
      "s_cselect_b32 %[s2], %[nxt], %[s2]\n\t" \
      "s_cselect_b32 %[s1], %[sx], %[sw]\n\t" \
      "s_lshl_b32 %[s2], %[s2], 11\n\t" \
      "s_add_u32 %[s1], %[s1], %[s2]\n\t" \
      "s_cmp_lt_u32 %[nxt], %[npair]\n\t" \
      "s_cselect_b32 %[s1], %[s1], 0x80000000\n\t" \
      "v_add_u32 v118, %[s1], %[lo]\n\t" \
      "buffer_load_dwordx4 " YA ", v118, %[srs], 0 offen sc0 sc1\n\t" \
      "buffer_store_dwordx4 " XA ", v120, %[srd], 0 offen sc0 sc1\n\t" \
      VERIFY("4", XB0, XB1, XB2, XB3, LB) \
      "buffer_load_dwordx4 " YB ", v118, %[srs], 0 offen offset:1024 sc0 sc1\n\t" \
      "buffer_store_dwordx4 " XB ", v120, %[srd], 0 offen offset:1024 sc0 sc1\n\t" \
      "s_cmp_lt_u32 %[nxt], %[xpair]\n\t" \
      "s_cselect_b32 %[s1], %[dx], %[dw]\n\t" \
      "s_add_u32 %[s1], %[s1], %[s2]\n\t" \
      "s_cmp_lt_u32 %[nxt], %[npair]\n\t" \
      "s_cselect_b32 %[s1], %[s1], 0x80000000\n\t" \
      "v_mov_b32 v119, v118\n\t" \
      "v_add_u32 v120, %[s1], %[lo]\n\t" \
      "s_mov_b32 %[cur], %[nxt]\n\t"
      asm volatile(
          "s_mov_b32 %[cur], %[inv]\n\t"
          "v_mov_b32 v117, %[first]\n\t"
          "v_mov_b32 v120, 0x80000000\n\t"
          "s_nop 4\n\t"
          ".Lloop_%=:\n\t"
          HALF("v[100:103]", "v100", "v101", "v102", "v103", "v[104:107]", "v104", "v105", "v106", "v107",
               "v[108:111]", "v[112:115]", "v117", "v116", "a", "b")
          "s_cmp_ge_u32 %[cur], %[npair]\n\t"
          "s_cbranch_scc1 .Ldone_%=\n\t"
          HALF("v[108:111]", "v108", "v109", "v110", "v111", "v[112:115]", "v112", "v113", "v114", "v115",
               "v[100:103]", "v[104:107]", "v116", "v117", "c", "d")
          "s_cmp_lt_u32 %[cur], %[npair]\n\t"
          "s_cbranch_scc1 .Lloop_%=\n\t"
          "s_branch .Ldone_%=\n\t"
          RETRY("v[100:103]", "", "a")
          RETRY("v[104:107]", "offset:1024", "b")
          RETRY("v[108:111]", "", "c")
          RETRY("v[112:115]", "offset:1024", "d")
          ".Lfail_%=:\n\t"
          "s_mov_b32 %[status], 1\n\t"
          ".Ldone_%=:\n\t"
          "s_waitcnt vmcnt(0)"
          : [cur] "=&s"(cur), [nxt] "=&s"(nxt), [s1] "=&s"(s1), [s2] "=&s"(s2), [tnow] "=&s"(tnow), [status] "+s"(status)
          : [srs] "s"(srd_s), [srd] "s"(srd_d), [ws] "s"(w0), [tend] "s"(t0 + DEADLINE_TICKS),
            [sx] "s"(rfl(sx0)), [sw] "s"(rfl(sw0)), [dx] "s"(rfl(dx0)), [dw] "s"(rfl(dw0)),
            [base] "n"(NPULL), [first] "s"(rfl((unsigned)(q - NPULL))),
            [ktab] "s"(ktab), [rot] "s"(rfl(k * 4)), [joff] "s"(rfl(joff * 4)), [npair] "n"(NPAIR), [xpair] "n"(XU / 2), [inv] "n"(INV),
            [tkt] "v"(tkt), [inc] "v"(1u), [lo] "v"(lane * 16)
          : "memory", "vcc", "scc", "v100", "v101", "v102", "v103", "v104", "v105", "v106", "v107", "v108", "v109",
            "v110", "v111", "v112", "v113", "v114", "v115", "v116", "v117", "v118", "v119", "v120", "v121", "v122", "v123");
    }
    if (status) __hip_atomic_store(tmo, call, __ATOMIC_RELAXED, __HIP_MEMORY_SCOPE_AGENT);
    return;
  }

  if (wave == 8) {
    sleep_until(t0, OWN0);
    const gchar* w_even = pw + (size_t)fr * N_ + fg * 16 + tj * 64;
    const gchar* w_odd = pw + (size_t)(fr ^ 8) * N_ + fg * 16 + tj * 64;
    const gchar* w_ex = pw + (size_t)(fg * 16 + (fr ^ ((fg & 1) << 3))) * N_ + tx * 64 + tu * 16;
    for (int kb = 0; kb < NKB; ++kb) {
#pragma unroll
      for (int v = 0; v < 8; ++v)
        __builtin_amdgcn_global_load_lds((const GLB_AS void*)(((v & 1) ? w_odd : w_even) + (size_t)(kb * 128 + v * 16) * N_),
                                         (LDS_AS void*)(lds + LDS_OWN + kb * SLOT + v * 1024), 16, 0, 0);
      if (extra) {
        __builtin_amdgcn_global_load_lds((const GLB_AS void*)(w_ex + (size_t)(kb * 128) * N_),
                                         (LDS_AS void*)(lds + LDS_OWN + kb * SLOT + 8192), 16, 0, 0);
        __builtin_amdgcn_global_load_lds((const GLB_AS void*)(w_ex + (size_t)(kb * 128 + 64) * N_),
                                         (LDS_AS void*)(lds + LDS_OWN + kb * SLOT + 9216), 16, 0, 0);
      }
      wait_vm<0>();
      if (lane == 0) ownrdy[kb] = 1;
    }
    gu32* const tk = (gu32*)(myws + OFF_TKT + (call & 3) * 8 * 128);
    if (win) {
      sleep_until(t0, 1000 + (GRID - 1 - t) * 15);
      __builtin_amdgcn_fence(__ATOMIC_ACQUIRE, "");
      wait_vm<0>();
      for (unsigned it = 1;; ++it) {
        const unsigned f = __hip_atomic_load(tk + (lane & 7), __ATOMIC_RELAXED, __HIP_MEMORY_SCOPE_AGENT);
        if (__builtin_amdgcn_ballot_w64(f == 0) == 0) {
          sched[lane & 7] = f;
          asm volatile("s_waitcnt lgkmcnt(0)" ::: "memory");
          if (lane == 0) *invdone = 1;
          break;
        }
        if ((it & 63) == 0 && expired(tmo, call, t0)) { if (lane == 0) *abortf = 1; break; }
        __builtin_amdgcn_s_sleep(8);
      }
    } else if (t == NCOMM + 3 * 32) {
      for (unsigned it = 1;; ++it) {
        const unsigned ta = (unsigned)__builtin_amdgcn_s_memrealtime();
        const unsigned f = __hip_atomic_load(tk + (1 + lane % 7) * 32, __ATOMIC_RELAXED, __HIP_MEMORY_SCOPE_AGENT);
        unsigned mx = 0;
#pragma unroll
        for (int k = 0; k < NR - 1; ++k) mx = max(mx, (unsigned)__builtin_amdgcn_readlane(f, k));
        const unsigned tb = (unsigned)__builtin_amdgcn_s_memrealtime();
        if (mx + SNAP >= NPAIR - NPULL && (tb - ta < 80 || mx + SNAP / 2 >= NPAIR - NPULL)) {
          const int left = max((int)(NPAIR - NPULL) - (int)mx, 0);
          const unsigned fmin = ta + 25 + left * SLOT_T / 40 + FOFF;
          const unsigned lag = __builtin_amdgcn_update_dpp(0u, f ? min((mx - f + 1) / 2, 80u) + 1 : 1000u, 0x111  , 0xf, 0xf, false);
          if (lane < NR) __hip_atomic_store(tk + lane, lane ? lag : fmin | 1u, __ATOMIC_RELAXED, __HIP_MEMORY_SCOPE_AGENT);
          break;
        }
        if ((it & 63) == 0 && expired(tmo, call, t0)) break;
        __builtin_amdgcn_s_sleep(8);
      }
    }
    return;
  }

  if (wave > 8) {
    const int sv = wave - 8;
    const int s = (rank + sv) & (NR - 1);
    const unsigned ex_lane = fg * 1024 + tu * 256 + fr * 16;
    sleep_until(t0, OWN0);
    const unsigned zo = OFF_W + ((call + 1) & 1) * WSTAGE + s * WSRC + ti * NJ * WPIECE;
    auto rezero = [&](int part) {
      const u32x4 z = {0u, 0u, 0u, 0u};
      st16(z, my_rs, zo + tj * WPIECE + lane * 16 + part * 3072);
      st16(z, my_rs, zo + tj * WPIECE + lane * 16 + part * 3072 + 1024);
      if (part < 2) {
        st16(z, my_rs, zo + tj * WPIECE + lane * 16 + part * 3072 + 2048);
        if (extra) st16(z, my_rs, zo + tx * WPIECE + ex_lane + part * 4096);
      }
    };
    LDS_AS unsigned char* slot = (LDS_AS unsigned char*)lds + (sv - 1) * SLOT + lane * 16;
    const unsigned sb = OFF_W + (call & 1) * WSTAGE + s * WSRC;
    auto slot_free = [&](unsigned kb) {
      while (__builtin_amdgcn_ballot_w64(consumed[sv * 8 + (lane & 7)] < kb) && !rfl(*abortf)) __builtin_amdgcn_s_sleep(2);
    };
    for (int kb = 0; kb < (win ? NKB - 1 : NKB); ++kb) {
      const unsigned so = sb + (kb_of(kb) * NJ + tj) * WPIECE, so_ex = sb + (kb * NJ + tx) * WPIECE;
      bool dead = false;
      for (unsigned it = 1;; ++it) {
        if (rfl(ld4(my_rs, 0, so)) != 0) break;
        if ((it & 63) == 0 && expired(tmo, call, t0)) { dead = true; break; }
        __builtin_amdgcn_s_sleep(2);
      }
      u32x4 d[10];
      for (unsigned it = 1; !dead; ++it) {
        asm volatile("" ::: "memory");
        unsigned m = 0xffffffffu;
#pragma unroll
        for (int v = 0; v < 8; ++v) d[v] = ld16<AUX_SYS>(my_rs, lane * 16 + v * 1024, so);
        if (extra) {
          d[8] = ld16<AUX_SYS>(my_rs, ex_lane, so_ex);
          d[9] = ld16<AUX_SYS>(my_rs, ex_lane + 4096, so_ex);
          m = min(min4(d[8]), min4(d[9]));
        }
#pragma unroll
        for (int v = 0; v < 8; ++v) m = min(m, min4(d[v]));
        if (__builtin_amdgcn_ballot_w64(m == 0) == 0) break;
        if ((it & 63) == 0 && expired(tmo, call, t0)) dead = true;
      }
      if (dead) { if (lane == 0) *abortf = 1; return; }
      slot_free(kb);
#pragma unroll
      for (int v = 0; v < 8; ++v) *(LDS_AS u32x4*)(slot + v * 1024) = d[v];
      if (extra) { *(LDS_AS u32x4*)(slot + 8192) = d[8]; *(LDS_AS u32x4*)(slot + 9216) = d[9]; }
      asm volatile("s_waitcnt lgkmcnt(0)" ::: "memory");
      if (lane == 0) ready[sv] = kb + 1;
      if (kb < NKB - 1) rezero(kb);
      if (kb == 1 && ti == 0) {
        const unsigned f = ld4(my_rs, lane * 128, sb + ((NKB - 1) * NJ + tj) * WPIECE);
        asm volatile("" :: "v"(f));
      }
    }
    if (!win) return;
    while (!rfl(*invdone) && !rfl(*abortf)) __builtin_amdgcn_s_sleep(2);
    if (rfl(*abortf)) return;
    const unsigned po = (kbs * NJ + tj) * WPIECE, eo = OFF_WE + (call & 3) * WSRC + po;
    const unsigned sc_l = sched[lane & 7];
    int c0 = 0, c1 = 0;
#pragma unroll
    for (int k = 1; k < NR; ++k) {
      const int ak = (int)__builtin_amdgcn_readlane(sc_l, k) - 1;
      c0 += max(lane - ak, 0);
      c1 += max(lane + 64 - ak, 0);
    }
    constexpr int NJOB = 2 * WIN;
    const int base = __builtin_popcountll(__builtin_amdgcn_ballot_w64(c0 < NJOB * (NR - 1))) +
                     __builtin_popcountll(__builtin_amdgcn_ballot_w64(c1 < NJOB * (NR - 1))) - NJOB;
    int shed = 0;
#pragma unroll
    for (int k = 1; k < NR - 1; ++k)
      if (k < sv) shed += min(max((int)__builtin_amdgcn_readlane(sc_l, k) - 1 - base, 0), NJOB);
    unsigned fire[2], voff[2];
    u64 via[2];
#pragma unroll
    for (int h = 0; h < 2; ++h) {
      const int i = h * WIN + (unsigned)(tl - sv * (WIN / 8)) % WIN;
      int row = base + i, lk = sv;
      if (row < (int)__builtin_amdgcn_readlane(sc_l, sv) - 1) {
        const int J = shed + i;
        row = __builtin_popcountll(__builtin_amdgcn_ballot_w64(c0 <= J)) + __builtin_popcountll(__builtin_amdgcn_ballot_w64(c1 <= J)) - 1;
        const int p = J - (int)(row < 64 ? __builtin_amdgcn_readlane(c0, row & 63) : __builtin_amdgcn_readlane(c1, row & 63));
        const unsigned act = (unsigned)__builtin_amdgcn_ballot_w64((int)sc_l - 1 <= row) & 0xfeu;
#pragma unroll
        for (int k = NR - 1; k >= 1; --k)
          if (((act >> k) & 1u) && __builtin_popcount(act & ((1u << k) - 1u)) == p) lk = k;
      }
      fire[h] = (unsigned)__builtin_amdgcn_readlane(sc_l, 0) + row * SLOT_T / 20;
      via[h] = (u64)ws_of(lk);
      voff[h] = (lk == sv ? eo : sb + po) + h * 512;
      asm volatile("" : "+s"(fire[h]), "+s"(via[h]), "+s"(voff[h]));
    }
    slot_free(NKB - 1);
    const unsigned hl = (lane >> 5) * 1024 + (lane & 31) * 16;
    LDS_AS unsigned char* hslot = (LDS_AS unsigned char*)lds + (sv - 1) * SLOT + hl;
#pragma nounroll
    for (unsigned h = 0; h < 2; ++h) {
      asm volatile("" : "+s"(h));
      const unsigned fi = h ? fire[1] : fire[0], vo = h ? voff[1] : voff[0];
      const auto vrs = mk_rsrc((gchar*)(h ? via[1] : via[0]), OFF_CNT);
      while ((int)(fi - (unsigned)__builtin_amdgcn_s_memrealtime()) > 0) __builtin_amdgcn_s_sleep(1);
      u32x4 d[4];
#pragma unroll
      for (int v = 0; v < 4; ++v) d[v] = ld16<0>(vrs, hl + v * 2048, vo);
      unsigned m = 0xffffffffu;
#pragma unroll
      for (int v = 0; v < 4; ++v) {
        *(LDS_AS u32x4*)(hslot + h * 512 + v * 2048) = d[v];
        m = min(m, min4(d[v]));
        __builtin_amdgcn_sched_barrier(0);
      }
      for (unsigned it = 1; __builtin_expect(__builtin_amdgcn_ballot_w64(m == 0) != 0, 0); ++it) {
        if ((it & 63) == 0 && expired(tmo, call, t0)) { if (lane == 0) *abortf = 1; return; }
        m = 0xffffffffu;
#pragma unroll
        for (int v = 0; v < 4; ++v) {
          const u32x4 r = ld16<AUX_SYS>(mk_rsrc(ws_of(sv), OFF_CNT), hl + v * 2048, eo + h * 512);
          *(LDS_AS u32x4*)(hslot + h * 512 + v * 2048) = r;
          m = min(m, min4(r));
        }
      }
      asm volatile("s_waitcnt lgkmcnt(0)" ::: "memory");
      if (lane == 0) ready[sv] = NKB + h;
    }
    return;
  }

  sleep_until(t0, OWN0);
  const int mb = ti * 8 + wave;
  const auto x_own_rs = mk_rsrc(px, M_ * KS);
  const unsigned x_own_lane = (unsigned)(rank * MS + mb * 16 + fr) * KS + fg * 16;
  const unsigned x_st_lane = OFF_X + (call & 3) * XSRC + mb * XFRAG + lane * 16;
  float sc = __builtin_bit_cast(float, rfl(__builtin_bit_cast(unsigned, *(const GLB_AS float*)asx * *(const GLB_AS float*)asw)));
  asm volatile("" : "+s"(sc));
  auto ldx_own = [&](unsigned kb) {
    const u32x4 lo = ld16<0>(x_own_rs, x_own_lane, kb * 128);
    const u32x4 hi = ld16<0>(x_own_rs, x_own_lane + 64, kb * 128);
    return __builtin_shufflevector(lo, hi, 0, 1, 2, 3, 4, 5, 6, 7);
  };
  auto ldx_st = [&](int sv, unsigned kb) {
    const unsigned so = ((rank + sv) & (NR - 1)) * XPEER + kb * 32 * XFRAG;
    const u32x4 lo = ld16<AUX_SYS>(my_rs, x_st_lane, so);
    const u32x4 hi = ld16<AUX_SYS>(my_rs, x_st_lane + 1024, so);
    return __builtin_shufflevector(lo, hi, 0, 1, 2, 3, 4, 5, 6, 7);
  };
  const unsigned tr_lane = fg * 1024 + ((fg & 1) << 7) + fr * 8;

  f32x4 acc[5] = {};
  bool failed = false;
  auto step = [&](unsigned base, u32x8 xv) {
    const i32x8 bx = __builtin_bit_cast(i32x8, xv);
    unsigned tl = tr_lane;
    asm volatile("" : "+v"(tl));
    LDS_AS unsigned char* sl = (LDS_AS unsigned char*)lds + base;
    if (extra) {
      i32x8 aw;
      const unsigned te = tl - fg * 768 + 8192;
#pragma unroll
      for (int c = 0; c < 4; ++c) {
        const i32x2 r = __builtin_amdgcn_ds_read_tr8_b64_v2i32((LDS_AS i32x2*)(sl + (c >> 1) * 1024 + (te ^ ((c & 1) << 7))));
        aw[2 * c] = r[0]; aw[2 * c + 1] = r[1];
      }
      acc[4] = __builtin_amdgcn_mfma_scale_f32_16x16x128_f8f6f4(aw, bx, acc[4], 0, 0, 0, 0x7f7f7f7f, 0, 0x7f7f7f7f);
    }
#pragma unroll
    for (int q = 0; q < 4; ++q) {
      i32x8 aw;
#pragma unroll
      for (int c = 0; c < 4; ++c) {
        const i32x2 r = __builtin_amdgcn_ds_read_tr8_b64_v2i32((LDS_AS i32x2*)(sl + (c >> 1) * 4096 + q * 256 + (tl ^ ((c & 1) << 7))));
        aw[2 * c] = r[0]; aw[2 * c + 1] = r[1];
      }
      acc[q] = __builtin_amdgcn_mfma_scale_f32_16x16x128_f8f6f4(aw, bx, acc[q], 0, 0, 0, 0x7f7f7f7f, 0, 0x7f7f7f7f);
    }
  };
#pragma unroll
  for (unsigned kb = 0; kb < NKB; ++kb) {
    const u32x8 xv = ldx_own(kb);
    while (!rfl(ownrdy[kb])) __builtin_amdgcn_s_sleep(1);
    step(LDS_OWN + kb * SLOT, xv);
  }
  while (!((unsigned)__builtin_amdgcn_ballot_w64(ctl[lane & 15] != 0u) & 0x1feu)) __builtin_amdgcn_s_sleep(4);
  rezero3();
  u32x8 xf[NR];
  auto load_x = [&](unsigned kb) {
    for (unsigned it = 1;; ++it) {
      asm volatile("" ::: "memory");
      unsigned m = 0xffffffffu;
#pragma unroll
      for (int sv = 1; sv < NR; ++sv) xf[sv] = ldx_st(sv, kb);
#pragma unroll
      for (int sv = 1; sv < NR; ++sv) m = min(m, min8(xf[sv]));
      if (__builtin_amdgcn_ballot_w64(m == 0) == 0) break;
      if ((it & 15) == 0 && expired(tmo, call, t0)) { failed = true; break; }
      __builtin_amdgcn_s_sleep(32);
    }
  };
  auto next_ready = [&](unsigned& pend, unsigned level) {
    unsigned rdy;
    for (;;) {
      const unsigned rv = ctl[lane & 15];
      rdy = (unsigned)__builtin_amdgcn_ballot_w64(rv > ((lane & 15) < 8 ? level : 0u)) & (pend | 0x100u);
      if (rdy) break;
      __builtin_amdgcn_s_sleep(1);
    }
    pend &= ~rdy;
    if (rdy & 0x100u) { failed = true; pend = 0; rdy = 0; }
    return rdy;
  };
  for (unsigned kb = 0; kb < (win ? NKB - 1 : NKB) && !failed; ++kb) {
    load_x(kb_of(kb));
    for (unsigned pend = failed ? 0u : 0xfeu; pend;) {
      const unsigned rdy = next_ready(pend, kb);
#pragma unroll
      for (int sv = 1; sv < NR; ++sv)
        if (rdy & (1u << sv)) {
          step((sv - 1) * SLOT, xf[sv]);
          asm volatile("s_waitcnt lgkmcnt(0)" ::: "memory");
          consumed[sv * 8 + wave] = kb + 1;
        }
    }
  }
  if (win && !failed) load_x(kbs);
  const auto o_rs = mk_rsrc((gchar*)aout, MS * N_ * 4);
  constexpr float LOG2E = 1.4426950408889634f;
  auto silu4 = [&](f32x4 a) {
    const f32x4 y = a * (failed ? __builtin_nanf("") : sc);
    f32x4 z = y * -LOG2E;
#pragma unroll
    for (int e = 0; e < 4; ++e) z[e] = __builtin_amdgcn_exp2f(__builtin_amdgcn_fmed3f(z[e], -60.0f * LOG2E, 60.0f * LOG2E));
    z += 1.0f;
#pragma unroll
    for (int e = 0; e < 4; ++e) z[e] = __builtin_amdgcn_rcpf(z[e]);
    return __builtin_bit_cast(u32x4, y * z);
  };
  if (extra) st16(silu4(acc[4]), o_rs, ((unsigned)(mb * 16 + fr) * N_ + fg * 4 + tx * 64 + tu * 16) * 4);
#pragma nounroll
  for (unsigned h = 0; h < 2; ++h) {
    asm volatile("" : "+s"(h));
    for (unsigned pend = win && !failed ? 0xfeu : 0u; pend;) {
      const unsigned rdy = next_ready(pend, NKB - 1 + h);
#pragma unroll
      for (int sv = 1; sv < NR; ++sv)
        if (rdy & (1u << sv)) {
          const i32x8 bx = __builtin_bit_cast(i32x8, xf[sv]);
          unsigned tl2 = tr_lane + h * 512;
          asm volatile("" : "+v"(tl2));
          LDS_AS unsigned char* sl = (LDS_AS unsigned char*)lds + (sv - 1) * SLOT;
#pragma unroll
          for (int q = 0; q < 2; ++q) {
            i32x8 aw;
#pragma unroll
            for (int c = 0; c < 4; ++c) {
              const i32x2 r = __builtin_amdgcn_ds_read_tr8_b64_v2i32((LDS_AS i32x2*)(sl + (c >> 1) * 4096 + q * 256 + (tl2 ^ ((c & 1) << 7))));
              aw[2 * c] = r[0]; aw[2 * c + 1] = r[1];
            }
            acc[q] = __builtin_amdgcn_mfma_scale_f32_16x16x128_f8f6f4(aw, bx, acc[q], 0, 0, 0, 0x7f7f7f7f, 0, 0x7f7f7f7f);
          }
        }
    }
    if (failed && lane == 0) *abortf = 1;
    const unsigned o_pair = ((unsigned)(mb * 16 + (fr & 7)) * N_ + (fr >> 3) * 16 + fg * 4 + tj * 64 + h * 32) * 4;
    const u32x4 o0 = silu4(acc[0]), o1 = silu4(acc[1]);
    u32x4 lo, hi;
#pragma unroll
    for (int e = 0; e < 4; ++e) {
      lo[e] = __builtin_amdgcn_update_dpp(o0[e], o1[e], 0x128  , 0xf, 0xc, false);
      hi[e] = __builtin_amdgcn_update_dpp(o1[e], o0[e], 0x128  , 0xf, 0x3, false);
    }
    st16(lo, o_rs, o_pair);
    st16(hi, o_rs, o_pair + 8 * N_ * 4);
    acc[0] = acc[2];
    acc[1] = acc[3];
  }
}

}

extern "C" void kernel_launch(int rank, int nranks, ncclComm_t comm,
                              hipStream_t stream,
                              void* const* d_in, const int* in_sizes, int n_in,
                              void* d_out, int out_size,
                              const int* in_dtypes, int out_dtype,
                              void* const* d_out_all,
                              void* const* d_ws_all, size_t ws_bytes) {
  if (nranks != NR || (unsigned)rank >= (unsigned)NR || ws_bytes < WS_NEED || n_in < 4) {
    fprintf(stderr, "kernel_launch: unexpected configuration\n");
    return;
  }
  Params p;
  p.x = (const uint8_t*)d_in[0];
  p.w = (const uint8_t*)d_in[1];
  p.sx = (const float*)d_in[2];
  p.sw = (const float*)d_in[3];
  p.out = (float*)d_out;
  for (int i = 0; i < NR; ++i) p.ws[i] = (char*)d_ws_all[(rank + i) & (NR - 1)];
  p.rank = rank;
  p.pad = 0;
  hipLaunchKernelGGL(gemm_rs_kernel, dim3(GRID), dim3(1024), 0, stream, p);
}
```
